```python
import numpy as np

kernel_name = "gpu_a2a_gemm_m4096_k4096_n4096_e5m2e5m2_dqsilu"


ABSMAX_RTOL = 0.05

COMPUTE_DTYPE = "float32"
EPILOGUE = "dequant_silu"
IN_QUANT = None
OUT_QUANT = None
IN_FP8 = "e5m2e5m2"
IN_DTYPES = {"x": "e5m2", "w_mat": "e5m2",
             "scale_x": "float32", "scale_w": "float32"}
WORLD_SIZE = 8

def _snap_fp8(x, mant_bits, max_abs, min_normal):
    a = np.abs(x.astype(np.float64))
    _m, _e = np.frexp(a)
    q = np.where(a >= min_normal, np.ldexp(1.0, _e - 1 - mant_bits),
                 min_normal * 2.0 ** -mant_bits)
    snapped = np.minimum(np.round(a / q) * q, max_abs)
    return (np.sign(x) * snapped).astype(np.float32)

def setup_inputs():

    x = _snap_fp8(np.random.randn(4096, 4096).astype(np.float32), 2, 57344.0, 2.0 ** -14)
    w_mat = _snap_fp8(np.random.randn(4096, 4096).astype(np.float32), 2, 57344.0, 2.0 ** -14)
    scale_x = (0.02 * (1.0 + np.random.rand(1))).astype(np.float32)
    scale_w = (0.02 * (1.0 + np.random.rand(1))).astype(np.float32)
    return {"x": x, "w_mat": w_mat, "scale_x": scale_x, "scale_w": scale_w}

def reference(x, w_mat, scale_x, scale_w):
    acc = (x @ w_mat).astype(np.float32)
    y = (acc * (scale_x * scale_w)).astype(np.float32)
    return (y / (1.0 + np.exp(-np.clip(y, -60.0, 60.0)))).astype(np.float32)
```

```cpp
#include <hip/hip_runtime.h>
#include <rccl/rccl.h>
#include <cstdio>

#define HIP_CHECK(expr) do { hipError_t _e = (expr); if (_e != hipSuccess) { \
    fprintf(stderr, "HIP error %s at %s:%d\n", hipGetErrorString(_e), __FILE__, __LINE__); \
    return; } } while (0)

#define LAS __attribute__((address_space(3)))
#define GAS __attribute__((address_space(1)))
typedef unsigned u32x4 __attribute__((ext_vector_type(4)));
typedef int i32x2 __attribute__((ext_vector_type(2)));
typedef int i32x8 __attribute__((ext_vector_type(8)));
typedef float f32x4 __attribute__((ext_vector_type(4)));
typedef float f32x16 __attribute__((ext_vector_type(16)));

namespace {
constexpr int NRANK = 8;
constexpr int KS = 512;
constexpr int NN = 4096;
constexpr int NWG = 256;
constexpr size_t OFF_CNT = 0;
constexpr size_t OFF_TMO = 32768;
constexpr size_t OFF_FLAG = 65536;
constexpr size_t OFF_DBG = 262144;
constexpr size_t OFF_DATA = 1u << 20;
constexpr size_t SLOT_BYTES = 2u << 20;
constexpr size_t WS_NEED = OFF_DATA + 2 * SLOT_BYTES;
constexpr unsigned long long DEADLINE_TICKS = 100000000ull;
constexpr int RING = 4;
constexpr int WAVE_LDS = RING * 8192;
constexpr int CTRL_OFF = 4 * WAVE_LDS;
constexpr int LDS_TOTAL = CTRL_OFF + 64;
constexpr int DLOOK = 3;
constexpr int SWIN = 6;

struct Args {
  const unsigned char* x; const unsigned char* w; const float* sx; const float* sw; float* out;
  unsigned char* myws;
  int rank; int pad;
  unsigned char* peer[NRANK];
};

__device__ __forceinline__ unsigned long long rt() { return __builtin_amdgcn_s_memrealtime(); }
__device__ __forceinline__ unsigned rfl(unsigned v) { return __builtin_amdgcn_readfirstlane(v); }

constexpr int waitN(int step) {
  int ops = 1 + 8, posA[20] = {}, posB[24] = {};
  for (int j = 0; j <= RING; ++j) { ops += 8; posB[j] = ops; }
  for (int i = 0; i < 16; ++i) {
    if (i == 2) { for (int j = 2; j <= 2 + DLOOK; ++j) { ops += 4; posA[j] = ops; } }
    else if (i > 2 && i + DLOOK <= 15) { ops += 4; posA[i + DLOOK] = ops; }
    int need = posA[i];
    if (i < 15 && posB[i + 1] > need) need = posB[i + 1];
    if (i == step) return ops - need;
    if (i + 1 + RING <= 15) { ops += 8; posB[i + 1 + RING] = ops; }
  }
  return 0;
}
constexpr int sendN(int w) {
  int T = 32, s = w + SWIN, n = 0;
  for (int i = w + 1; i < (s < T ? s : T); ++i) ++n;
  for (int t = w + 1; t <= s - 1; ++t) if (t - SWIN >= 0 && (t - SWIN + 1) % 4 == 0) ++n;
  return n;
}

__device__ __forceinline__ void dma8(unsigned lds, unsigned voff, __amdgpu_buffer_rsrc_t rs, unsigned soff, unsigned step) {
  unsigned t;
  asm volatile(
      "s_mov_b32 m0, %1\n\ts_mov_b32 %0, %4\n\tbuffer_load_dwordx4 %2, %3, %0 offen lds\n\t"
      "s_add_u32 m0, m0, 0x400\n\ts_add_u32 %0, %0, %5\n\tbuffer_load_dwordx4 %2, %3, %0 offen lds\n\t"
      "s_add_u32 m0, m0, 0x400\n\ts_add_u32 %0, %0, %5\n\tbuffer_load_dwordx4 %2, %3, %0 offen lds\n\t"
      "s_add_u32 m0, m0, 0x400\n\ts_add_u32 %0, %0, %5\n\tbuffer_load_dwordx4 %2, %3, %0 offen lds\n\t"
      "s_add_u32 m0, m0, 0x400\n\ts_add_u32 %0, %0, %5\n\tbuffer_load_dwordx4 %2, %3, %0 offen lds\n\t"
      "s_add_u32 m0, m0, 0x400\n\ts_add_u32 %0, %0, %5\n\tbuffer_load_dwordx4 %2, %3, %0 offen lds\n\t"
      "s_add_u32 m0, m0, 0x400\n\ts_add_u32 %0, %0, %5\n\tbuffer_load_dwordx4 %2, %3, %0 offen lds\n\t"
      "s_add_u32 m0, m0, 0x400\n\ts_add_u32 %0, %0, %5\n\tbuffer_load_dwordx4 %2, %3, %0 offen lds"
      : "=&s"(t) : "s"(lds), "v"(voff), "s"(rs), "s"(soff), "s"(step) : "memory", "scc");
}
__device__ __forceinline__ i32x8 cat(u32x4 a, u32x4 b) {
  return __builtin_bit_cast(i32x8, __builtin_shufflevector(a, b, 0, 1, 2, 3, 4, 5, 6, 7));
}
__device__ __forceinline__ i32x8 read_b(LAS unsigned char* p) {
  i32x2 a = __builtin_amdgcn_ds_read_tr8_b64_v2i32((LAS i32x2*)(p));
  i32x2 b = __builtin_amdgcn_ds_read_tr8_b64_v2i32((LAS i32x2*)(p + 1024));
  i32x2 c = __builtin_amdgcn_ds_read_tr8_b64_v2i32((LAS i32x2*)(p + 2048));
  i32x2 d = __builtin_amdgcn_ds_read_tr8_b64_v2i32((LAS i32x2*)(p + 3072));
  i32x8 r; r[0] = a[0]; r[1] = a[1]; r[2] = b[0]; r[3] = b[1]; r[4] = c[0]; r[5] = c[1]; r[6] = d[0]; r[7] = d[1];
  return r;
}
#define SCHED() __builtin_amdgcn_sched_barrier(0)

struct CW {
  f32x16 acc[2][4];
  i32x8 bf[4];
  u32x4 af[DLOOK + 1][4];
  LAS unsigned char* rd[4];
  LAS unsigned* ctrl;
  __amdgpu_buffer_rsrc_t rsW, rsA;
  unsigned ring, voffW, voffA, n0, abase;
  int rank, wave; unsigned mt7;
};
__device__ __forceinline__ void step_id(const CW& c, int i, unsigned& src, unsigned& lvl) {
  if (i < 2) { src = c.rank; lvl = c.wave * 2 + i; return; }
  const unsigned q = 4u * (i - 2) + c.wave;
  lvl = q / 7u;
  unsigned s = q - lvl * 7u + c.mt7;
  s = s < s - 7u ? s : s - 7u;
  src = (c.rank + 1 + s) & 7u;
}
template <int J> __device__ __forceinline__ void issue_w(CW& c) {
  unsigned src, lvl; step_id(c, J, src, lvl);
  dma8(c.ring + (J % RING) * 8192, c.voffW, c.rsW, (src * 8 + lvl) * (64u * NN) + c.n0, 8u * NN);
}
template <int J> __device__ __forceinline__ void issue_a(CW& c) {
  unsigned src, lvl; step_id(c, J, src, lvl);
  unsigned v;
  while (true) {
    v = rfl(__hip_atomic_load(c.ctrl + src, __ATOMIC_ACQUIRE, __HIP_MEMORY_SCOPE_WORKGROUP));
    if (v > lvl) break;
    __builtin_amdgcn_s_sleep(1);
  }
  const unsigned so = (c.abase + (src * 8 + lvl) * 4096u) | (v & 0x80000000u);
  u32x4* d = c.af[J % (DLOOK + 1)];
  asm volatile("buffer_load_dwordx4 %0, %4, %5, %6 offen sc0 sc1\n\t"
               "buffer_load_dwordx4 %1, %4, %5, %6 offen offset:1024 sc0 sc1\n\t"
               "buffer_load_dwordx4 %2, %4, %5, %6 offen offset:2048 sc0 sc1\n\t"
               "buffer_load_dwordx4 %3, %4, %5, %6 offen offset:3072 sc0 sc1"
               : "=&v"(d[0]), "=&v"(d[1]), "=&v"(d[2]), "=&v"(d[3]) : "v"(c.voffA), "s"(c.rsA), "s"(so) : "memory");
}
template <int I> __device__ __forceinline__ void step(CW& c) {
  if constexpr (I == 2) { issue_a<2>(c); issue_a<3>(c); issue_a<4>(c); if constexpr (DLOOK >= 3) issue_a<5>(c); }
  else if constexpr (I > 2 && I + DLOOK <= 15) issue_a<I + DLOOK>(c);
  u32x4* d = c.af[I % (DLOOK + 1)];
  asm volatile("s_waitcnt vmcnt(%4)" : "+v"(d[0]), "+v"(d[1]), "+v"(d[2]), "+v"(d[3]) : "i"(waitN(I)) : "memory");
  SCHED();
  const i32x8 a0 = cat(d[0], d[1]), a1 = cat(d[2], d[3]);
#pragma unroll
  for (int n = 0; n < 4; ++n) {
    c.acc[0][n] = __builtin_amdgcn_mfma_scale_f32_32x32x64_f8f6f4(a0, c.bf[n], c.acc[0][n], 1, 1, 0, 0, 0, 0);
    c.acc[1][n] = __builtin_amdgcn_mfma_scale_f32_32x32x64_f8f6f4(a1, c.bf[n], c.acc[1][n], 1, 1, 0, 0, 0, 0);
    SCHED();
    if constexpr (I < 15) c.bf[n] = read_b(c.rd[n] + ((I + 1) % RING) * 8192);
    SCHED();
  }
  if constexpr (I + 1 + RING <= 15) {
    asm volatile("s_waitcnt lgkmcnt(0)" ::: "memory");
    issue_w<I + 1 + RING>(c);
    SCHED();
  }
}
template <int I> __device__ __forceinline__ void steps(CW& c) { step<I>(c); if constexpr (I < 15) steps<I + 1>(c); }

}

__global__ void __launch_bounds__(384, 2) a2a_gemm_kernel(Args a) {
  extern __shared__ __attribute__((aligned(16))) unsigned char lds_raw[];
  LAS unsigned char* L = (LAS unsigned char*)lds_raw;
  const unsigned lbase = (unsigned)(size_t)L;
  const int wave = rfl(threadIdx.x >> 6), lane = threadIdx.x & 63;
  const int b = blockIdx.x;
  const int rank = a.rank;
  unsigned char* myws = a.myws;
  unsigned* cnt = (unsigned*)(myws + OFF_CNT + (size_t)b * 128);
  unsigned* tmo = (unsigned*)(myws + OFF_TMO);
  const int mt = (b >> 3) & 7, nt = (b & 7) * 4 + (b >> 6);
  LAS unsigned* ctrl = (LAS unsigned*)(L + CTRL_OFF);

  if (wave == 4) {
    if (b >= 56) { __builtin_amdgcn_s_barrier(); return; }
    __builtin_amdgcn_s_setprio(3);
    const int dd = b % 7, smt = b / 7;
    const unsigned dest = (unsigned)(rank + 1 + dd) & 7u;
    u32x4 v[32];
    unsigned c0;
    asm volatile("global_load_dword %0, %1, off sc1" : "=v"(c0) : "v"(cnt) : "memory");
    const unsigned char* xs = a.x + (size_t)(dest * KS + smt * 64 + (lane & 31)) * KS + (lane >> 5) * 32;
#pragma unroll
    for (int s = 0; s < 32; ++s)
      asm volatile("global_load_dwordx4 %0, %1, off offset:%2" : "=v"(v[s]) : "v"(xs + ((s >> 1) & 1) * 32 * KS), "i"((s >> 2) * 64 + (s & 1) * 16) : "memory");
    asm volatile("s_waitcnt vmcnt(32)" : "+v"(c0) :: "memory");
    const unsigned call = c0 + 1;
    unsigned char* pw = a.peer[dd];
    unsigned char* dst = pw + OFF_DATA + (call & 1) * SLOT_BYTES + (size_t)(smt * 64 + rank * 8) * 4096 + lane * 16;
    unsigned* flag = (unsigned*)(pw + OFF_FLAG + (smt * 8 + rank) * 128);
#pragma unroll
    for (int s = 0; s < 32 + SWIN; ++s) {
      const int w = s - SWIN;
      if (w == 0) __builtin_amdgcn_s_barrier();
      if (w >= 0) {
        asm volatile("s_waitcnt vmcnt(%0)" :: "i"(sendN(w)) : "memory");
        if ((w & 3) == 3) {
          const unsigned val = (call - 1) * 8 + (w >> 2) + 1;
          if (lane == 0) asm volatile("global_store_dword %0, %1, off sc0 sc1" :: "v"(flag), "v"(val) : "memory");
        }
      }
      if (s < 32) {
        if (w < 0) asm volatile("s_waitcnt vmcnt(31)" : "+v"(v[s]) :: "memory");
        else asm volatile("" : "+v"(v[s]));
        asm volatile("global_store_dwordx4 %0, %1, off sc0 sc1\n\ts_nop 1" :: "v"(dst + s * 1024), "v"(v[s]) : "memory");
      }
    }
    return;
  }
  if (wave == 5) {
    if (lane < 16) ctrl[lane] = 0;
    const unsigned call = __hip_atomic_load(cnt, __ATOMIC_RELAXED, __HIP_MEMORY_SCOPE_AGENT) + 1;
    const float scale = a.sx[0] * a.sw[0];
    asm volatile("s_waitcnt lgkmcnt(0)" ::: "memory");
    __builtin_amdgcn_s_barrier();
    if (lane == 0) ctrl[9] = __builtin_bit_cast(unsigned, scale);
    const bool act = lane < 8 && lane != rank;
    unsigned* flag = lane < 8 ? (unsigned*)(myws + OFF_FLAG + (mt * 8 + lane) * 128) : tmo;
    const unsigned base = (call - 1) * 8;
    const unsigned long long tp = rt();
    unsigned seen = 0;
    bool bad = false;
    while (true) {
      const unsigned raw = __hip_atomic_load(flag, __ATOMIC_RELAXED, __HIP_MEMORY_SCOPE_SYSTEM);
      unsigned v = raw - base;
      v = (int)v < 0 ? 0u : (v > 8u ? 8u : v);
      if (act && v != seen) { __hip_atomic_store(ctrl + lane, v, __ATOMIC_RELEASE, __HIP_MEMORY_SCOPE_WORKGROUP); seen = v; }
      if (__ballot(act && v < 8u) == 0) break;
      if (__ballot(lane >= 8 && raw == call) != 0 || rt() - tp > DEADLINE_TICKS) { bad = true; break; }
    }
    if (bad) {
      if (lane == 0) __hip_atomic_store(tmo, call, __ATOMIC_RELAXED, __HIP_MEMORY_SCOPE_AGENT);
      if (lane < 8) __hip_atomic_store(ctrl + lane, 0x80000008u, __ATOMIC_RELEASE, __HIP_MEMORY_SCOPE_WORKGROUP);
    } else {
      bad = __hip_atomic_load(tmo, __ATOMIC_RELAXED, __HIP_MEMORY_SCOPE_AGENT) == call;
    }
    if (lane == 0) __hip_atomic_store(ctrl + 8, bad ? 2u : 1u, __ATOMIC_RELEASE, __HIP_MEMORY_SCOPE_WORKGROUP);
    asm volatile("s_waitcnt vmcnt(0) lgkmcnt(0)" ::: "memory");
    return;
  }

  CW c;
  c.rank = rank; c.mt7 = mt == 7 ? 0 : mt; c.wave = wave; c.ctrl = ctrl;
  c.ring = lbase + wave * WAVE_LDS;
  c.n0 = nt * 128;
  c.rsW = __builtin_amdgcn_make_buffer_rsrc((void*)a.w, (short)0, NN * NN, 0x00020000);
  c.rsA = __builtin_amdgcn_make_buffer_rsrc((void*)(myws + OFF_DATA), (short)0, (int)(2 * SLOT_BYTES), 0x00020000);
  const __amdgpu_buffer_rsrc_t rsX = __builtin_amdgcn_make_buffer_rsrc((void*)a.x, (short)0, NN * KS, 0x00020000);
  c.voffW = (lane >> 3) * NN + (((lane & 7) ^ ((lane >> 3) & 6)) << 4);
  c.voffA = lane * 16;
  unsigned c0;
  asm volatile("global_load_dword %0, %1, off sc1" : "=v"(c0) : "v"(cnt) : "memory");
  {
    const unsigned lds = c.ring + (RING - 1) * 8192;
    const unsigned so = (rank * KS + mt * 64) * KS + wave * 128;
    const unsigned v0 = (lane >> 3) * KS + (((lane & 7) ^ (lane >> 4)) << 4), v1 = v0 ^ 64u;
    unsigned t;
    asm volatile(
        "s_mov_b32 m0, %1\n\ts_mov_b32 %0, %5\n\tbuffer_load_dwordx4 %2, %4, %0 offen lds\n\t"
        "s_add_u32 m0, m0, 0x400\n\ts_add_u32 %0, %0, 0x1000\n\tbuffer_load_dwordx4 %3, %4, %0 offen lds\n\t"
        "s_add_u32 m0, m0, 0x400\n\ts_add_u32 %0, %0, 0x1000\n\tbuffer_load_dwordx4 %2, %4, %0 offen lds\n\t"
        "s_add_u32 m0, m0, 0x400\n\ts_add_u32 %0, %0, 0x1000\n\tbuffer_load_dwordx4 %3, %4, %0 offen lds\n\t"
        "s_add_u32 m0, m0, 0x400\n\ts_add_u32 %0, %0, 0x1000\n\tbuffer_load_dwordx4 %2, %4, %0 offen lds\n\t"
        "s_add_u32 m0, m0, 0x400\n\ts_add_u32 %0, %0, 0x1000\n\tbuffer_load_dwordx4 %3, %4, %0 offen lds\n\t"
        "s_add_u32 m0, m0, 0x400\n\ts_add_u32 %0, %0, 0x1000\n\tbuffer_load_dwordx4 %2, %4, %0 offen lds\n\t"
        "s_add_u32 m0, m0, 0x400\n\ts_add_u32 %0, %0, 0x1000\n\tbuffer_load_dwordx4 %3, %4, %0 offen lds"
        : "=&s"(t) : "s"(lds), "v"(v0), "v"(v1), "s"(rsX), "s"(so) : "memory", "scc");
  }
  issue_w<0>(c); issue_w<1>(c); issue_w<2>(c);
  {
    const int h = lane >> 5, g = (lane >> 4) & 1, q = (lane & 15) >> 1, p = lane & 1;
#pragma unroll
    for (int j = 0; j < 4; ++j)
      c.rd[j] = L + wave * WAVE_LDS + h * 4096 + q * 128 + (((((j ^ wave) << 1) + g) ^ (q & 6)) << 4) + p * 8;
  }
#pragma unroll
  for (int m = 0; m < 2; ++m)
#pragma unroll
    for (int n = 0; n < 4; ++n)
#pragma unroll
      for (int r = 0; r < 16; ++r) c.acc[m][n][r] = 0.f;
  __builtin_amdgcn_s_barrier();
  asm volatile("s_waitcnt vmcnt(16)" : "+v"(c0) :: "memory");
  const unsigned call = rfl(c0) + 1;
  c.abase = (call & 1) * (unsigned)SLOT_BYTES + mt * (64u * 4096u);
  {
    const int r = lane & 31, h = lane >> 5;
    LAS unsigned char* xb = L + wave * WAVE_LDS + (RING - 1) * 8192 + r * 128;
#pragma unroll
    for (int cc = 0; cc < 2; ++cc)
#pragma unroll
      for (int f = 0; f < 2; ++f)
#pragma unroll
        for (int i = 0; i < 2; ++i)
          c.af[cc][f * 2 + i] = *(LAS u32x4*)(xb + f * 4096 + (((cc * 4 + h * 2 + i) ^ ((r >> 1) & 7)) << 4));
#pragma unroll
    for (int n = 0; n < 4; ++n) c.bf[n] = read_b(c.rd[n]);
  }
  asm volatile("s_waitcnt lgkmcnt(0)" ::: "memory");
  issue_w<3>(c); issue_w<4>(c);
  SCHED();
  steps<0>(c);
  {
    LAS unsigned char* wb = L + wave * WAVE_LDS + lane * 16;
#pragma unroll
    for (int j = 1; j < 4; ++j)
#pragma unroll
      for (int m = 0; m < 2; ++m)
#pragma unroll
        for (int q = 0; q < 4; ++q) {
          f32x4 t; t[0] = c.acc[m][j][q * 4]; t[1] = c.acc[m][j][q * 4 + 1]; t[2] = c.acc[m][j][q * 4 + 2]; t[3] = c.acc[m][j][q * 4 + 3];
          *(LAS f32x4*)(wb + j * 8192 + (m * 4 + q) * 1024) = t;
        }
    asm volatile("s_waitcnt lgkmcnt(0)" ::: "memory");
    __builtin_amdgcn_s_barrier();
    asm volatile("" ::: "memory");
#pragma unroll
    for (int j = 1; j < 4; ++j) {
      LAS unsigned char* rb = L + (wave ^ j) * WAVE_LDS + j * 8192 + lane * 16;
#pragma unroll
      for (int m = 0; m < 2; ++m)
#pragma unroll
        for (int q = 0; q < 4; ++q) {
          const f32x4 t = *(LAS f32x4*)(rb + (m * 4 + q) * 1024);
          c.acc[m][0][q * 4] += t[0]; c.acc[m][0][q * 4 + 1] += t[1]; c.acc[m][0][q * 4 + 2] += t[2]; c.acc[m][0][q * 4 + 3] += t[3];
        }
    }
  }
  const bool bad = rfl(ctrl[8]) != 1u;
  const float scale = __builtin_bit_cast(float, (unsigned)ctrl[9]);
  if (threadIdx.x == 0) __hip_atomic_store(cnt, call, __ATOMIC_RELAXED, __HIP_MEMORY_SCOPE_AGENT);
  float* o = a.out + (size_t)(mt * 64 + 4 * (lane >> 5)) * NN + c.n0 + wave * 32 + (lane & 31);
#pragma unroll
  for (int m = 0; m < 2; ++m)
#pragma unroll
    for (int r = 0; r < 16; ++r) {
      const float y = c.acc[m][0][r] * scale;
      const float e = __builtin_amdgcn_exp2f(-1.44269504088896341f * __builtin_amdgcn_fmed3f(y, -60.f, 60.f));
      float z = y * __builtin_amdgcn_rcpf(1.f + e);
      if (bad) z = __builtin_nanf("");
      o[(size_t)(m * 32 + (r & 3) + 8 * (r >> 2)) * NN] = z;
    }
}

extern "C" void kernel_launch(int rank, int nranks, ncclComm_t comm, hipStream_t stream,
                              void* const* d_in, const int* in_sizes, int n_in,
                              void* d_out, int out_size, const int* in_dtypes, int out_dtype,
                              void* const* d_out_all, void* const* d_ws_all, size_t ws_bytes) {
  if (nranks != NRANK || (unsigned)rank >= (unsigned)NRANK || ws_bytes < WS_NEED || n_in < 4 ||
      in_sizes[0] != NN * KS || in_sizes[1] != NN * NN || out_size != KS * NN) {
    fprintf(stderr, "a2a_gemm: unexpected problem shape\n");
    return;
  }
  static bool attr_set = false;
  if (!attr_set) {
    HIP_CHECK(hipFuncSetAttribute((const void*)a2a_gemm_kernel, hipFuncAttributeMaxDynamicSharedMemorySize, LDS_TOTAL));
    attr_set = true;
  }
  Args a;
  a.x = (const unsigned char*)d_in[0]; a.w = (const unsigned char*)d_in[1];
  a.sx = (const float*)d_in[2]; a.sw = (const float*)d_in[3];
  a.out = (float*)d_out;
  a.myws = (unsigned char*)d_ws_all[rank];
  for (int p = 0; p < NRANK; ++p) a.peer[p] = (unsigned char*)d_ws_all[(rank + 1 + p) % NRANK];
  a.rank = rank; a.pad = 0;
  a2a_gemm_kernel<<<NWG, 384, LDS_TOTAL, stream>>>(a);
  HIP_CHECK(hipGetLastError());
}
```

```cpp
#include <hip/hip_runtime.h>
#include <rccl/rccl.h>
#include <cstdio>

#define HIP_CHECK(expr) do { hipError_t _e = (expr); if (_e != hipSuccess) { \
    fprintf(stderr, "HIP error %s at %s:%d\n", hipGetErrorString(_e), __FILE__, __LINE__); \
    return; } } while (0)

#define LAS __attribute__((address_space(3)))
#define GAS __attribute__((address_space(1)))
typedef unsigned u32x2 __attribute__((ext_vector_type(2)));
typedef unsigned u32x4 __attribute__((ext_vector_type(4)));
typedef int i32x2 __attribute__((ext_vector_type(2)));
typedef int i32x8 __attribute__((ext_vector_type(8)));
typedef float f32x4 __attribute__((ext_vector_type(4)));
typedef unsigned long long u64;

#define SWIN 7
#define PBS 8
#define SMAP 2
#define LX 4

namespace {
constexpr int NRANK = 8;
constexpr int KS = 512;
constexpr int NN = 4096;
constexpr int NWG = 256;
constexpr int NSEND = 32;
constexpr int NTILE = 7;
constexpr size_t OFF_CNT = 0;
constexpr size_t OFF_TMO = 32768;
constexpr size_t OFF_FLAG = 65536;
constexpr size_t OFF_DBG = 262144;
constexpr size_t OFF_DATA = 1u << 20;
constexpr size_t SLOT_BYTES = 2u << 20;
constexpr size_t WS_NEED = OFF_DATA + 2 * SLOT_BYTES;
constexpr u64 DEADLINE_TICKS = 100000000ull;
constexpr int NB = 12;
constexpr int NA = 12;
constexpr int LDS_A = NB * 8192;
constexpr int CTRL_OFF = LDS_A + NA * 5120;
constexpr int LDS_TOTAL = CTRL_OFF + 128 + 512;
constexpr int C_STATE = 0;
constexpr int C_EARLY = 4;
constexpr int TX = 4 + PBS;
constexpr int NT = TX + 28;
constexpr unsigned OOR = 0x80000000u;
static_assert(NB == 12 && NA % 2 == 0 && LX >= 3 && 2 * (LX + 1) <= NA && PBS >= 6, "ring schedule");

struct Args {
  const unsigned char* x; const unsigned char* w; const float* sx; const float* sw; float* out;
  unsigned char* myws;
  int rank; int pad;
  unsigned char* peer[NRANK];
};

__device__ __forceinline__ u64 rt() { return __builtin_amdgcn_s_memrealtime(); }
__device__ __forceinline__ unsigned rfl(unsigned v) { return __builtin_amdgcn_readfirstlane(v); }
__device__ __forceinline__ unsigned umin(unsigned a, unsigned b) { return a < b ? a : b; }
#define SCHED() __builtin_amdgcn_sched_barrier(0)
#define TICK() do { asm volatile("" ::: "memory"); __builtin_amdgcn_s_barrier(); asm volatile("" ::: "memory"); } while (0)
#define VMWAIT(n) asm volatile("s_waitcnt vmcnt(%0)" :: "i"(n) : "memory")
__device__ __forceinline__ int tile_g0(int t) { return t < 4 ? 5 * t : 4 * t + 4; }

#define DBG(i, v)

__device__ __forceinline__ void dma_w(unsigned lds, unsigned v0, unsigned v1, __amdgpu_buffer_rsrc_t rs, unsigned soff) {
  unsigned t;
  asm volatile("s_mov_b32 m0, %1\n\ts_mov_b32 %0, %5\n\tbuffer_load_dwordx4 %2, %4, %0 offen lds\n\t"
               "s_xor_b32 m0, %1, 0x400\n\ts_xor_b32 %0, %5, 0x8000\n\tbuffer_load_dwordx4 %2, %4, %0 offen lds\n\t"
               "s_xor_b32 m0, %1, 0x800\n\ts_xor_b32 %0, %5, 0x10000\n\tbuffer_load_dwordx4 %2, %4, %0 offen lds\n\t"
               "s_xor_b32 m0, %1, 0xc00\n\ts_xor_b32 %0, %5, 0x18000\n\tbuffer_load_dwordx4 %2, %4, %0 offen lds\n\t"
               "s_xor_b32 m0, %1, 0x1000\n\ts_xor_b32 %0, %5, 0x20000\n\tbuffer_load_dwordx4 %3, %4, %0 offen lds\n\t"
               "s_xor_b32 m0, %1, 0x1400\n\ts_xor_b32 %0, %5, 0x28000\n\tbuffer_load_dwordx4 %3, %4, %0 offen lds\n\t"
               "s_xor_b32 m0, %1, 0x1800\n\ts_xor_b32 %0, %5, 0x30000\n\tbuffer_load_dwordx4 %3, %4, %0 offen lds\n\t"
               "s_xor_b32 m0, %1, 0x1c00\n\ts_xor_b32 %0, %5, 0x38000\n\tbuffer_load_dwordx4 %3, %4, %0 offen lds"
               : "=&s"(t) : "s"(lds), "v"(v0), "v"(v1), "s"(rs), "s"(soff) : "memory", "scc");
}
template <unsigned LO, unsigned SO>
__device__ __forceinline__ void dma_a(unsigned lds, unsigned voff, __amdgpu_buffer_rsrc_t rs, unsigned soff) {
  unsigned t;
  asm volatile("s_add_u32 m0, %1, %5\n\ts_add_u32 %0, %4, %6\n\tbuffer_load_dwordx4 %2, %3, %0 offen lds"
               : "=&s"(t) : "s"(lds), "v"(voff), "s"(rs), "s"(soff), "i"(LO), "i"(SO) : "memory", "scc");
}
template <int NG>
__device__ __forceinline__ void dma_piece(unsigned lds, unsigned voff, __amdgpu_buffer_rsrc_t rs, unsigned soff) {
  unsigned t;
  asm volatile("s_mov_b32 m0, %1\n\ts_mov_b32 %0, %4\n\t"
               "buffer_load_dwordx4 %2, %3, %0 offen sc0 sc1 lds\n\t"
               "buffer_load_dwordx4 %2, %3, %0 offen offset:1024 sc0 sc1 lds\n\t"
               "buffer_load_dwordx4 %2, %3, %0 offen offset:2048 sc0 sc1 lds\n\t"
               "buffer_load_dwordx4 %2, %3, %0 offen offset:3072 sc0 sc1 lds"
               : "=&s"(t) : "s"(lds), "v"(voff), "s"(rs), "s"(soff) : "memory");
  if constexpr (NG == 5)
    asm volatile("s_add_u32 m0, %1, 0x1000\n\ts_add_u32 %0, %4, 0x1000\n\tbuffer_load_dwordx4 %2, %3, %0 offen sc0 sc1 lds"
                 : "=&s"(t) : "s"(lds), "v"(voff), "s"(rs), "s"(soff) : "memory", "scc");
}
__device__ __forceinline__ i32x8 cat(u32x4 a, u32x4 b) {
  return __builtin_bit_cast(i32x8, __builtin_shufflevector(a, b, 0, 1, 2, 3, 4, 5, 6, 7));
}

constexpr int sendN(int NG, int w) {
  int n = 0;
  for (int t = w; t < w + SWIN; ++t) {
    if (t > w && t < 8 * NG) ++n;
    if (t - SWIN >= 0 && (t - SWIN + 1) % NG == 0) ++n;
  }
  return n;
}
template <int NG>
__device__ __forceinline__ unsigned send_stream(__amdgpu_buffer_rsrc_t rsX, unsigned vx, unsigned sx, unsigned char* dst0, unsigned* flag, GAS unsigned* cnt, int lane) {
  constexpr int ST = 8 * NG;
  u32x4 v[ST];
  unsigned c0;
  asm volatile("global_load_dword %0, %1, off sc1" : "=v"(c0) : "v"(cnt) : "memory");
#pragma unroll
  for (int s = 0; s < ST; ++s)
    asm volatile("buffer_load_dwordx4 %0, %1, %2, %3 offen" : "=v"(v[s]) : "v"(vx), "s"(rsX), "s"((sx ^ ((s / NG) * 64)) + (s % NG) * 16 * KS) : "memory");
  asm volatile("s_waitcnt vmcnt(%1)" : "+v"(c0) : "i"(ST) : "memory");
  const unsigned call = c0 + 1;
  unsigned char* dst = dst0 + (call & 1) * SLOT_BYTES;
#pragma unroll
  for (int s = 0; s < ST + SWIN; ++s) {
    const int w = s - SWIN;
    if (w >= 0) VMWAIT(sendN(NG, w));
    if (s < ST) {
      if (w < 0) asm volatile("s_waitcnt vmcnt(%1)" : "+v"(v[s]) : "i"(ST - 1) : "memory");
      else asm volatile("" : "+v"(v[s]));
      asm volatile("global_store_dwordx4 %0, %1, off sc0 sc1\n\ts_nop 1" :: "v"(dst + (s / NG) * 32768 + (s % NG) * 1024), "v"(v[s]) : "memory");
    }
    if (w >= 0 && (w + 1) % NG == 0) {
      const unsigned val = (call - 1) * 8 + w / NG + 1;
      if (lane == 0) asm volatile("global_store_dword %0, %1, off sc0 sc1" :: "v"(flag), "v"(val) : "memory");
    }
  }
  return call;
}

template <int NG> struct CW {
  f32x4 acc[NG][2];
  i32x8 bf[32][2];
  i32x8 af[NG];
  LAS unsigned char* rb[2][2];
  LAS unsigned char* ra[2];
};
#define STAMP(x)
#define STAMPC(x)
template <int NG, int U, int CT> __device__ __forceinline__ void read_b(CW<NG>& c) {
  constexpr int S = (2 * U) % NB;
  LAS unsigned char* p = c.rb[CT][S >> 3] + (S & 7) * 8192;
  i32x2 a = __builtin_amdgcn_ds_read_tr8_b64_v2i32((LAS i32x2*)(p));
  i32x2 b = __builtin_amdgcn_ds_read_tr8_b64_v2i32((LAS i32x2*)(p + 1024));
  i32x2 e = __builtin_amdgcn_ds_read_tr8_b64_v2i32((LAS i32x2*)(p + 2048));
  i32x2 d = __builtin_amdgcn_ds_read_tr8_b64_v2i32((LAS i32x2*)(p + 3072));
  i32x8 r; r[0] = a[0]; r[1] = a[1]; r[2] = b[0]; r[3] = b[1]; r[4] = e[0]; r[5] = e[1]; r[6] = d[0]; r[7] = d[1];
  c.bf[U][CT] = r;
}
template <int NG, int U, int M> __device__ __forceinline__ void read_a(CW<NG>& c) {
  constexpr int O = ((2 * U) % NA) * 5120 + M * 1024;
  c.af[M] = cat(*(LAS u32x4*)(c.ra[0] + O), *(LAS u32x4*)(c.ra[1] + O));
}
template <int NG, int U, int M> __device__ __forceinline__ void read_a_all(CW<NG>& c) {
  read_a<NG, U, M>(c);
  if constexpr (M + 1 < NG) read_a_all<NG, U, M + 1>(c);
}
constexpr int b_hi(int u) { return u < 4 ? u + 1 : u + PBS; }
template <int NG, int U, int E> __device__ __forceinline__ void prefill(CW<NG>& c) {
  asm volatile("s_waitcnt lgkmcnt(0)" ::: "memory");
  TICK();
  read_b<NG, U, 0>(c); read_b<NG, U, 1>(c);
  SCHED();
  if constexpr (U + 1 < E) prefill<NG, U + 1, E>(c);
}
template <int NG, int U, int M> __device__ __forceinline__ void row_tile(CW<NG>& c) {
  constexpr bool RA = U != 3 && U + 1 < 32, RB = b_hi(U) < 32;
  c.acc[M][0] = __builtin_amdgcn_mfma_scale_f32_16x16x128_f8f6f4(c.bf[U][0], c.af[M], c.acc[M][0], 1, 1, 0, 0, 0, 0);
  c.acc[M][1] = __builtin_amdgcn_mfma_scale_f32_16x16x128_f8f6f4(c.bf[U][1], c.af[M], c.acc[M][1], 1, 1, 0, 0, 0, 0);
  SCHED();
  if constexpr (M == 0 && U + 1 < 32) {
    asm volatile("s_waitcnt lgkmcnt(%0)" :: "i"(2 * (NG - 3)) : "memory");
    TICK();
  }
  if constexpr (RA) read_a<NG, U + 1, M>(c);
  if constexpr (RB && (M == 1 || M == 2)) read_b<NG, b_hi(U), M - 1>(c);
  SCHED();
  if constexpr (M + 1 < NG) row_tile<NG, U, M + 1>(c);
}
template <int NG, int U> __device__ __forceinline__ void steps(CW<NG>& c) {
  row_tile<NG, U, 0>(c);
  if constexpr (U == 3) {
    prefill<NG, 5, TX>(c);
    asm volatile("s_waitcnt lgkmcnt(0)" ::: "memory");
    STAMP(c.st[16]);
    TICK();
    read_a_all<NG, 4, 0>(c);
    SCHED();
  }
  if constexpr (U + 1 < 32) steps<NG, U + 1>(c);
}

struct XL {
  __amdgpu_buffer_rsrc_t rsA, rsF;
  GAS unsigned* tmo;
  volatile LAS unsigned* seen;
  unsigned va, aoff, lda, ld, scratch, vflag, need, call, q;
  u64 arrived, tp;
  unsigned dead;
  int lane;
};
__device__ __forceinline__ unsigned poll_off(const XL& x) {
  return __builtin_amdgcn_inverse_ballot_w64(x.arrived & ~0xffull) ? OOR : x.vflag;
}
__device__ __forceinline__ void poll_use(XL& x, unsigned r) { x.arrived |= __ballot((int)(r - x.need) > 0) & ~0xffull; }
__device__ __forceinline__ void poll_async(XL& x) {
  asm volatile("s_mov_b32 m0, %0\n\ts_nop 0\n\tbuffer_load_dword %1, %2, 0 offen sc0 sc1 lds" :: "s"(x.scratch), "v"(poll_off(x)), "s"(x.rsF) : "memory");
}
template <int NG> __device__ __forceinline__ void x_next(XL& x) {
  poll_async(x);
  while (__builtin_expect(!((x.arrived >> x.q) & 1), 0)) {
    unsigned r;
    asm volatile("buffer_load_dword %0, %1, %2, 0 offen sc0 sc1\n\ts_waitcnt vmcnt(0)" : "=&v"(r) : "v"(poll_off(x)), "s"(x.rsF) : "memory");
    poll_use(x, r);
    if (__ballot(x.lane < 8 && r == x.call) != 0 || rt() - x.tp > DEADLINE_TICKS) {
      __hip_atomic_store(x.tmo, x.call, __ATOMIC_RELAXED, __HIP_MEMORY_SCOPE_AGENT);
      x.dead = OOR; x.arrived = ~0ull;
    }
  }
  dma_piece<NG>(x.ld, x.va, x.rsA, __builtin_amdgcn_readlane(x.aoff, x.q) | x.dead);
  x.q += 2;
  x.ld = x.ld == x.lda + (NA - 2) * 5120 ? x.lda : x.ld + 10240;
}
template <int NG, int K> __device__ __forceinline__ void x_drain() {
  VMWAIT(K * (NG + 1)); TICK();
  if constexpr (K > 0) x_drain<NG, K - 1>();
}

struct Tile { const unsigned char* ax; const unsigned char* aw; float* aout; unsigned char* myws; int rank, b, tile, nt; };

template <int NG>
__device__ __forceinline__ void tile_body(const Args& a, const Tile& t, LAS unsigned char* L, const int wave, const int lane) {
  const unsigned lbase = (unsigned)(size_t)L;
  LAS unsigned* ctrl = (LAS unsigned*)(L + CTRL_OFF);
  const int rank = t.rank, g0 = tile_g0(t.tile);
  GAS unsigned* cnt = (GAS unsigned*)(t.myws + OFF_CNT + (size_t)t.b * 128);
  unsigned call = 0;
  if (wave >= 4) {
    __builtin_amdgcn_s_setprio(2);
    const unsigned j = wave & 1;
    unsigned psrc, plvl;
    {
      const unsigned p = lane - 8, l7 = (p * 37u) >> 8;
      unsigned s = p - l7 * 7u + t.tile;
      s = umin(s, s - 7u);
      psrc = lane < 8 ? rank : (rank + 1 + s) & 7u;
      plvl = lane < 8 ? (lane + 2 * t.tile) & 7u : l7;
    }
    if (wave < 6) {
      unsigned c0;
      asm volatile("global_load_dword %0, %1, off sc1" : "=v"(c0) : "v"(cnt) : "memory");
      const unsigned swz = ((lane & 3) ^ (lane >> 5)) << 4;
      XL x;
      x.lda = lbase + LDS_A + j * 5120;
      {
        const __amdgpu_buffer_rsrc_t rsX = __builtin_amdgcn_make_buffer_rsrc((void*)t.ax, (short)0, NN * KS, 0x00020000);
        const unsigned vx = (lane >> 2) * KS + swz;
        const unsigned xo = (rank * KS + g0 * 16) * KS + j * 64;
#pragma unroll
        for (int i = 0; i < 4; ++i) {
          const unsigned so = xo + ((2 * i + 2 * t.tile) & 7) * 64, ld = x.lda + i * 10240;
          dma_a<0 * 1024, 0 * 8192>(ld, vx, rsX, so); dma_a<1 * 1024, 1 * 8192>(ld, vx, rsX, so);
          dma_a<2 * 1024, 2 * 8192>(ld, vx, rsX, so); dma_a<3 * 1024, 3 * 8192>(ld, vx, rsX, so);
          if constexpr (NG == 5) dma_a<4 * 1024, 4 * 8192>(ld, vx, rsX, so);
        }
      }
      x.rsA = __builtin_amdgcn_make_buffer_rsrc((void*)(t.myws + OFF_DATA), (short)0, (int)(2 * SLOT_BYTES), 0x00020000);
      x.rsF = __builtin_amdgcn_make_buffer_rsrc((void*)t.myws, (short)0, (int)OFF_DBG, 0x00020000);
      x.tmo = (GAS unsigned*)(t.myws + OFF_TMO);
      x.va = (lane >> 2) * 64 + swz;
      x.vflag = lane >= 8 ? (unsigned)OFF_FLAG + (t.tile * 8 + psrc) * 128 : (unsigned)OFF_TMO;
      x.lane = lane; x.arrived = 0; x.dead = 0; x.q = 8 + j; x.ld = x.lda + (4 % (NA / 2)) * 10240;
      x.scratch = lbase + CTRL_OFF + 128 + j * 256;
      x.seen = (volatile LAS unsigned*)(L + CTRL_OFF + 128 + j * 256) + lane;
      *x.seen = 0;
      asm volatile("s_waitcnt vmcnt(%1) lgkmcnt(0)" : "+v"(c0) : "i"(NG * 3) : "memory");
      TICK();
      x.call = call = rfl(c0) + 1;
      x.need = (call - 1) * 8 + plvl;
      x.aoff = (call & 1) * (unsigned)SLOT_BYTES + (psrc * 8 + plvl) * 32768u + g0 * 1024;
      x.tp = rt();
      VMWAIT(NG * 2); TICK();
      VMWAIT(NG); TICK();
      VMWAIT(0); TICK();
#pragma unroll
      for (int i = 4; i < TX - 6; ++i) TICK();
      poll_async(x);
      TICK(); TICK(); TICK();
      poll_use(x, *x.seen);
      constexpr u64 FIRST = (0x5555555555555555ull & ((1ull << (2 * (LX - 1))) - 1)) << 8;
      const bool early = ((x.arrived >> j) & FIRST) == FIRST;
      *(volatile LAS unsigned*)(ctrl + C_EARLY + j) = early;
      asm volatile("s_waitcnt lgkmcnt(0)" ::: "memory");
      if (!early) { TICK(); TICK(); TICK(); }
#pragma unroll 1
      for (int i = 0; i < LX - 1; ++i) x_next<NG>(x);
      if (early) { TICK(); TICK(); TICK(); }
      VMWAIT((LX - 2) * (NG + 1));
      TICK();
#pragma unroll 1
      for (int v = 3; v < 32 - LX; ++v) {
        x_next<NG>(x);
        VMWAIT((LX - 2) * (NG + 1));
        const unsigned r = *x.seen;
        TICK();
        poll_use(x, r);
      }
      x_drain<NG, LX - 3>();
      DBG(20 + j, rt() - t0);
      const bool late = x.dead || __hip_atomic_load(x.tmo, __ATOMIC_RELAXED, __HIP_MEMORY_SCOPE_AGENT) == call;
      *(volatile LAS unsigned*)(ctrl + C_STATE + j) = late ? 2u : 1u;
    } else {
      const unsigned rot = (t.tile + 4 * j) & 7;
      const __amdgpu_buffer_rsrc_t rsW = __builtin_amdgcn_make_buffer_rsrc((void*)t.aw, (short)0, NN * NN, 0x00020000);
      const unsigned woff = (psrc * 8 + (lane < 8 ? plvl : plvl ^ psrc)) * (64u * NN) + t.nt * 128 + (rot << 15);
      const unsigned vw0 = (lane >> 3) * NN + (((lane & 7) ^ ((lane >> 3) & 6) ^ (rot >> 2)) << 4), vw1 = vw0 ^ 16u;
      const unsigned ld0 = (lbase + j * 8192) | (rot << 10);
      unsigned q = j, ld = ld0;
      auto issue = [&]() {
        dma_w(ld, vw0, vw1, rsW, __builtin_amdgcn_readlane(woff, q));
        q += 2; ld = ld == ld0 + (NB - 2) * 8192 ? ld0 : ld + 16384;
      };
      issue(); issue();
#pragma unroll 1
      for (int i = 0; i < TX; ++i) { issue(); VMWAIT(16); TICK(); }
      const u32x2 ea = *(volatile LAS u32x2*)(ctrl + C_EARLY);
      if (rfl(ea[0] & ea[1])) {
        TICK();
#pragma unroll 1
        for (int i = TX; i < 30; ++i) { issue(); VMWAIT(16); TICK(); }
      } else {
        issue(); issue(); issue();
        TICK();
        VMWAIT(32); TICK();
#pragma unroll 1
        for (int i = TX + 1; i < 28; ++i) { issue(); VMWAIT(32); TICK(); }
        VMWAIT(24); TICK();
        VMWAIT(16); TICK();
      }
      VMWAIT(8); TICK();
      VMWAIT(0); TICK();
      DBG(24 + j, rt() - t0);
#pragma unroll
      for (int i = 33; i < NT; ++i) TICK();
    }
    __builtin_amdgcn_s_setprio(0);
  } else {
    CW<NG> c;
    {
      const int g = lane >> 4, q = (lane & 15) >> 1, p = lane & 1, r = lane & 15;
      const unsigned b0 = lbase + g * 4096 + q * 128 + (((wave << 1) ^ (q & 6) ^ (g & 1)) << 4) + p * 8;
      unsigned b1 = b0 ^ 16u, b2 = b0 + 65536, b3 = b1 + 65536;
      asm volatile("" : "+v"(b2), "+v"(b3));
      c.rb[0][0] = (LAS unsigned char*)(size_t)b0; c.rb[1][0] = (LAS unsigned char*)(size_t)b1;
      c.rb[0][1] = (LAS unsigned char*)(size_t)b2; c.rb[1][1] = (LAS unsigned char*)(size_t)b3;
      unsigned a0 = lbase + LDS_A + (g >> 1) * 5120 + r * 64 + (((2 * (g & 1)) ^ (r >> 3)) << 4), a1 = a0 ^ 16u;
      asm volatile("" : "+v"(a0), "+v"(a1));
      c.ra[0] = (LAS unsigned char*)(size_t)a0; c.ra[1] = (LAS unsigned char*)(size_t)a1;
    }
#pragma unroll
    for (int m = 0; m < NG; ++m)
#pragma unroll
      for (int r = 0; r < 4; ++r) { c.acc[m][0][r] = 0.f; c.acc[m][1][r] = 0.f; }
    if (wave == 0) {
      const float sc = *(const __attribute__((address_space(4))) float*)a.sx * *(const __attribute__((address_space(4))) float*)a.sw;
      *(volatile LAS unsigned*)(ctrl + C_STATE + 2) = __builtin_bit_cast(unsigned, sc);
    }
    TICK();
    read_b<NG, 0, 0>(c); read_b<NG, 0, 1>(c);
    SCHED();
    read_a_all<NG, 0, 0>(c);
    SCHED();
    steps<NG, 0>(c);
    DBG(wave, rt() - t0);
    const unsigned wr = lbase + (lane & 15) * 512 + wave * 128 + (((lane >> 4) ^ (lane & 7)) << 4);
#pragma unroll
    for (int m = 0; m < NG; ++m) {
      *(LAS f32x4*)(size_t)(wr + m * 8192) = c.acc[m][0];
      *(LAS f32x4*)(size_t)((wr ^ 64u) + m * 8192) = c.acc[m][1];
    }
  }
  const __amdgpu_buffer_rsrc_t rsO = __builtin_amdgcn_make_buffer_rsrc((void*)t.aout, (short)0, KS * NN * 4, 0x00020000);
  const unsigned row = wave * 2 + (lane >> 5);
  const unsigned vo = row * (NN * 4) + (lane & 31) * 16;
  const unsigned so = (g0 * 16 * NN + t.nt * 128) * 4;
  const unsigned rd = lbase + row * 512 + (((lane & 31) ^ (row & 7)) << 4);
  asm volatile("s_waitcnt lgkmcnt(0)" ::: "memory");
  __builtin_amdgcn_s_barrier();
  asm volatile("" ::: "memory");
  if (wave < 4) DBG(16 + wave, rt() - t0);
  const u32x4 st = *(volatile LAS u32x4*)(ctrl + C_STATE);
  f32x4 s[NG];
#pragma unroll
  for (int m = 0; m < NG; ++m) s[m] = *(LAS f32x4*)(size_t)(rd + m * 8192);
  SCHED();
  const unsigned sbits = st[2];
  const float scale = (st[0] == 1u && st[1] == 1u) ? __builtin_bit_cast(float, sbits) : __builtin_nanf("");
  const float scale2 = -1.44269504088896341f * scale;
#pragma unroll
  for (int m = 0; m < NG; ++m) {
    f32x4 e, z;
#pragma unroll
    for (int r = 0; r < 4; ++r) e[r] = __builtin_amdgcn_exp2f(__builtin_amdgcn_fmed3f(s[m][r] * scale2, -86.5617024533378f, 86.5617024533378f));
#pragma unroll
    for (int r = 0; r < 4; ++r) z[r] = (s[m][r] * scale) * __builtin_amdgcn_rcpf(1.f + e[r]);
    __builtin_amdgcn_raw_buffer_store_b128(__builtin_bit_cast(u32x4, z), rsO, vo, so + m * 16 * (NN * 4), 16);
    SCHED();
    asm volatile("s_nop 1");
    SCHED();
  }
  if (wave == 4 && lane == 0) __hip_atomic_store(cnt, call, __ATOMIC_RELAXED, __HIP_MEMORY_SCOPE_AGENT);
}
}

__global__ void __launch_bounds__(512, 2) a2a_gemm_kernel(Args a) {
  extern __shared__ __attribute__((aligned(16))) unsigned char lds_raw[];
  const int wave = rfl(threadIdx.x >> 6), lane = threadIdx.x & 63;
  const int b = blockIdx.x;
  Tile t;
  t.ax = a.x; t.aw = a.w; t.aout = a.out; t.myws = a.myws; t.rank = a.rank; t.b = b;
  if (b < NSEND) {
    const int sid = b + NSEND * wave;
    if (wave >= 2) return;
    GAS unsigned* cnt = (GAS unsigned*)(t.myws + OFF_CNT + (size_t)b * 128 + wave * 64);
    if (sid >= (NRANK - 1) * NTILE) return;
    const int dd = sid % 7, tile = sid / 7, g0 = tile_g0(tile);
    unsigned char* pw = a.peer[dd];
    const unsigned dest = (unsigned)(t.rank + 1 + dd) & 7u;
    const __amdgpu_buffer_rsrc_t rsX = __builtin_amdgcn_make_buffer_rsrc((void*)t.ax, (short)0, NN * KS, 0x00020000);
    const unsigned vx = (lane >> 2) * KS + (lane & 3) * 16, sx = (dest * KS + g0 * 16) * KS + t.rank * 64;
    unsigned char* dst = pw + OFF_DATA + (size_t)t.rank * 8 * 32768 + g0 * 1024 + lane * 16;
    unsigned* flag = (unsigned*)(pw + OFF_FLAG + (tile * 8 + t.rank) * 128);
    const unsigned call = tile < 4 ? send_stream<5>(rsX, vx, sx, dst, flag, cnt, lane) : send_stream<4>(rsX, vx, sx, dst, flag, cnt, lane);
    DBG(26 + wave, rt() - t0);
    if (lane == 0) __hip_atomic_store(cnt, call, __ATOMIC_RELAXED, __HIP_MEMORY_SCOPE_AGENT);
    return;
  }
  asm volatile("" : "+s"(t.ax), "+s"(t.aw), "+s"(t.aout), "+s"(t.myws), "+s"(t.rank));
  const int cj = (b - NSEND) >> 3;
  t.tile = cj % NTILE;
  t.nt = (cj / NTILE) * 8 + ((b + 2 * (cj / NTILE)) & 7);
  if (t.tile < 4) tile_body<5>(a, t, (LAS unsigned char*)lds_raw, wave, lane);
  else tile_body<4>(a, t, (LAS unsigned char*)lds_raw, wave, lane);
}

extern "C" void kernel_launch(int rank, int nranks, ncclComm_t comm, hipStream_t stream,
                              void* const* d_in, const int* in_sizes, int n_in,
                              void* d_out, int out_size, const int* in_dtypes, int out_dtype,
                              void* const* d_out_all, void* const* d_ws_all, size_t ws_bytes) {
  if (nranks != NRANK || (unsigned)rank >= (unsigned)NRANK || ws_bytes < WS_NEED || n_in < 4 ||
      in_sizes[0] != NN * KS || in_sizes[1] != NN * NN || out_size != KS * NN) {
    fprintf(stderr, "a2a_gemm: unexpected problem shape\n");
    return;
  }
  static bool attr_set = false;
  if (!attr_set) {
    HIP_CHECK(hipFuncSetAttribute((const void*)a2a_gemm_kernel, hipFuncAttributeMaxDynamicSharedMemorySize, LDS_TOTAL));
    attr_set = true;
  }
  Args a;
  a.x = (const unsigned char*)d_in[0]; a.w = (const unsigned char*)d_in[1];
  a.sx = (const float*)d_in[2]; a.sw = (const float*)d_in[3];
  a.out = (float*)d_out;
  a.myws = (unsigned char*)d_ws_all[rank];
  for (int p = 0; p < NRANK; ++p) a.peer[p] = (unsigned char*)d_ws_all[(rank + 1 + p) % NRANK];
  a.rank = rank; a.pad = 0;
  a2a_gemm_kernel<<<NWG, 512, LDS_TOTAL, stream>>>(a);
  HIP_CHECK(hipGetLastError());
}
```

```cpp
#include <hip/hip_runtime.h>
#include <rccl/rccl.h>
#include <cstdio>

#define HIP_CHECK(expr) do { hipError_t _e = (expr); if (_e != hipSuccess) { \
    fprintf(stderr, "HIP error %s at %s:%d\n", hipGetErrorString(_e), __FILE__, __LINE__); \
    return; } } while (0)

#define LAS __attribute__((address_space(3)))
#define GAS __attribute__((address_space(1)))
typedef unsigned u32x2 __attribute__((ext_vector_type(2)));
typedef unsigned u32x4 __attribute__((ext_vector_type(4)));
typedef int i32x2 __attribute__((ext_vector_type(2)));
typedef int i32x8 __attribute__((ext_vector_type(8)));
typedef float f32x4 __attribute__((ext_vector_type(4)));
typedef unsigned long long u64;

#define SWIN 7
#define PROT 10
#define PLOW 1

namespace {
constexpr int NRANK = 8;
constexpr int KS = 512;
constexpr int NN = 4096;
constexpr int NWG = 256;
constexpr int NSEND = 32;
constexpr int NTILE = 7;
constexpr size_t OFF_CNT = 0;
constexpr size_t OFF_TMO = 32768;
constexpr size_t OFF_FLAG = 65536;
constexpr size_t OFF_DBG = 262144;
constexpr size_t OFF_DATA = 1u << 20;
constexpr size_t SLOT_BYTES = 2u << 20;
constexpr size_t WS_NEED = OFF_DATA + 2 * SLOT_BYTES;
constexpr u64 DEADLINE_TICKS = 100000000ull;
constexpr int NB = 12;
constexpr int NA = 12;
constexpr int LDS_A = NB * 8192;
constexpr int CTRL_OFF = LDS_A + NA * 5120;
constexpr int LDS_JUNK = CTRL_OFF + 128 + 256;
constexpr int LDS_TOTAL = LDS_JUNK + 256;
constexpr int C_STATE = 0;
constexpr int NF = 7;
constexpr unsigned OOR = 0x80000000u;
static_assert(NB == 12 && NA == 12, "ring schedule");

struct Args {
  const unsigned char* x; const unsigned char* w; const float* sx; const float* sw; float* out;
  unsigned char* myws;
  int rank; int pad;
  unsigned char* peer[NRANK];
};

__device__ __forceinline__ u64 rt() { return __builtin_amdgcn_s_memrealtime(); }
__device__ __forceinline__ unsigned rfl(unsigned v) { return __builtin_amdgcn_readfirstlane(v); }
__device__ __forceinline__ unsigned umin(unsigned a, unsigned b) { return a < b ? a : b; }
#define SCHED() __builtin_amdgcn_sched_barrier(0)
#define TICK() do { asm volatile("" ::: "memory"); __builtin_amdgcn_s_barrier(); asm volatile("" ::: "memory"); } while (0)
#define VMWAIT(n) asm volatile("s_waitcnt vmcnt(%0)" :: "i"(n) : "memory")
__device__ __forceinline__ int tile_g0(int t) { return t < 4 ? 5 * t : 4 * t + 4; }

#define DBG(i, v)

__device__ __forceinline__ void dma_w_head(unsigned lds, unsigned v0, __amdgpu_buffer_rsrc_t rs, unsigned soff) {
  unsigned t;
  asm volatile("s_mov_b32 m0, %1\n\ts_mov_b32 %0, %4\n\tbuffer_load_dwordx4 %2, %3, %0 offen lds\n\t"
               "s_xor_b32 m0, %1, 0x400\n\ts_xor_b32 %0, %4, 0x8000\n\tbuffer_load_dwordx4 %2, %3, %0 offen lds"
               : "=&s"(t) : "s"(lds), "v"(v0), "s"(rs), "s"(soff) : "memory", "scc");
}
__device__ __forceinline__ void dma_w_tail(unsigned lds, unsigned v0, unsigned v1, __amdgpu_buffer_rsrc_t rs, unsigned soff) {
  unsigned t;
  asm volatile("s_xor_b32 m0, %1, 0x800\n\ts_xor_b32 %0, %5, 0x10000\n\tbuffer_load_dwordx4 %2, %4, %0 offen lds\n\t"
               "s_xor_b32 m0, %1, 0xc00\n\ts_xor_b32 %0, %5, 0x18000\n\tbuffer_load_dwordx4 %2, %4, %0 offen lds\n\t"
               "s_xor_b32 m0, %1, 0x1000\n\ts_xor_b32 %0, %5, 0x20000\n\tbuffer_load_dwordx4 %3, %4, %0 offen lds\n\t"
               "s_xor_b32 m0, %1, 0x1400\n\ts_xor_b32 %0, %5, 0x28000\n\tbuffer_load_dwordx4 %3, %4, %0 offen lds\n\t"
               "s_xor_b32 m0, %1, 0x1800\n\ts_xor_b32 %0, %5, 0x30000\n\tbuffer_load_dwordx4 %3, %4, %0 offen lds\n\t"
               "s_xor_b32 m0, %1, 0x1c00\n\ts_xor_b32 %0, %5, 0x38000\n\tbuffer_load_dwordx4 %3, %4, %0 offen lds"
               : "=&s"(t) : "s"(lds), "v"(v0), "v"(v1), "s"(rs), "s"(soff) : "memory", "scc");
}
__device__ __forceinline__ void touch(unsigned lds, unsigned voff, __amdgpu_buffer_rsrc_t rs, unsigned soff) {
  asm volatile("s_mov_b32 m0, %0\n\ts_nop 0\n\tbuffer_load_dword %1, %2, %3 offen lds" :: "s"(lds), "v"(voff), "s"(rs), "s"(soff) : "memory");
}
template <unsigned LO, unsigned SO>
__device__ __forceinline__ void dma_a(unsigned lds, unsigned voff, __amdgpu_buffer_rsrc_t rs, unsigned soff) {
  unsigned t;
  asm volatile("s_add_u32 m0, %1, %5\n\ts_add_u32 %0, %4, %6\n\tbuffer_load_dwordx4 %2, %3, %0 offen lds"
               : "=&s"(t) : "s"(lds), "v"(voff), "s"(rs), "s"(soff), "i"(LO), "i"(SO) : "memory", "scc");
}
template <int NG>
__device__ __forceinline__ void dma_piece(unsigned lds, unsigned voff, __amdgpu_buffer_rsrc_t rs, unsigned soff) {
  unsigned t;
  asm volatile("s_mov_b32 m0, %1\n\ts_mov_b32 %0, %4\n\t"
               "buffer_load_dwordx4 %2, %3, %0 offen sc0 sc1 lds\n\t"
               "buffer_load_dwordx4 %2, %3, %0 offen offset:1024 sc0 sc1 lds\n\t"
               "buffer_load_dwordx4 %2, %3, %0 offen offset:2048 sc0 sc1 lds\n\t"
               "buffer_load_dwordx4 %2, %3, %0 offen offset:3072 sc0 sc1 lds"
               : "=&s"(t) : "s"(lds), "v"(voff), "s"(rs), "s"(soff) : "memory");
  if constexpr (NG == 5)
    asm volatile("s_add_u32 m0, %1, 0x1000\n\ts_add_u32 %0, %4, 0x1000\n\tbuffer_load_dwordx4 %2, %3, %0 offen sc0 sc1 lds"
                 : "=&s"(t) : "s"(lds), "v"(voff), "s"(rs), "s"(soff) : "memory", "scc");
}
__device__ __forceinline__ i32x8 cat(u32x4 a, u32x4 b) {
  return __builtin_bit_cast(i32x8, __builtin_shufflevector(a, b, 0, 1, 2, 3, 4, 5, 6, 7));
}

constexpr int sendN(int NG, int w) {
  int n = 0;
  for (int t = w; t < w + SWIN; ++t) {
    if (t > w && t >= SWIN && t + SWIN < 8 * NG) ++n;
    if (t > w && t < 8 * NG) ++n;
    if (t - SWIN >= 0 && (t - SWIN + 1) % NG == 0) ++n;
  }
  return n;
}
template <int NG> struct SD {
  u32x4 v[8 * NG];
  __amdgpu_buffer_rsrc_t rsX;
  unsigned vx, sx, lv0, call;
  unsigned char* dst;
  unsigned* flag;
  int lane;
};
template <int NG, int S> __device__ __forceinline__ void send_load(SD<NG>& d) {
  asm volatile("buffer_load_dwordx4 %0, %1, %2, %3 offen" : "=v"(d.v[S])
               : "v"(d.vx), "s"(d.rsX), "s"(d.sx + ((d.lv0 + S / NG) & 7) * 64 + (S % NG) * 16 * KS) : "memory");
}
template <int NG, int S> __device__ __forceinline__ void send_first(SD<NG>& d) {
  send_load<NG, S>(d);
  if constexpr (S + 1 < 2 * SWIN) send_first<NG, S + 1>(d);
}
template <int NG, int S> __device__ __forceinline__ void send_iter(SD<NG>& d) {
  constexpr int ST = 8 * NG, W = S - SWIN;
  if constexpr (W >= 0) {
    constexpr int N = sendN(NG, W);
    VMWAIT(N);
    if constexpr (S + SWIN < ST) send_load<NG, S + SWIN>(d);
  }
  if constexpr (S < ST) {
    if constexpr (W < 0) asm volatile("s_waitcnt vmcnt(%1)" : "+v"(d.v[S]) : "i"(2 * SWIN - 1) : "memory");
    else asm volatile("" : "+v"(d.v[S]));
    asm volatile("global_store_dwordx4 %0, %1, off sc0 sc1\n\ts_nop 1" :: "v"(d.dst + (S / NG) * 32768 + (S % NG) * 1024), "v"(d.v[S]) : "memory");
  }
  if constexpr (W >= 0 && (W + 1) % NG == 0) {
    const unsigned val = (d.call - 1) * 8 + W / NG + 1;
    if (d.lane == 0) asm volatile("global_store_dword %0, %1, off sc0 sc1" :: "v"(d.flag), "v"(val) : "memory");
  }
  if constexpr (S + 1 < ST + SWIN) send_iter<NG, S + 1>(d);
}
template <int NG>
__device__ __forceinline__ unsigned send_stream(__amdgpu_buffer_rsrc_t rsX, unsigned vx, unsigned sx, unsigned lv0, unsigned char* dst0,
                                                unsigned* flag, GAS unsigned* cnt, int lane, unsigned* ts) {
  static_assert(2 * SWIN <= 8 * NG, "window");
  SD<NG> d;
  d.rsX = rsX; d.vx = vx; d.sx = sx; d.lv0 = lv0; d.flag = flag; d.lane = lane;
  unsigned c0;
  asm volatile("global_load_dword %0, %1, off sc1" : "=v"(c0) : "v"(cnt) : "memory");
  send_first<NG, 0>(d);
  asm volatile("s_waitcnt vmcnt(%1)" : "+v"(c0) : "i"(2 * SWIN) : "memory");
  d.call = c0 + 1;
  d.dst = dst0 + (d.call & 1) * SLOT_BYTES;
  send_iter<NG, 0>(d);
  return d.call;
}

template <int NG> struct CW {
  f32x4 acc[NG][2];
  i32x8 bf[32][2];
  i32x8 af[NG];
  LAS unsigned char* rb[2][2];
  LAS unsigned char* ra[2];
  LAS unsigned char* rl[2];
};
#define STAMP(x)
#define STAMPC(x)
template <int NG, int U, int CT> __device__ __forceinline__ void read_b(CW<NG>& c) {
  constexpr int S = (2 * (U < 4 ? U : U >= 32 - NF ? U - (28 - NF) : U + NF)) % NB;
  LAS unsigned char* p = c.rb[CT][S >> 3] + (S & 7) * 8192;
  i32x2 a = __builtin_amdgcn_ds_read_tr8_b64_v2i32((LAS i32x2*)(p));
  i32x2 b = __builtin_amdgcn_ds_read_tr8_b64_v2i32((LAS i32x2*)(p + 1024));
  i32x2 e = __builtin_amdgcn_ds_read_tr8_b64_v2i32((LAS i32x2*)(p + 2048));
  i32x2 d = __builtin_amdgcn_ds_read_tr8_b64_v2i32((LAS i32x2*)(p + 3072));
  i32x8 r; r[0] = a[0]; r[1] = a[1]; r[2] = b[0]; r[3] = b[1]; r[4] = e[0]; r[5] = e[1]; r[6] = d[0]; r[7] = d[1];
  c.bf[U][CT] = r;
}
template <int NG, int U, int M> __device__ __forceinline__ void read_a(CW<NG>& c) {
  constexpr int O = ((2 * U) % NA) * 5120 + M * (U < 4 ? 2048 : 1024);
  if constexpr (U < 4) c.af[M] = cat(*(LAS u32x4*)(c.rl[0] + O), *(LAS u32x4*)(c.rl[1] + O));
  else c.af[M] = cat(*(LAS u32x4*)(c.ra[0] + O), *(LAS u32x4*)(c.ra[1] + O));
}
template <int NG, int U, int M> __device__ __forceinline__ void read_a_all(CW<NG>& c) {
  read_a<NG, U, M>(c);
  if constexpr (M + 1 < NG) read_a_all<NG, U, M + 1>(c);
}
template <int NG, int U, int E> __device__ __forceinline__ void prefill(CW<NG>& c) {
  asm volatile("s_waitcnt lgkmcnt(0)" ::: "memory");
  TICK();
  STAMP(c.pt[U - (32 - NF)]);
  read_b<NG, U, 0>(c); read_b<NG, U, 1>(c);
  SCHED();
  if constexpr (U + 1 < E) prefill<NG, U + 1, E>(c);
}
template <int NG, int U, int M> __device__ __forceinline__ void row_tile(CW<NG>& c) {
  constexpr bool RA = U != 3 && U + 1 < 32, RB = U != 3 && U + 1 < 32 - NF;
  c.acc[M][0] = __builtin_amdgcn_mfma_scale_f32_16x16x128_f8f6f4(c.bf[U][0], c.af[M], c.acc[M][0], 1, 1, 0, 0, 0, 0);
  c.acc[M][1] = __builtin_amdgcn_mfma_scale_f32_16x16x128_f8f6f4(c.bf[U][1], c.af[M], c.acc[M][1], 1, 1, 0, 0, 0, 0);
  SCHED();
  if constexpr (M == 0 && RA) TICK();
  if constexpr (RA) read_a<NG, U + 1, M>(c);
  if constexpr (RB && M < 2) read_b<NG, U + 1, M>(c);
  SCHED();
  if constexpr (M + 1 < NG) row_tile<NG, U, M + 1>(c);
}
template <int NG, int U, int E> __device__ __forceinline__ void steps(CW<NG>& c) {
  row_tile<NG, U, 0>(c);
  if constexpr (U + 1 < E) steps<NG, U + 1, E>(c);
}
template <int NG, int U> __device__ __forceinline__ void start(CW<NG>& c) {
  TICK();
  read_b<NG, U, 0>(c); read_b<NG, U, 1>(c);
  SCHED();
  read_a_all<NG, U, 0>(c);
  SCHED();
}

struct LD {
  __amdgpu_buffer_rsrc_t rsA, rsF, rsW;
  GAS unsigned* tmo;
  volatile LAS unsigned* seen;
  unsigned va, aoff, lda0, lda, vw0, vw1, woff, ldw0, ldw, scratch, vflag, need, call;
  u64 arrived, tp;
  unsigned dead;
  int lane;
};
__device__ __forceinline__ unsigned poll_off(const LD& x) {
  return __builtin_amdgcn_inverse_ballot_w64(x.arrived & ~0xffull) ? OOR : x.vflag;
}
__device__ __forceinline__ void poll_use(LD& x, unsigned r) { x.arrived |= __ballot((int)(r - x.need) > 0) & ~0xffull; }
__device__ __forceinline__ void poll_async(LD& x, bool on) {
  asm volatile("s_mov_b32 m0, %0\n\ts_nop 0\n\tbuffer_load_dword %1, %2, %3 offen sc0 sc1 lds"
               :: "s"(on ? x.scratch : x.scratch + 256), "v"(x.vflag), "s"(x.rsF), "s"(on ? 0u : OOR) : "memory");
}
__device__ __forceinline__ bool low(const LD& x, u64 want, u64 all) {
  return __builtin_popcountll(x.arrived & want) <= PLOW && (~x.arrived & all) != 0;
}
__device__ __forceinline__ unsigned pick(LD& x, u64 want, u64 first = ~0ull) {
  while (__builtin_expect((x.arrived & want) == 0, 0)) {
    unsigned r;
    asm volatile("buffer_load_dword %0, %1, %2, 0 offen sc0 sc1\n\ts_waitcnt vmcnt(0)" : "=&v"(r) : "v"(poll_off(x)), "s"(x.rsF) : "memory");
    poll_use(x, r);
    if (__ballot(x.lane < 8 && r == x.call) != 0 || rt() - x.tp > DEADLINE_TICKS) {
      __hip_atomic_store(x.tmo, x.call, __ATOMIC_RELAXED, __HIP_MEMORY_SCOPE_AGENT);
      x.dead = OOR; x.arrived = ~0ull;
    }
  }
  const u64 c = x.arrived & want;
  return __builtin_ctzll((c & first) ? (c & first) : c);
}
__device__ __forceinline__ unsigned load_w_head(LD& x, unsigned q) {
  const unsigned so = __builtin_amdgcn_readlane(x.woff, q);
  dma_w_head(x.ldw, x.vw0, x.rsW, so);
  return so;
}
__device__ __forceinline__ void load_w_tail(LD& x, unsigned so) {
  dma_w_tail(x.ldw, x.vw0, x.vw1, x.rsW, so);
  x.ldw = x.ldw == x.ldw0 + 8 * 8192 ? x.ldw0 : x.ldw + 4 * 8192;
}
__device__ __forceinline__ void load_w(LD& x, unsigned q) { load_w_tail(x, load_w_head(x, q)); }
template <int NG> __device__ __forceinline__ void load_x(LD& x, unsigned q) {
  dma_piece<NG>(x.lda, x.va, x.rsA, __builtin_amdgcn_readlane(x.aoff, q) | x.dead);
  x.lda = x.lda == x.lda0 + 8 * 5120 ? x.lda0 : x.lda + 4 * 5120;
}

struct Tile { const unsigned char* ax; const unsigned char* aw; float* aout; unsigned char* myws; int rank, b, tile, nt, sx, slow, help; };

template <int NG>
__device__ __forceinline__ void tile_body(const Args& a, const Tile& t, LAS unsigned char* L, const int wave, const int lane) {
  const unsigned lbase = (unsigned)(size_t)L;
  LAS unsigned* ctrl = (LAS unsigned*)(L + CTRL_OFF);
  const int rank = t.rank, g0 = tile_g0(t.tile);
  GAS unsigned* cnt = (GAS unsigned*)(t.myws + OFF_CNT + (size_t)t.b * 128);
  unsigned call = 0;
  if (wave >= 4) {
    __builtin_amdgcn_s_setprio(2);
    const unsigned j = wave & 1;
    unsigned psrc, plvl;
    {
      const unsigned p = lane - 8, l7 = (p * 37u) >> 8;
      unsigned s = p - l7 * 7u + t.tile;
      s = umin(s, s - 7u);
      psrc = lane < 8 ? rank : (rank + 1 + s) & 7u;
      plvl = lane < 8 ? (lane + 2 * t.tile) & 7u : l7;
    }
    const unsigned P = (wave >> 1) & 1;
    unsigned c0;
    asm volatile("global_load_dword %0, %1, off sc1" : "=v"(c0) : "v"(cnt) : "memory");
    LD x;
    const unsigned rot = (t.tile + 2 * wave) & 7;
    x.rsW = __builtin_amdgcn_make_buffer_rsrc((void*)t.aw, (short)0, NN * NN, 0x00020000);
    x.woff = (psrc * 8 + (lane < 8 ? plvl : (plvl + (t.tile & 6)) & 7)) * (64u * NN) + t.nt * 128 + (rot << 15);
    x.vw0 = (lane >> 3) * NN + (((lane & 7) ^ ((lane >> 3) & 6) ^ (rot >> 2)) << 4); x.vw1 = x.vw0 ^ 16u;
    x.ldw = x.ldw0 = (lbase + (2 * P + j) * 8192) | (rot << 10);
    load_w(x, 2 * P + j);
    load_w(x, 2 * P + 4 + j);
    unsigned q = 2 * (32 - NF + P) + j;
    x.rsA = __builtin_amdgcn_make_buffer_rsrc((void*)(t.myws + OFF_DATA), (short)0, (int)(2 * SLOT_BYTES), 0x00020000);
    x.rsF = __builtin_amdgcn_make_buffer_rsrc((void*)t.myws, (short)0, (int)OFF_DBG, 0x00020000);
    x.tmo = (GAS unsigned*)(t.myws + OFF_TMO);
    x.va = (lane >> 2) * 64 + (((lane & 3) ^ (lane >> 5)) << 4);
    x.vflag = lane >= 8 ? (unsigned)OFF_FLAG + (t.tile * 8 + psrc) * 128 : (unsigned)OFF_TMO;
    x.lane = lane; x.arrived = 0; x.dead = 0;
    x.lda0 = lbase + LDS_A + (2 * (1 - P) + j) * 5120; x.lda = x.lda0 + 8 * 5120;
    x.scratch = lbase + CTRL_OFF + 128;
    x.seen = (volatile LAS unsigned*)(L + CTRL_OFF + 128) + lane;
    if (wave == 4) *x.seen = 0;
    if (P) TICK();
    asm volatile("s_waitcnt vmcnt(8) lgkmcnt(0)" : "+v"(c0) :: "memory");
    TICK();
    x.call = call = rfl(c0) + 1;
    x.need = (call - 1) * 8 + plvl;
    x.aoff = (call & 1) * (unsigned)SLOT_BYTES + (psrc * 8 + plvl) * 32768u + g0 * 1024;
    x.tp = rt();
#pragma unroll 1
    for (unsigned i = P; i < 4; ++i) {
      poll_async(x, wave == 4 && i == 2);
      load_w(x, q); q += 4;
      TICK(); VMWAIT(8 + 1); TICK();
    }
    poll_use(x, *x.seen);
    const u64 mine = 0x1111111111111111ull << (2 * (1 - P) + j);
    u64 want = mine & 0x0003ffffffffff00ull;
    const u64 first = ~0ull << (8 + PROT * t.sx);
#pragma unroll 1
    for (unsigned i = 1 - P; i < 11; ++i) {
      q = pick(x, want, first);
      want &= ~(1ull << q);
      poll_async(x, low(x, want, mine & ~0xffull));
      load_x<NG>(x, q);
      const unsigned so = load_w_head(x, q);
      TICK();
      load_w_tail(x, so);
      VMWAIT(1 + NG + 8);
      const unsigned r = *x.seen;
      TICK();
      poll_use(x, r);
    }
    DBG(24 + j, rt() - t0);
    want = mine & 0xfffc000000000000ull;
#pragma unroll 1
    for (unsigned i = P; i < 4; ++i) {
      q = pick(x, want & -want);
      want &= want - 1;
      poll_async(x, (~x.arrived & want) != 0);
      load_x<NG>(x, q);
      TICK(); VMWAIT(1 + NG);
      const unsigned r = *x.seen;
      TICK();
      poll_use(x, r);
    }
    unsigned tw;
    asm volatile("global_load_dword %0, %1, off sc1" : "=v"(tw) : "v"(x.tmo) : "memory");
    TICK(); VMWAIT(1); TICK();
    if (P) TICK();
    DBG(20 + j, rt() - t0);
    asm volatile("s_waitcnt vmcnt(0)" : "+v"(tw) :: "memory");
    *(volatile LAS unsigned*)(ctrl + C_STATE + wave - 4) = (x.dead || tw == call) ? 2u : 1u;
    __builtin_amdgcn_s_setprio(0);
  } else {
    CW<NG> c;
    const __amdgpu_buffer_rsrc_t rsX = __builtin_amdgcn_make_buffer_rsrc((void*)t.ax, (short)0, NN * KS, 0x00020000);
    if (wave == 1 || wave == 2) {
      const unsigned u = t.sx * NTILE + t.tile + 28 * (wave - 1), k = (u * 52) >> 8, sid = (t.b & 7) + 8 * k, g = u - 5 * k;
      const unsigned st = (sid * 37) >> 8, dd = sid - 7 * st;
      if (sid < (NRANK - 1) * NTILE && g < (st < 4 ? 5u : 4u))
        touch(lbase + LDS_JUNK, (lane >> 2) * KS + (lane & 3) * 128, rsX, (((rank + 1 + dd) & 7) * KS + (tile_g0(st) + g) * 16) * KS);
    }
    const __amdgpu_buffer_rsrc_t rsW = __builtin_amdgcn_make_buffer_rsrc((void*)t.aw, (short)0, NN * NN, 0x00020000);
    if (wave == 3 && t.help)
      touch(lbase + LDS_JUNK, lane < 25 ? umin(((t.help - 1) * NTILE + t.tile) * 25 + lane, 511) * NN : OOR, rsW, rank * (512u * NN) + t.slow * 128);
    {
      const unsigned v0 = (lane >> 3) * KS + (((lane & 7) ^ (lane >> 4)) << 4), v1 = v0 ^ 64u;
      const unsigned so = (rank * KS + g0 * 16) * KS + ((wave + t.tile) & 3) * 128, ld = lbase + LDS_A + wave * 10240;
      dma_a<0 * 1024, 0 * 4096>(ld, v0, rsX, so); dma_a<1 * 1024, 1 * 4096>(ld, v1, rsX, so);
      dma_a<2 * 1024, 2 * 4096>(ld, v0, rsX, so); dma_a<3 * 1024, 3 * 4096>(ld, v1, rsX, so);
      dma_a<4 * 1024, 4 * 4096>(ld, v0, rsX, so); dma_a<5 * 1024, 5 * 4096>(ld, v1, rsX, so);
      dma_a<6 * 1024, 6 * 4096>(ld, v0, rsX, so); dma_a<7 * 1024, 7 * 4096>(ld, v1, rsX, so);
      if constexpr (NG == 5) { dma_a<8 * 1024, 8 * 4096>(ld, v0, rsX, so); dma_a<9 * 1024, 9 * 4096>(ld, v1, rsX, so); }
      const int g = lane >> 4, r = lane & 15;
      unsigned l0 = lbase + LDS_A + r * 128 + (((2 * g) ^ (r >> 1)) << 4), l1 = l0 ^ 16u;
      asm volatile("" : "+v"(l0), "+v"(l1));
      c.rl[0] = (LAS unsigned char*)(size_t)l0; c.rl[1] = (LAS unsigned char*)(size_t)l1;
    }
    {
      const int g = lane >> 4, q = (lane & 15) >> 1, p = lane & 1, r = lane & 15;
      const unsigned b0 = lbase + g * 4096 + q * 128 + (((wave << 1) ^ (q & 6) ^ (g & 1)) << 4) + p * 8;
      unsigned b1 = b0 ^ 16u, b2 = b0 + 65536, b3 = b1 + 65536;
      asm volatile("" : "+v"(b2), "+v"(b3));
      c.rb[0][0] = (LAS unsigned char*)(size_t)b0; c.rb[1][0] = (LAS unsigned char*)(size_t)b1;
      c.rb[0][1] = (LAS unsigned char*)(size_t)b2; c.rb[1][1] = (LAS unsigned char*)(size_t)b3;
      unsigned a0 = lbase + LDS_A + (g >> 1) * 5120 + r * 64 + (((2 * (g & 1)) ^ (r >> 3)) << 4), a1 = a0 ^ 16u;
      asm volatile("" : "+v"(a0), "+v"(a1));
      c.ra[0] = (LAS unsigned char*)(size_t)a0; c.ra[1] = (LAS unsigned char*)(size_t)a1;
    }
#pragma unroll
    for (int m = 0; m < NG; ++m)
#pragma unroll
      for (int r = 0; r < 4; ++r) { c.acc[m][0][r] = 0.f; c.acc[m][1][r] = 0.f; }
    if (wave == 0) {
      const float sc = *(const __attribute__((address_space(4))) float*)a.sx * *(const __attribute__((address_space(4))) float*)a.sw;
      *(volatile LAS unsigned*)(ctrl + C_STATE + 4) = __builtin_bit_cast(unsigned, sc);
    }
    VMWAIT(0);
    start<NG, 0>(c);
    if (wave == 3 && t.help) {
      const unsigned h = (t.help - 1) * 2 + (t.tile & 1);
#pragma unroll
      for (int i = 0; i < 2; ++i) {
        touch(lbase + LDS_JUNK, lane * NN, rsW, (((rank + 1 + h) & 7) * 8 + ((6 + i + (t.tile & 6)) & 7)) * (64u * NN) + t.slow * 128);
        if (t.tile == 6) touch(lbase + LDS_JUNK, lane * NN, rsW, (((rank + 2 + h) & 7) * 8 + 4 + i) * (64u * NN) + t.slow * 128);
      }
    }
    steps<NG, 0, 4>(c);
    prefill<NG, 32 - NF, 32>(c);
    STAMP(c.st[16]);
    asm volatile("s_waitcnt lgkmcnt(0)" ::: "memory");
    start<NG, 4>(c);
    steps<NG, 4, 32>(c);
    DBG(wave, rt() - t0);
    const unsigned wr = lbase + (lane & 15) * 512 + wave * 128 + (((lane >> 4) ^ (lane & 7)) << 4);
#pragma unroll
    for (int m = 0; m < NG; ++m) {
      *(LAS f32x4*)(size_t)(wr + m * 8192) = c.acc[m][0];
      *(LAS f32x4*)(size_t)((wr ^ 64u) + m * 8192) = c.acc[m][1];
    }
  }
  const __amdgpu_buffer_rsrc_t rsO = __builtin_amdgcn_make_buffer_rsrc((void*)t.aout, (short)0, KS * NN * 4, 0x00020000);
  const unsigned row = wave * 2 + (lane >> 5);
  const unsigned vo = row * (NN * 4) + (lane & 31) * 16;
  const unsigned so = (g0 * 16 * NN + t.nt * 128) * 4;
  const unsigned rd = lbase + row * 512 + (((lane & 31) ^ (row & 7)) << 4);
  asm volatile("s_waitcnt lgkmcnt(0)" ::: "memory");
  __builtin_amdgcn_s_barrier();
  asm volatile("" ::: "memory");
  if (wave < 4) DBG(16 + wave, rt() - t0);
  const u32x4 st = *(volatile LAS u32x4*)(ctrl + C_STATE);
  f32x4 s[NG];
#pragma unroll
  for (int m = 0; m < NG; ++m) s[m] = *(LAS f32x4*)(size_t)(rd + m * 8192);
  SCHED();
  const unsigned sbits = *(volatile LAS unsigned*)(ctrl + C_STATE + 4);
  const float scale = (st[0] & st[1] & st[2] & st[3]) == 1u ? __builtin_bit_cast(float, sbits) : __builtin_nanf("");
  const float scale2 = -1.44269504088896341f * scale;
#pragma unroll
  for (int m = 0; m < NG; ++m) {
    f32x4 e, z;
#pragma unroll
    for (int r = 0; r < 4; ++r) e[r] = __builtin_amdgcn_exp2f(__builtin_amdgcn_fmed3f(s[m][r] * scale2, -86.5617024533378f, 86.5617024533378f));
#pragma unroll
    for (int r = 0; r < 4; ++r) z[r] = (s[m][r] * scale) * __builtin_amdgcn_rcpf(1.f + e[r]);
    __builtin_amdgcn_raw_buffer_store_b128(__builtin_bit_cast(u32x4, z), rsO, vo, so + m * 16 * (NN * 4), 16);
    SCHED();
    asm volatile("s_nop 1");
    SCHED();
  }
  if (wave == 4 && lane == 0) __hip_atomic_store(cnt, call, __ATOMIC_RELAXED, __HIP_MEMORY_SCOPE_AGENT);
}
}

__global__ void __launch_bounds__(512, 2) a2a_gemm_kernel(Args a) {
  extern __shared__ __attribute__((aligned(16))) unsigned char lds_raw[];
  const int wave = rfl(threadIdx.x >> 6), lane = threadIdx.x & 63;
  const int b = blockIdx.x;
  Tile t;
  t.ax = a.x; t.aw = a.w; t.aout = a.out; t.myws = a.myws; t.rank = a.rank; t.b = b;
  if (b < NSEND) {
    const int sid = b + NSEND * wave;
    if (sid >= (NRANK - 1) * NTILE) return;
    const int tile = (sid * 37) >> 8, dd = sid - 7 * tile, g0 = tile_g0(tile);
    unsigned char* pw = a.peer[dd];
    asm volatile("" : "+s"(t.ax), "+s"(t.myws), "+s"(t.rank), "+s"(pw));
    const unsigned dest = (unsigned)(t.rank + 1 + dd) & 7u;
    const __amdgpu_buffer_rsrc_t rsX = __builtin_amdgcn_make_buffer_rsrc((void*)t.ax, (short)0, NN * KS, 0x00020000);
    const unsigned vx = (lane >> 2) * KS + (lane & 3) * 16, sx = (dest * KS + g0 * 16) * KS;
    GAS unsigned* cnt = (GAS unsigned*)(t.myws + OFF_CNT + (size_t)b * 128 + wave * 64);
    unsigned char* dst = pw + OFF_DATA + (size_t)t.rank * 8 * 32768 + g0 * 1024 + lane * 16;
    unsigned* flag = (unsigned*)(pw + OFF_FLAG + (tile * 8 + t.rank) * 128);
    unsigned ts[10];
    const unsigned call = tile < 4 ? send_stream<5>(rsX, vx, sx, tile & 6, dst, flag, cnt, lane, ts) : send_stream<4>(rsX, vx, sx, tile & 6, dst, flag, cnt, lane, ts);
    DBG(26 + wave, rt() - t0);
    if (lane == 0) __hip_atomic_store(cnt, call, __ATOMIC_RELAXED, __HIP_MEMORY_SCOPE_AGENT);
    return;
  }
  asm volatile("" : "+s"(t.ax), "+s"(t.aw), "+s"(t.aout), "+s"(t.myws), "+s"(t.rank));
  const int cj = (b - NSEND) >> 3;
  t.tile = cj % NTILE; t.sx = cj / NTILE;
  t.nt = t.sx * 8 + ((b + 2 * t.sx) & 7);
  {
    const unsigned d = (3u - ((unsigned)(size_t)t.aw >> 7) - b) & 7u, ss = d >> 1;
    t.slow = (d & 1) ? -1 : ss * 8 + ((b + 2 * ss) & 7);
    t.help = (d & 1) ? 0 : (t.sx - ss) & 3;
  }
  if (t.tile < 4) tile_body<5>(a, t, (LAS unsigned char*)lds_raw, wave, lane);
  else tile_body<4>(a, t, (LAS unsigned char*)lds_raw, wave, lane);
}

extern "C" void kernel_launch(int rank, int nranks, ncclComm_t comm, hipStream_t stream,
                              void* const* d_in, const int* in_sizes, int n_in,
                              void* d_out, int out_size, const int* in_dtypes, int out_dtype,
                              void* const* d_out_all, void* const* d_ws_all, size_t ws_bytes) {
  if (nranks != NRANK || (unsigned)rank >= (unsigned)NRANK || ws_bytes < WS_NEED || n_in < 4 ||
      in_sizes[0] != NN * KS || in_sizes[1] != NN * NN || out_size != KS * NN) {
    fprintf(stderr, "a2a_gemm: unexpected problem shape\n");
    return;
  }
  static bool attr_set = false;
  if (!attr_set) {
    HIP_CHECK(hipFuncSetAttribute((const void*)a2a_gemm_kernel, hipFuncAttributeMaxDynamicSharedMemorySize, LDS_TOTAL));
    attr_set = true;
  }
  Args a;
  a.x = (const unsigned char*)d_in[0]; a.w = (const unsigned char*)d_in[1];
  a.sx = (const float*)d_in[2]; a.sw = (const float*)d_in[3];
  a.out = (float*)d_out;
  a.myws = (unsigned char*)d_ws_all[rank];
  for (int p = 0; p < NRANK; ++p) a.peer[p] = (unsigned char*)d_ws_all[(rank + 1 + p) % NRANK];
  a.rank = rank; a.pad = 0;
  a2a_gemm_kernel<<<NWG, 512, LDS_TOTAL, stream>>>(a);
  HIP_CHECK(hipGetLastError());
}
```

```cpp
#include <hip/hip_runtime.h>
#include <rccl/rccl.h>
#include <cstdio>

#define HIP_CHECK(expr) do { hipError_t _e = (expr); if (_e != hipSuccess) { \
    fprintf(stderr, "HIP error %s at %s:%d\n", hipGetErrorString(_e), __FILE__, __LINE__); \
    return; } } while (0)

#define LAS __attribute__((address_space(3)))
#define GAS __attribute__((address_space(1)))
typedef unsigned u32x2 __attribute__((ext_vector_type(2)));
typedef unsigned u32x4 __attribute__((ext_vector_type(4)));
typedef int i32x2 __attribute__((ext_vector_type(2)));
typedef int i32x8 __attribute__((ext_vector_type(8)));
typedef float f32x4 __attribute__((ext_vector_type(4)));
typedef unsigned long long u64;

#define SW5 8
#define SW2 2
#define SWIN (NG == 5 ? SW5 : SW2)
#define HA 6
#define HB 6
#define HC 6
#define PW 4
#define PI 1
#define HOLD4 0
#define XLT 3
#define TS 7
#define TSS 4
#define WARMI 0
#define PLOW 1

namespace {
constexpr int NRANK = 8;
constexpr int KS = 512;
constexpr int NN = 4096;
constexpr int NWG = 256;
constexpr int NSEND = 32;
constexpr int NTILE = 7;
constexpr size_t OFF_CNT = 0;
constexpr size_t OFF_TMO = 32768;
constexpr size_t OFF_FLAG = 65536;
constexpr size_t OFF_DBG = 262144;
constexpr size_t OFF_DATA = 1u << 20;
constexpr size_t SLOT_BYTES = 2u << 20;
constexpr size_t WS_NEED = OFF_DATA + 2 * SLOT_BYTES;
constexpr u64 DEADLINE_TICKS = 100000000ull;
constexpr int NB = 12;
constexpr int NA = 12;
constexpr int LDS_A = NB * 8192;
constexpr int CTRL_OFF = LDS_A + NA * 5120;
constexpr int LDS_JUNK = CTRL_OFF + 128 + 1024;
constexpr int LDS_TOTAL = LDS_JUNK + 256;
constexpr int C_STATE = 0;
constexpr int NF = 7;
constexpr bool fixed(int u) { return u >= 4 && (u & 3) == 3; }
constexpr int witem(int u) { return fixed(u) ? u / 4 - 1 : u + NF; }
constexpr int fq(int k, int j) { return 8 + 7 * k + (k + 3 * j) % 7; }
constexpr u64 fmask(int j) { u64 m = 0; for (int k = 0; k < NF; ++k) m |= 1ull << fq(k, j); return m; }
constexpr u64 dmask(int P, int j) {
  u64 m = 0;
  int u = 4, h = 0;
  for (int q = 8; q < 64; ++q) {
    if (((fmask(0) | fmask(1)) >> q) & 1) continue;
    if (((u + 7) & 1) == P && h == j) m |= 1ull << q;
    if (++h == 2) { h = 0; ++u; if (fixed(u)) ++u; }
  }
  return m;
}
constexpr unsigned levels(int s) {
  unsigned m = 0, nf = 0, nd = 2;
  for (int i = 0; i < 8; ++i) m |= ((((fmask(0) | fmask(1)) >> (8 + 7 * i + s)) & 1) ? nf++ : nd++) << (3 * i);
  return m;
}
__host__ __device__ constexpr unsigned level_add(int tile) { return tile < 4 ? 0 : 2 * tile - 6; }
constexpr int kfix(int q) { return q < 8 ? q : ((q - 8) % 7 + 1) * 8 + ((levels((q - 8) % 7) >> (3 * ((q - 8) / 7))) & 7); }
constexpr u64 wseq(int a, int b, int c, int d, int e, int f) {
  return (u64)kfix(a) | (u64)kfix(b) << 8 | (u64)kfix(c) << 16 | (u64)kfix(d) << 24 | (u64)kfix(e) << 32 | (u64)kfix(f) << 40;
}
constexpr u64 FM0 = fmask(0), FM1 = fmask(1), DM00 = dmask(0, 0), DM01 = dmask(0, 1), DM10 = dmask(1, 0), DM11 = dmask(1, 1);
constexpr unsigned OOR = 0x80000000u;
static_assert(NB == 12 && NA == 12, "ring schedule");

struct Stream {
  unsigned char* peer;
  unsigned lv;
  unsigned pos;
};
struct Args {
  const unsigned char* x; const unsigned char* w; const float* sx; const float* sw; float* out;
  unsigned char* myws;
  int rank; int pad[3];
  Stream st[NSEND * 4];
  unsigned short xt[NWG];
};

__device__ __forceinline__ u64 rt() { return __builtin_amdgcn_s_memrealtime(); }
__device__ __forceinline__ unsigned rfl(unsigned v) { return __builtin_amdgcn_readfirstlane(v); }
__device__ __forceinline__ unsigned umin(unsigned a, unsigned b) { return a < b ? a : b; }
#define SCHED() __builtin_amdgcn_sched_barrier(0)
#define TICK() do { asm volatile("" ::: "memory"); __builtin_amdgcn_s_barrier(); asm volatile("" ::: "memory"); } while (0)
#define VMWAIT(n) asm volatile("s_waitcnt vmcnt(%0)" :: "i"(n) : "memory")
__host__ __device__ constexpr int tile_g0(int t) { return t < 4 ? 5 * t : 4 * t + 4; }

#define DBG(i, v)

template <int G, int E>
__device__ __forceinline__ void dma_w(unsigned lds, unsigned v0, unsigned v1, __amdgpu_buffer_rsrc_t rs, unsigned soff) {
  if constexpr (G + 1 < E) {
    unsigned t;
    asm volatile("s_xor_b32 m0, %1, %6\n\ts_xor_b32 %0, %5, %7\n\tbuffer_load_dwordx4 %2, %4, %0 offen lds\n\t"
                 "s_xor_b32 m0, %1, %8\n\ts_xor_b32 %0, %5, %9\n\tbuffer_load_dwordx4 %3, %4, %0 offen lds"
                 : "=&s"(t) : "s"(lds), "v"(G < 4 ? v0 : v1), "v"(G + 1 < 4 ? v0 : v1), "s"(rs), "s"(soff),
                   "i"(G << 10), "i"(G << 15), "i"((G + 1) << 10), "i"((G + 1) << 15) : "memory", "scc");
    dma_w<G + 2, E>(lds, v0, v1, rs, soff);
  } else if constexpr (G < E) {
    unsigned t;
    asm volatile("s_xor_b32 m0, %1, %5\n\ts_xor_b32 %0, %4, %6\n\tbuffer_load_dwordx4 %2, %3, %0 offen lds"
                 : "=&s"(t) : "s"(lds), "v"(G < 4 ? v0 : v1), "s"(rs), "s"(soff), "i"(G << 10), "i"(G << 15) : "memory", "scc");
  }
}
__device__ __forceinline__ void touch(unsigned lds, unsigned voff, __amdgpu_buffer_rsrc_t rs, unsigned soff) {
  asm volatile("s_mov_b32 m0, %0\n\ts_nop 3\n\tbuffer_load_dword %1, %2, %3 offen lds" :: "s"(lds), "v"(voff), "s"(rs), "s"(rfl(soff)) : "memory");
}
template <unsigned LO, unsigned SO>
__device__ __forceinline__ void dma_a(unsigned lds, unsigned voff, __amdgpu_buffer_rsrc_t rs, unsigned soff) {
  unsigned t;
  asm volatile("s_add_u32 m0, %1, %5\n\ts_add_u32 %0, %4, %6\n\tbuffer_load_dwordx4 %2, %3, %0 offen lds"
               : "=&s"(t) : "s"(lds), "v"(voff), "s"(rs), "s"(soff), "i"(LO), "i"(SO) : "memory", "scc");
}
template <int NG>
__device__ __forceinline__ void dma_piece(unsigned lds, unsigned voff, __amdgpu_buffer_rsrc_t rs, unsigned soff) {
  unsigned t;
  asm volatile("s_mov_b32 m0, %1\n\ts_mov_b32 %0, %4\n\t"
               "buffer_load_dwordx4 %2, %3, %0 offen sc0 sc1 lds\n\t"
               "buffer_load_dwordx4 %2, %3, %0 offen offset:1024 sc0 sc1 lds\n\t"
               "buffer_load_dwordx4 %2, %3, %0 offen offset:2048 sc0 sc1 lds\n\t"
               "buffer_load_dwordx4 %2, %3, %0 offen offset:3072 sc0 sc1 lds"
               : "=&s"(t) : "s"(lds), "v"(voff), "s"(rs), "s"(soff) : "memory");
  if constexpr (NG == 5)
    asm volatile("s_add_u32 m0, %1, 0x1000\n\ts_add_u32 %0, %4, 0x1000\n\tbuffer_load_dwordx4 %2, %3, %0 offen sc0 sc1 lds"
                 : "=&s"(t) : "s"(lds), "v"(voff), "s"(rs), "s"(soff) : "memory", "scc");
}
__device__ __forceinline__ i32x8 cat(u32x4 a, u32x4 b) {
  return __builtin_bit_cast(i32x8, __builtin_shufflevector(a, b, 0, 1, 2, 3, 4, 5, 6, 7));
}

constexpr int sendN(int NG, int w) {
  int n = 0;
  for (int t = w; t < w + SWIN; ++t) {
    if (t > w && t < 8 * NG) ++n;
    if (t + SWIN + 1 < 8 * NG) ++n;
    if (t - SWIN >= 0 && (t - SWIN + 1) % NG == 0) ++n;
  }
  return n;
}
template <int NG> struct SD {
  u32x4 v[8 * NG];
  __amdgpu_buffer_rsrc_t rsX;
  unsigned vx, sx, lv, call;
  unsigned char* dst;
  unsigned* flag;
  int lane;
};
template <int NG, int S> __device__ __forceinline__ void send_load(SD<NG>& d) {
  asm volatile("buffer_load_dwordx4 %0, %1, %2, %3 offen" : "=v"(d.v[S])
               : "v"(d.vx), "s"(d.rsX), "s"(d.sx + ((d.lv >> (3 * (S / NG))) & 7) * 64 + (S % NG) * 16 * KS) : "memory");
}
template <int NG, int S> __device__ __forceinline__ void send_first(SD<NG>& d) {
  send_load<NG, S>(d);
  if constexpr (S < SWIN) send_first<NG, S + 1>(d);
}
template <int NG, int S> __device__ __forceinline__ void send_iter(SD<NG>& d) {
  constexpr int ST = 8 * NG, W = S - SWIN;
  if constexpr (W >= 0) {
    constexpr int N = sendN(NG, W);
    VMWAIT(N);
  }
  if constexpr (S < ST) {
    if constexpr (W < 0) asm volatile("s_waitcnt vmcnt(%1)" : "+v"(d.v[S]) : "i"(SWIN + S) : "memory");
    else asm volatile("" : "+v"(d.v[S]));
    asm volatile("global_store_dwordx4 %0, %1, off sc0 sc1\n\ts_nop 1" :: "v"(d.dst + (S / NG) * 32768 + (S % NG) * 1024), "v"(d.v[S]) : "memory");
    if constexpr (S + SWIN + 1 < ST) send_load<NG, S + SWIN + 1>(d);
  }
  if constexpr (W >= 0 && (W + 1) % NG == 0) {
    const unsigned val = (d.call - 1) * 8 + W / NG + 1;
    if (d.lane < (NG == 5 ? 2 : 1)) asm volatile("global_store_dword %0, %1, off sc0 sc1" :: "v"(d.flag), "v"(val) : "memory");
  }
  if constexpr (S + 1 < ST + SWIN) send_iter<NG, S + 1>(d);
}
template <int NG>
__device__ __forceinline__ unsigned send_stream(const unsigned char* x, const Stream& e, int rank, GAS unsigned* cnt, int lane, unsigned* ts) {
  static_assert(2 * SWIN + 1 <= 8 * NG, "window");
  SD<NG> d;
  d.rsX = __builtin_amdgcn_make_buffer_rsrc((void*)x, (short)0, NN * KS, 0x00020000);
  d.vx = (lane >> 2) * KS + (lane & 3) * 16; d.sx = (e.pos & 4095) * KS; d.lv = e.lv; d.lane = lane;
  unsigned c0;
  asm volatile("global_load_dword %0, %1, off sc1" : "=v"(c0) : "v"(cnt) : "memory");
  send_first<NG, 0>(d);
  SCHED();
  d.flag = (unsigned*)(e.peer + OFF_FLAG) + (e.pos >> 17) + (lane & 1);
  unsigned char* dst0 = e.peer + OFF_DATA + (size_t)rank * 8 * 32768 + ((e.pos >> 12) & 31) * 1024 + lane * 16;
  asm volatile("s_waitcnt vmcnt(%1)" : "+v"(c0) : "i"(SWIN + 1) : "memory");
  d.call = c0 + 1;
  d.dst = dst0 + (d.call & 1) * SLOT_BYTES;
  send_iter<NG, 0>(d);
  return d.call;
}

template <int NG> struct CW {
  f32x4 acc[NG][2];
  i32x8 bf[32][2];
  i32x8 af[NG];
  LAS unsigned char* rb[2][2];
  LAS unsigned char* ra[2];
  unsigned wr;
  LAS unsigned char* rl[2];
};
#define STAMP(x)
#define STAMPC(x)
template <int NG, int U, int CT> __device__ __forceinline__ void read_b(CW<NG>& c) {
  constexpr int S = (2 * witem(U)) % NB;
  LAS unsigned char* p = c.rb[CT][S >> 3] + (S & 7) * 8192;
  i32x2 a = __builtin_amdgcn_ds_read_tr8_b64_v2i32((LAS i32x2*)(p));
  i32x2 b = __builtin_amdgcn_ds_read_tr8_b64_v2i32((LAS i32x2*)(p + 1024));
  i32x2 e = __builtin_amdgcn_ds_read_tr8_b64_v2i32((LAS i32x2*)(p + 2048));
  i32x2 d = __builtin_amdgcn_ds_read_tr8_b64_v2i32((LAS i32x2*)(p + 3072));
  i32x8 r; r[0] = a[0]; r[1] = a[1]; r[2] = b[0]; r[3] = b[1]; r[4] = e[0]; r[5] = e[1]; r[6] = d[0]; r[7] = d[1];
  c.bf[U][CT] = r;
}
template <int NG, int U, int M> __device__ __forceinline__ void read_a(CW<NG>& c) {
  constexpr int O = ((2 * U) % NA) * 5120 + M * (U < 4 ? 2048 : 1024);
  if constexpr (U < 4) c.af[M] = cat(*(LAS u32x4*)(c.rl[0] + O), *(LAS u32x4*)(c.rl[1] + O));
  else c.af[M] = cat(*(LAS u32x4*)(c.ra[0] + O), *(LAS u32x4*)(c.ra[1] + O));
}
template <int NG, int U, int M> __device__ __forceinline__ void read_a_all(CW<NG>& c) {
  read_a<NG, U, M>(c);
  if constexpr (M + 1 < NG) read_a_all<NG, U, M + 1>(c);
}
template <int NG, int U, int E> __device__ __forceinline__ void prefill(CW<NG>& c) {
  asm volatile("s_waitcnt lgkmcnt(0)" ::: "memory");
  TICK();
  STAMP(c.pt[U / 4 - 1]);
  read_b<NG, U, 0>(c); read_b<NG, U, 1>(c);
  SCHED();
  if constexpr (U + 4 < E) prefill<NG, U + 4, E>(c);
}
template <int NG, int M> __device__ __forceinline__ void put(CW<NG>& c) {
  *(LAS f32x4*)(size_t)(c.wr + M * 8192) = c.acc[M][0];
  *(LAS f32x4*)(size_t)((c.wr ^ 64u) + M * 8192) = c.acc[M][1];
}
template <int NG, int U, int M> __device__ __forceinline__ void row_tile(CW<NG>& c) {
  constexpr bool RA = U + 1 < 32, RB = RA && !fixed(U + 1);
  c.acc[M][0] = __builtin_amdgcn_mfma_scale_f32_16x16x128_f8f6f4(c.bf[U][0], c.af[M], c.acc[M][0], 1, 1, 0, 0, 0, 0);
  c.acc[M][1] = __builtin_amdgcn_mfma_scale_f32_16x16x128_f8f6f4(c.bf[U][1], c.af[M], c.acc[M][1], 1, 1, 0, 0, 0, 0);
  SCHED();
  if constexpr (M == 0 && RA) TICK();
  if constexpr (RA) read_a<NG, U + 1, M>(c);
  if constexpr (RB && M < 2) read_b<NG, U + 1, M>(c);
  if constexpr (U == 31 && M > 0) put<NG, M - 1>(c);
  if constexpr (U == 31 && M == NG - 1) put<NG, M>(c);
  SCHED();
  if constexpr (M + 1 < NG) row_tile<NG, U, M + 1>(c);
}
template <int NG, int U, int E> __device__ __forceinline__ void steps(CW<NG>& c) {
  row_tile<NG, U, 0>(c);
  if constexpr (U + 1 < E) steps<NG, U + 1, E>(c);
}
template <int NG, int U> __device__ __forceinline__ void start(CW<NG>& c) {
  TICK();
  read_b<NG, U, 0>(c); read_b<NG, U, 1>(c);
  SCHED();
  read_a_all<NG, U, 0>(c);
  SCHED();
}

struct LD {
  __amdgpu_buffer_rsrc_t rsA, rsF, rsW;
  GAS unsigned* tmo;
  volatile LAS u32x2* seen;
  unsigned va, aoff, lda0, lda, vw0, vw1, woff, ldw0, ldw, scratch, vflag, need, call;
  u64 arrived, tp;
  u64 want, fix;
  unsigned dead;
  unsigned sox, sow;
  bool poll;
  int lane;
};
__device__ __forceinline__ void poll_use(LD& x, u32x2 r) {
  x.arrived |= __ballot((int)(r[0] - x.need) > 0 && (int)(r[1] - x.need) > 0) & ~0xffull;
}
__device__ __forceinline__ void poll_async(LD& x) {
  asm volatile("s_mov_b32 m0, %0\n\ts_nop 0\n\tbuffer_load_dwordx4 %1, %2, 0 offen sc0 sc1 lds"
               :: "s"(x.scratch), "v"(x.vflag), "s"(x.rsF) : "memory");
}
template <bool FIX> __device__ __forceinline__ void choose(LD& x) {
  if (FIX) {
    const unsigned q = __builtin_ctzll(x.fix | (1ull << 63));
    const bool ok = (x.arrived >> q) & 1;
    const unsigned so = __builtin_amdgcn_readlane(x.aoff, q);
    x.sox = ok ? so : OOR;
    x.fix &= ~(x.arrived & (1ull << q));
    x.poll = !ok;
  } else {
    const u64 c = x.arrived & x.want;
    const unsigned q = __builtin_ctzll(c | (1ull << 63));
    const unsigned so = __builtin_amdgcn_readlane(x.aoff, q);
    x.sow = __builtin_amdgcn_readlane(x.woff, q);
    x.sox = c ? so : OOR;
    x.poll = __builtin_popcountll(c) <= PLOW + 1 && x.arrived != ~0xffull;
    x.want &= ~(c & (1ull << q));
  }
}
template <bool FIX> __device__ __forceinline__ void choose_wait(LD& x) {
  if (__builtin_expect(x.sox == OOR, 0)) {
    poll_use(x, *x.seen);
    choose<FIX>(x);
    while (x.sox == OOR) {
      u32x2 r;
      asm volatile("buffer_load_dwordx2 %0, %1, %2, 0 offen sc0 sc1\n\ts_waitcnt vmcnt(0)" : "=&v"(r) : "v"(x.vflag), "s"(x.rsF) : "memory");
      *x.seen = r;
      poll_use(x, r);
      if (__ballot(x.lane < 8 && r[0] == x.call) != 0 || rt() - x.tp > DEADLINE_TICKS) {
        __hip_atomic_store(x.tmo, x.call, __ATOMIC_RELAXED, __HIP_MEMORY_SCOPE_AGENT);
        x.dead = OOR; x.arrived = ~0xffull;
      }
      choose<FIX>(x);
    }
  }
}
__device__ __forceinline__ void next_w(LD& x) { x.ldw = x.ldw == x.ldw0 + 8 * 8192 ? x.ldw0 : x.ldw + 4 * 8192; }
__device__ __forceinline__ void load_w(LD& x, unsigned so) {
  dma_w<0, 8>(x.ldw, x.vw0, x.vw1, x.rsW, so);
  next_w(x);
}
template <int NG, bool FIX, bool NEXTFIX, int HEAD> __device__ __forceinline__ void item(LD& x) {
  choose_wait<FIX>(x);
  dma_piece<NG>(x.lda, x.va, x.rsA, x.sox | x.dead);
  if (!FIX) dma_w<0, HEAD>(x.ldw, x.vw0, x.vw1, x.rsW, x.sow);
  const u32x2 r = *x.seen;
  TICK();
  if (!FIX) dma_w<HEAD, 8>(x.ldw, x.vw0, x.vw1, x.rsW, x.sow);
  SCHED();
  x.lda = x.lda == x.lda0 + 8 * 5120 ? x.lda0 : x.lda + 4 * 5120;
  next_w(x);
  poll_use(x, r);
  choose<NEXTFIX>(x);
  SCHED();
  VMWAIT(NG + (FIX ? 0 : 8));
  if (__builtin_expect(x.poll, 0)) poll_async(x);
  TICK();
}

struct Tile { const unsigned char* ax; const unsigned char* aw; float* aout; unsigned char* myws; int rank, b, tile, nt, sx, slow, help; unsigned xt; };

template <int NG>
__device__ __forceinline__ void tile_body(const Args& a, const Tile& t, LAS unsigned char* L, const int wave, const int lane) {
  const unsigned lbase = (unsigned)(size_t)L;
  LAS unsigned* ctrl = (LAS unsigned*)(L + CTRL_OFF);
  const int rank = t.rank, g0 = tile_g0(t.tile);
  GAS unsigned* cnt = (GAS unsigned*)(t.myws + OFF_CNT + (size_t)t.b * 128);
  unsigned call = 0;
  u32x4 st; unsigned sbits;
  if (wave >= 4) {
    __builtin_amdgcn_s_setprio(2);
    const unsigned j = wave & 1;
    const unsigned P = (wave >> 1) & 1;
    unsigned c0;
    asm volatile("global_load_dword %0, %1, off sc1" : "=v"(c0) : "v"(cnt) : "memory");
    LD x;
    const unsigned rot = (t.tile + 2 * wave) & 7;
    x.rsW = __builtin_amdgcn_make_buffer_rsrc((void*)t.aw, (short)0, NN * NN, 0x00020000);
    const unsigned wcol = t.nt * 128 + (rot << 15);
    x.vw0 = (lane >> 3) * NN + (((lane & 7) ^ ((lane >> 3) & 6) ^ (rot >> 2)) << 4); x.vw1 = x.vw0 ^ 16u;
    x.ldw = x.ldw0 = (lbase + (2 * P + j) * 8192) | (rot << 10);
    constexpr u64 W0 = wseq(fq(0, 0), fq(2, 0), fq(4, 0), fq(6, 0), 2, 6), W1 = wseq(fq(0, 1), fq(2, 1), fq(4, 1), fq(6, 1), 3, 7);
    constexpr u64 W2 = wseq(fq(1, 0), fq(3, 0), fq(5, 0), 0, 4, 0), W3 = wseq(fq(1, 1), fq(3, 1), fq(5, 1), 1, 5, 0);
    u64 ws = P ? (j ? W3 : W2) : (j ? W1 : W0);
    auto wnext = [&]() {
      const unsigned k = ws & 63;
      ws >>= 8;
      return (((k + rank * 8) & 63) + (k < 8 ? 0 : level_add(t.tile))) * (64u * NN) + wcol;
    };
    if (NG == 4 && HOLD4) __builtin_amdgcn_s_sleep(HOLD4);
    load_w(x, wnext());
    load_w(x, wnext());
    SCHED();
    unsigned psrc, plvl;
    {
      const unsigned p = lane - 8, i = (p * 37u) >> 8, sr = p - i * 7u;
      const unsigned lo = sr < 3 ? sr : sr - 3, hi = sr < 3 ? sr + 4 : sr;
      const unsigned kl = i == lo ? 0 : i == hi ? 1 : 2 + i - (i > lo) - (i > hi);
      psrc = (rank + 1 + sr) & 7u;
      plvl = i;
      x.woff = (psrc * 8 + ((kl + level_add(t.tile)) & 7)) * (64u * NN) + wcol;
    }
    x.rsA = __builtin_amdgcn_make_buffer_rsrc((void*)(t.myws + OFF_DATA), (short)0, (int)(2 * SLOT_BYTES), 0x00020000);
    x.rsF = __builtin_amdgcn_make_buffer_rsrc((void*)t.myws, (short)0, (int)OFF_DBG, 0x00020000);
    x.tmo = (GAS unsigned*)(t.myws + OFF_TMO);
    x.va = (lane >> 2) * 64 + (((lane & 3) ^ (lane >> 5)) << 4);
    x.vflag = lane >= 8 ? (unsigned)OFF_FLAG + (t.tile * 8 + psrc) * 128 : (unsigned)OFF_TMO;
    x.lane = lane; x.arrived = 0; x.dead = 0;
    x.lda0 = lbase + LDS_A + (2 * (1 - P) + j) * 5120; x.lda = x.lda0 + 8 * 5120;
    x.scratch = lbase + CTRL_OFF + 128;
    x.seen = (volatile LAS u32x2*)(L + CTRL_OFF + 128 + lane * 16);
    if (wave == 4) *x.seen = u32x2{0, 0};
    if (P) TICK();
    asm volatile("s_waitcnt vmcnt(8) lgkmcnt(0)" : "+v"(c0) :: "memory");
    TICK();
    x.call = call = rfl(c0) + 1;
    x.need = (call - 1) * 8 + plvl;
    x.aoff = (call & 1) * (unsigned)SLOT_BYTES + (psrc * 8 + plvl) * 32768u + g0 * 1024;
    x.tp = rt();
    x.fix = j ? FM1 : FM0;
    x.want = P ? (j ? DM11 : DM10) : (j ? DM01 : DM00);
#pragma unroll 1
    for (unsigned i = P; i < 4; ++i) {
      load_w(x, wnext());
      TICK(); VMWAIT(8);
      if (wave == PW && i == PI) poll_async(x);
      TICK();
    }
    poll_use(x, *x.seen);
    if (P) {
      choose<false>(x);
#pragma unroll 1
      for (unsigned i = 0; i < 7; ++i) { item<NG, false, false, HA>(x); item<NG, false, false, HC>(x); }
    } else {
      choose<false>(x);
#pragma unroll 1
      for (unsigned i = 0; i < 7; ++i) { item<NG, false, true, HB>(x); item<NG, true, false, 0>(x); }
    }
    DBG(24 + j, rt() - t0);
    unsigned tw;
    asm volatile("global_load_dword %0, %1, off sc1" : "=v"(tw) : "v"(x.tmo) : "memory");
    TICK();
    asm volatile("s_waitcnt vmcnt(0)" : "+v"(tw) :: "memory");
    *(volatile LAS unsigned*)(ctrl + C_STATE + wave - 4) = (x.dead || tw == call) ? 2u : 1u;
    asm volatile("s_waitcnt lgkmcnt(0)" ::: "memory");
    TICK();
    if (P) TICK();
    DBG(20 + j, rt() - t0);
    __builtin_amdgcn_s_setprio(0);
    st = *(volatile LAS u32x4*)(ctrl + C_STATE); sbits = *(volatile LAS unsigned*)(ctrl + C_STATE + 4);
  } else {
    CW<NG> c;
    const __amdgpu_buffer_rsrc_t rsX = __builtin_amdgcn_make_buffer_rsrc((void*)t.ax, (short)0, NN * KS, 0x00020000);
    if (wave == 1 || wave == 2) {
      const unsigned u = (t.xt >> (8 * (wave - 1))) & 255;
      if ((u >> 5) != (unsigned)rank) touch(lbase + LDS_JUNK, (lane >> 2) * KS + (lane & 3) * 128, rsX, u * 16 * KS);
    }
    if (wave == 0) {
      touch(lbase + LDS_JUNK, lane * KS, rsX, (rank * KS + g0 * 16) * KS + t.sx * 128);
      if constexpr (NG == 5) touch(lbase + LDS_JUNK, lane < 16 ? lane * KS : OOR, rsX, (rank * KS + g0 * 16 + 64) * KS + t.sx * 128);
    }
    if constexpr (NG == 4) {
      const __amdgpu_buffer_rsrc_t rsW = __builtin_amdgcn_make_buffer_rsrc((void*)t.aw, (short)0, NN * NN, 0x00020000);
      auto tw = [&](unsigned u, unsigned strip) {
        const unsigned k = u >> 1, s0 = k + 3 * (u & 1), sr = umin(s0, s0 - 7u);
        const unsigned ks = u < 14 ? ((rank + 1 + sr) & 7) * 8 + (k > (sr < 3 ? sr : sr - 3)) : rank * 8 + u - 14;
        if (u < 22) touch(lbase + LDS_JUNK, lane * NN, rsW, ks * (64u * NN) + strip * 128);
      };
      const unsigned u0 = wave * 3 + t.tile - 4;
      if (t.nt != t.slow) { tw(2 * TS + u0, t.nt); tw(2 * TS + 12 + u0, t.nt); }
      if (t.help && wave >= 2) tw(2 * TSS + (t.help - 1) * 6 + u0 - 6, t.slow);
    }
    auto local_x = [&]() {
      const unsigned v0 = (lane >> 3) * KS + (((lane & 7) ^ (lane >> 4)) << 4), v1 = v0 ^ 64u;
      const unsigned so = (rank * KS + g0 * 16) * KS + wave * 128, ld = lbase + LDS_A + wave * 10240;
      dma_a<0 * 1024, 0 * 4096>(ld, v0, rsX, so); dma_a<1 * 1024, 1 * 4096>(ld, v1, rsX, so);
      dma_a<2 * 1024, 2 * 4096>(ld, v0, rsX, so); dma_a<3 * 1024, 3 * 4096>(ld, v1, rsX, so);
      dma_a<4 * 1024, 4 * 4096>(ld, v0, rsX, so); dma_a<5 * 1024, 5 * 4096>(ld, v1, rsX, so);
      dma_a<6 * 1024, 6 * 4096>(ld, v0, rsX, so); dma_a<7 * 1024, 7 * 4096>(ld, v1, rsX, so);
      if constexpr (NG == 5) { dma_a<8 * 1024, 8 * 4096>(ld, v0, rsX, so); dma_a<9 * 1024, 9 * 4096>(ld, v1, rsX, so); }
    };
    {
      const int g = lane >> 4, r = lane & 15;
      unsigned l0 = lbase + LDS_A + r * 128 + (((2 * g) ^ (r >> 1)) << 4), l1 = l0 ^ 16u;
      asm volatile("" : "+v"(l0), "+v"(l1));
      c.rl[0] = (LAS unsigned char*)(size_t)l0; c.rl[1] = (LAS unsigned char*)(size_t)l1;
    }
    {
      const int g = lane >> 4, q = (lane & 15) >> 1, p = lane & 1, r = lane & 15;
      const unsigned b0 = lbase + g * 4096 + q * 128 + (((wave << 1) ^ (q & 6) ^ (g & 1)) << 4) + p * 8;
      unsigned b1 = b0 ^ 16u, b2 = b0 + 65536, b3 = b1 + 65536;
      asm volatile("" : "+v"(b2), "+v"(b3));
      c.rb[0][0] = (LAS unsigned char*)(size_t)b0; c.rb[1][0] = (LAS unsigned char*)(size_t)b1;
      c.rb[0][1] = (LAS unsigned char*)(size_t)b2; c.rb[1][1] = (LAS unsigned char*)(size_t)b3;
      unsigned a0 = lbase + LDS_A + (g >> 1) * 5120 + r * 64 + (((2 * (g & 1)) ^ (r >> 3)) << 4), a1 = a0 ^ 16u;
      asm volatile("" : "+v"(a0), "+v"(a1));
      c.ra[0] = (LAS unsigned char*)(size_t)a0; c.ra[1] = (LAS unsigned char*)(size_t)a1;
    }
#pragma unroll
    for (int m = 0; m < NG; ++m)
#pragma unroll
      for (int r = 0; r < 4; ++r) { c.acc[m][0][r] = 0.f; c.acc[m][1][r] = 0.f; }
    if (wave == 0) {
      const float sc = *(const __attribute__((address_space(4))) float*)a.sx * *(const __attribute__((address_space(4))) float*)a.sw;
      *(volatile LAS unsigned*)(ctrl + C_STATE + 4) = __builtin_bit_cast(unsigned, sc);
    }
    prefill<NG, 7, 7 + 4 * XLT>(c);
    local_x();
    prefill<NG, 7 + 4 * XLT, 32>(c);
    asm volatile("s_waitcnt vmcnt(0) lgkmcnt(0)" ::: "memory");
    start<NG, 0>(c);
    steps<NG, 0, 31>(c);
    st = *(volatile LAS u32x4*)(ctrl + C_STATE); sbits = *(volatile LAS unsigned*)(ctrl + C_STATE + 4);
    c.wr = lbase + (lane & 15) * 512 + wave * 128 + (((lane >> 4) ^ (lane & 7)) << 4);
    SCHED();
    steps<NG, 31, 32>(c);
    DBG(wave, rt() - t0);
  }
  const __amdgpu_buffer_rsrc_t rsO = __builtin_amdgcn_make_buffer_rsrc((void*)t.aout, (short)0, KS * NN * 4, 0x00020000);
  const unsigned row = wave * 2 + (lane >> 5);
  const unsigned vo = row * (NN * 4) + (lane & 31) * 16;
  const unsigned so = (g0 * 16 * NN + t.nt * 128) * 4;
  const unsigned rd = lbase + row * 512 + (((lane & 31) ^ (row & 7)) << 4);
  const float scale = (st[0] & st[1] & st[2] & st[3]) == 1u ? __builtin_bit_cast(float, sbits) : __builtin_nanf("");
  const float scale2 = -1.44269504088896341f * scale;
#pragma unroll 1
  for (unsigned dry = WARMI && wave >= 4 ? OOR : 0u;; dry = 0) {
    if (!dry) {
      asm volatile("s_waitcnt lgkmcnt(0)" ::: "memory");
      TICK();
      if (wave < 4) DBG(16 + wave, rt() - t0);
    }
    f32x4 s[NG];
#pragma unroll
    for (int m = 0; m < NG; ++m) s[m] = *(LAS f32x4*)(size_t)(rd + m * 8192);
    SCHED();
#pragma unroll
    for (int m = 0; m < NG; ++m) {
      f32x4 e, z;
#pragma unroll
      for (int r = 0; r < 4; ++r) e[r] = __builtin_amdgcn_exp2f(__builtin_amdgcn_fmed3f(s[m][r] * scale2, -86.5617024533378f, 86.5617024533378f));
#pragma unroll
      for (int r = 0; r < 4; ++r) z[r] = (s[m][r] * scale) * __builtin_amdgcn_rcpf(1.f + e[r]);
      __builtin_amdgcn_raw_buffer_store_b128(__builtin_bit_cast(u32x4, z), rsO, vo, (so + m * 16 * (NN * 4)) | dry, 16);
      SCHED();
      asm volatile("s_nop 1");
      SCHED();
    }
    if (!dry) break;
  }
  if (wave == 4 && lane == 0) __hip_atomic_store(cnt, call, __ATOMIC_RELAXED, __HIP_MEMORY_SCOPE_AGENT);
}
}

__global__ void __launch_bounds__(512, 2) a2a_gemm_kernel(Args a) {
  extern __shared__ __attribute__((aligned(16))) unsigned char lds_raw[];
  const int wave = rfl(threadIdx.x >> 6), lane = threadIdx.x & 63;
  const int b = blockIdx.x;
  Tile t;
  t.ax = a.x; t.aw = a.w; t.aout = a.out; t.myws = a.myws; t.rank = a.rank; t.b = b;
  if (__builtin_expect(b < NSEND, 1)) {
    if (wave >= 4) return;
    Stream e = a.st[b * 4 + wave];
    asm volatile("" : "+s"(t.ax), "+s"(t.myws), "+s"(t.rank), "+s"(e.peer), "+s"(e.lv), "+s"(e.pos));
    if ((e.lv >> 24) == 0) return;
    GAS unsigned* cnt = (GAS unsigned*)(t.myws + OFF_CNT + (size_t)b * 128 + wave * 16);
    unsigned ts[10];
    const unsigned call = (e.lv >> 24) == 2 ? send_stream<2>(t.ax, e, t.rank, cnt, lane, ts) : send_stream<5>(t.ax, e, t.rank, cnt, lane, ts);
    DBG(26 + wave, rt() - t0);
    if (lane == 0) __hip_atomic_store(cnt, call, __ATOMIC_RELAXED, __HIP_MEMORY_SCOPE_AGENT);
    return;
  }
  t.xt = ((const unsigned*)a.xt)[b >> 1] >> (16 * (b & 1));
  asm volatile("" : "+s"(t.ax), "+s"(t.aw), "+s"(t.aout), "+s"(t.myws), "+s"(t.rank), "+s"(t.xt));
  const int cj = (b - NSEND) >> 3;
  t.tile = cj % NTILE; t.sx = cj / NTILE;
  t.nt = t.sx * 8 + ((b + 2 * t.sx) & 7);
  {
    const unsigned d = (3u - ((unsigned)(size_t)t.aw >> 7) - b) & 7u, ss = d >> 1;
    t.slow = (d & 1) ? -1 : ss * 8 + ((b + 2 * ss) & 7);
    t.help = (d & 1) ? 0 : (t.sx - ss) & 3;
  }
  if (t.tile < 4) tile_body<5>(a, t, (LAS unsigned char*)lds_raw, wave, lane);
  else tile_body<4>(a, t, (LAS unsigned char*)lds_raw, wave, lane);
}

extern "C" void kernel_launch(int rank, int nranks, ncclComm_t comm, hipStream_t stream,
                              void* const* d_in, const int* in_sizes, int n_in,
                              void* d_out, int out_size, const int* in_dtypes, int out_dtype,
                              void* const* d_out_all, void* const* d_ws_all, size_t ws_bytes) {
  if (nranks != NRANK || (unsigned)rank >= (unsigned)NRANK || ws_bytes < WS_NEED || n_in < 4 ||
      in_sizes[0] != NN * KS || in_sizes[1] != NN * NN || out_size != KS * NN) {
    fprintf(stderr, "a2a_gemm: unexpected problem shape\n");
    return;
  }
  static bool attr_set = false;
  if (!attr_set) {
    HIP_CHECK(hipFuncSetAttribute((const void*)a2a_gemm_kernel, hipFuncAttributeMaxDynamicSharedMemorySize, LDS_TOTAL));
    attr_set = true;
  }
  Args a;
  a.x = (const unsigned char*)d_in[0]; a.w = (const unsigned char*)d_in[1];
  a.sx = (const float*)d_in[2]; a.sw = (const float*)d_in[3];
  a.out = (float*)d_out;
  a.myws = (unsigned char*)d_ws_all[rank];
  a.rank = rank; a.pad[0] = a.pad[1] = a.pad[2] = 0;
  int nx[8] = {};
  for (int b = 0; b < NWG; ++b) a.xt[b] = (unsigned short)(rank * 32 * 257);
  for (int b = 0; b < NSEND; ++b)
    for (int w = 0; w < 4; ++w) {
      Stream& e = a.st[b * 4 + w];
      e.peer = nullptr; e.lv = 0; e.pos = 0;
      const bool five = b < 28 && w == 0;
      const int v = b < 28 ? b : b + 4 * w;
      if (!five && (v >= 42 || (b < 28 && w > 1))) continue;
      const int dd = five ? b % 7 : (v + 3) % 7, tile = five ? b / 7 : 4 + v / 14, half = five ? 0 : (v / 7) & 1;
      const int dest = (rank + 1 + dd) % NRANK, g0 = tile_g0(tile) + 2 * half, ng = five ? 5 : 2;
      e.peer = (unsigned char*)d_ws_all[dest];
      for (int i = 0; i < 8; ++i)
        e.lv |= (((levels(6 - dd) >> (3 * i)) + level_add(tile)) & 7u) << (3 * i);
      e.lv |= (unsigned)ng << 24;
      e.pos = (unsigned)(dest * KS + g0 * 16) | (unsigned)g0 << 12 | (unsigned)((tile * 8 + rank) * 32 + half) << 17;
      for (int g = 0; g < ng; ++g) {
        const int n = nx[b & 7]++, cb = NSEND + (n % 28) * 8 + (b & 7), sh = 8 * (n / 28);
        a.xt[cb] = (unsigned short)((a.xt[cb] & ~(255 << sh)) | (dest * 32 + g0 + g) << sh);
      }
    }
  a2a_gemm_kernel<<<NWG, 512, LDS_TOTAL, stream>>>(a);
  HIP_CHECK(hipGetLastError());
}
```

```cpp
#include <hip/hip_runtime.h>
#include <rccl/rccl.h>
#include <cstdio>

#define HIP_CHECK(expr) do { hipError_t _e = (expr); if (_e != hipSuccess) { \
    fprintf(stderr, "HIP error %s at %s:%d\n", hipGetErrorString(_e), __FILE__, __LINE__); \
    return; } } while (0)

#define LAS __attribute__((address_space(3)))
#define GAS __attribute__((address_space(1)))
typedef unsigned u32x2 __attribute__((ext_vector_type(2)));
typedef unsigned u32x4 __attribute__((ext_vector_type(4)));
typedef int i32x2 __attribute__((ext_vector_type(2)));
typedef int i32x8 __attribute__((ext_vector_type(8)));
typedef float f32x4 __attribute__((ext_vector_type(4)));
typedef unsigned long long u64;

#define SWIN (NG == 5 ? 7 : 2)

namespace {
constexpr int NRANK = 8;
constexpr int KS = 512;
constexpr int NN = 4096;
constexpr int NWG = 256;
constexpr int NSEND = 32;
constexpr int NTILE = 7;
constexpr size_t OFF_CNT = 0;
constexpr size_t OFF_TMO = 32768;
constexpr size_t OFF_FLAG = 65536;
constexpr size_t OFF_DATA = 1u << 20;
constexpr size_t SLOT_BYTES = 2u << 20;
constexpr size_t WS_NEED = OFF_DATA + 2 * SLOT_BYTES;
constexpr u64 DEADLINE_TICKS = 100000000ull;
constexpr int NB = 12;
constexpr int NA = 12;
constexpr int LDS_A = NB * 8192;
constexpr int CTRL_OFF = LDS_A + NA * 5120;
constexpr int LDS_JUNK = CTRL_OFF + 128 + 1024;
constexpr int LDS_TOTAL = LDS_JUNK + 256;
constexpr int C_STATE = 0;
constexpr int NF = 7;
constexpr bool fixed(int u) { return u >= 4 && (u & 3) == 3; }
constexpr int witem(int u) { return fixed(u) ? u / 4 - 1 : u + NF; }
constexpr int fq(int k, int j) { return 8 + 7 * k + (k + 3 * j) % 7; }
constexpr u64 fmask(int j) { u64 m = 0; for (int k = 0; k < NF; ++k) m |= 1ull << fq(k, j); return m; }
constexpr u64 dmask(int P, int j) {
  u64 m = 0;
  int u = 4, h = 0;
  for (int q = 8; q < 64; ++q) {
    if (((fmask(0) | fmask(1)) >> q) & 1) continue;
    if (((u + 7) & 1) == P && h == j) m |= 1ull << q;
    if (++h == 2) { h = 0; ++u; if (fixed(u)) ++u; }
  }
  return m;
}
constexpr unsigned levels(int s) {
  unsigned m = 0, nf = 0, nd = 2;
  for (int i = 0; i < 8; ++i) m |= ((((fmask(0) | fmask(1)) >> (8 + 7 * i + s)) & 1) ? nf++ : nd++) << (3 * i);
  return m;
}
__host__ __device__ constexpr unsigned level_add(int tile) { return tile < 4 ? 0 : 2 * tile - 6; }
constexpr int kfix(int q) { return q < 8 ? q : ((q - 8) % 7 + 1) * 8 + ((levels((q - 8) % 7) >> (3 * ((q - 8) / 7))) & 7); }
constexpr u64 wseq(int a, int b, int c, int d, int e, int f) {
  return (u64)kfix(a) | (u64)kfix(b) << 8 | (u64)kfix(c) << 16 | (u64)kfix(d) << 24 | (u64)kfix(e) << 32 | (u64)kfix(f) << 40;
}
constexpr u64 FM0 = fmask(0), FM1 = fmask(1), DM00 = dmask(0, 0), DM01 = dmask(0, 1), DM10 = dmask(1, 0), DM11 = dmask(1, 1);
constexpr unsigned OOR = 0x80000000u;
static_assert(NB == 12 && NA == 12, "ring schedule");

struct Stream {
  unsigned char* peer;
  unsigned lv;
  unsigned pos;
};
struct Args {
  const unsigned char* x; const unsigned char* w; const float* sx; const float* sw; float* out;
  unsigned char* myws;
  int rank; int pad[3];
  Stream st[NSEND * 4];
  unsigned short xt[NWG];
};

__device__ __forceinline__ u64 rt() { return __builtin_amdgcn_s_memrealtime(); }
__device__ __forceinline__ unsigned rfl(unsigned v) { return __builtin_amdgcn_readfirstlane(v); }
__device__ __forceinline__ unsigned umin(unsigned a, unsigned b) { return a < b ? a : b; }
#define SCHED() __builtin_amdgcn_sched_barrier(0)
#define TICK() do { asm volatile("" ::: "memory"); __builtin_amdgcn_s_barrier(); asm volatile("" ::: "memory"); } while (0)
#define VMWAIT(n) asm volatile("s_waitcnt vmcnt(%0)" :: "i"(n) : "memory")
__host__ __device__ constexpr int tile_g0(int t) { return t < 4 ? 5 * t : 4 * t + 4; }

template <int G, int E>
__device__ __forceinline__ void dma_w(unsigned lds, unsigned v0, unsigned v1, __amdgpu_buffer_rsrc_t rs, unsigned soff) {
  if constexpr (G + 1 < E) {
    unsigned t;
    asm volatile("s_xor_b32 m0, %1, %6\n\ts_xor_b32 %0, %5, %7\n\tbuffer_load_dwordx4 %2, %4, %0 offen lds\n\t"
                 "s_xor_b32 m0, %1, %8\n\ts_xor_b32 %0, %5, %9\n\tbuffer_load_dwordx4 %3, %4, %0 offen lds"
                 : "=&s"(t) : "s"(lds), "v"(G < 4 ? v0 : v1), "v"(G + 1 < 4 ? v0 : v1), "s"(rs), "s"(soff),
                   "i"(G << 10), "i"(G << 15), "i"((G + 1) << 10), "i"((G + 1) << 15) : "memory", "scc");
    dma_w<G + 2, E>(lds, v0, v1, rs, soff);
  } else if constexpr (G < E) {
    unsigned t;
    asm volatile("s_xor_b32 m0, %1, %5\n\ts_xor_b32 %0, %4, %6\n\tbuffer_load_dwordx4 %2, %3, %0 offen lds"
                 : "=&s"(t) : "s"(lds), "v"(G < 4 ? v0 : v1), "s"(rs), "s"(soff), "i"(G << 10), "i"(G << 15) : "memory", "scc");
  }
}
__device__ __forceinline__ void touch(unsigned lds, unsigned voff, __amdgpu_buffer_rsrc_t rs, unsigned soff) {
  asm volatile("s_mov_b32 m0, %0\n\ts_nop 3\n\tbuffer_load_dword %1, %2, %3 offen lds" :: "s"(lds), "v"(voff), "s"(rs), "s"(rfl(soff)) : "memory");
}
template <unsigned LO, unsigned SO>
__device__ __forceinline__ void dma_a(unsigned lds, unsigned voff, __amdgpu_buffer_rsrc_t rs, unsigned soff) {
  unsigned t;
  asm volatile("s_add_u32 m0, %1, %5\n\ts_add_u32 %0, %4, %6\n\tbuffer_load_dwordx4 %2, %3, %0 offen lds"
               : "=&s"(t) : "s"(lds), "v"(voff), "s"(rs), "s"(soff), "i"(LO), "i"(SO) : "memory", "scc");
}
template <int NG>
__device__ __forceinline__ void dma_piece(unsigned lds, unsigned voff, __amdgpu_buffer_rsrc_t rs, unsigned soff) {
  unsigned t;
  asm volatile("s_mov_b32 m0, %1\n\ts_mov_b32 %0, %4\n\t"
               "buffer_load_dwordx4 %2, %3, %0 offen sc0 sc1 lds\n\t"
               "buffer_load_dwordx4 %2, %3, %0 offen offset:1024 sc0 sc1 lds\n\t"
               "buffer_load_dwordx4 %2, %3, %0 offen offset:2048 sc0 sc1 lds\n\t"
               "buffer_load_dwordx4 %2, %3, %0 offen offset:3072 sc0 sc1 lds"
               : "=&s"(t) : "s"(lds), "v"(voff), "s"(rs), "s"(soff) : "memory");
  if constexpr (NG == 5)
    asm volatile("s_add_u32 m0, %1, 0x1000\n\ts_add_u32 %0, %4, 0x1000\n\tbuffer_load_dwordx4 %2, %3, %0 offen sc0 sc1 lds"
                 : "=&s"(t) : "s"(lds), "v"(voff), "s"(rs), "s"(soff) : "memory", "scc");
}
__device__ __forceinline__ i32x8 cat(u32x4 a, u32x4 b) {
  return __builtin_bit_cast(i32x8, __builtin_shufflevector(a, b, 0, 1, 2, 3, 4, 5, 6, 7));
}

constexpr int sendN(int NG, int w) {
  int n = 0;
  for (int t = w; t < w + SWIN; ++t) {
    if (t > w && t < 8 * NG) ++n;
    if (t + SWIN + 1 < 8 * NG) ++n;
    if (t - SWIN >= 0 && (t - SWIN + 1) % NG == 0) ++n;
  }
  return n;
}
template <int NG> struct SD {
  u32x4 v[8 * NG];
  __amdgpu_buffer_rsrc_t rsX;
  unsigned vx, sx, lv, call;
  unsigned char* dst;
  unsigned* flag;
};
template <int NG, int S> __device__ __forceinline__ void send_load(SD<NG>& d) {
  asm volatile("buffer_load_dwordx4 %0, %1, %2, %3 offen" : "=v"(d.v[S])
               : "v"(d.vx), "s"(d.rsX), "s"(d.sx + ((d.lv >> (3 * (S / NG))) & 7) * 64 + (S % NG) * 16 * KS) : "memory");
}
template <int NG, int S> __device__ __forceinline__ void send_first(SD<NG>& d) {
  send_load<NG, S>(d);
  if constexpr (S < SWIN) send_first<NG, S + 1>(d);
}
template <int NG, int S> __device__ __forceinline__ void send_iter(SD<NG>& d) {
  constexpr int ST = 8 * NG, W = S - SWIN;
  if constexpr (W >= 0) {
    constexpr int N = sendN(NG, W);
    VMWAIT(N);
  }
  if constexpr (S < ST) {
    if constexpr (W < 0) asm volatile("s_waitcnt vmcnt(%1)" : "+v"(d.v[S]) : "i"(SWIN + S) : "memory");
    else asm volatile("" : "+v"(d.v[S]));
    asm volatile("global_store_dwordx4 %0, %1, off sc0 sc1\n\ts_nop 1" :: "v"(d.dst + (S / NG) * 32768 + (S % NG) * 1024), "v"(d.v[S]) : "memory");
    if constexpr (S + SWIN + 1 < ST) send_load<NG, S + SWIN + 1>(d);
  }
  if constexpr (W >= 0 && (W + 1) % NG == 0) {
    const unsigned val = (d.call - 1) * 8 + W / NG + 1;
    asm volatile("s_mov_b64 exec, %2\n\tglobal_store_dword %0, %1, off sc0 sc1\n\ts_mov_b64 exec, -1"
                 :: "v"(d.flag), "v"(val), "i"(NG == 5 ? 3 : 1) : "memory");
  }
  if constexpr (S + 1 < ST + SWIN) send_iter<NG, S + 1>(d);
}
template <int NG>
__device__ __forceinline__ unsigned send_stream(const unsigned char* x, const Stream& e, int rank, GAS unsigned* cnt, int lane) {
  static_assert(2 * SWIN + 1 <= 8 * NG, "window");
  SD<NG> d;
  d.rsX = __builtin_amdgcn_make_buffer_rsrc((void*)x, (short)0, NN * KS, 0x00020000);
  d.vx = (lane >> 2) * KS + (lane & 3) * 16; d.sx = (e.pos & 4095) * KS; d.lv = e.lv;
  unsigned c0;
  asm volatile("global_load_dword %0, %1, off sc1" : "=v"(c0) : "v"(cnt) : "memory");
  send_first<NG, 0>(d);
  SCHED();
  d.flag = (unsigned*)(e.peer + OFF_FLAG) + (e.pos >> 17) + (lane & 1);
  unsigned char* dst0 = e.peer + OFF_DATA + (size_t)rank * 8 * 32768 + ((e.pos >> 12) & 31) * 1024 + lane * 16;
  asm volatile("s_waitcnt vmcnt(%1)" : "+v"(c0) : "i"(SWIN + 1) : "memory");
  d.call = c0 + 1;
  d.dst = dst0 + (d.call & 1) * SLOT_BYTES;
  send_iter<NG, 0>(d);
  return d.call;
}

template <int NG> struct CW {
  f32x4 acc[NG][2];
  i32x8 bf[32][2];
  i32x8 af[NG];
  LAS unsigned char* rb[2][2];
  LAS unsigned char* ra[2];
  unsigned wr;
  LAS unsigned char* rl[2];
};
template <int NG, int U, int CT> __device__ __forceinline__ void read_b(CW<NG>& c) {
  constexpr int S = (2 * witem(U)) % NB;
  LAS unsigned char* p = c.rb[CT][S >> 3] + (S & 7) * 8192;
  i32x2 a = __builtin_amdgcn_ds_read_tr8_b64_v2i32((LAS i32x2*)(p));
  i32x2 b = __builtin_amdgcn_ds_read_tr8_b64_v2i32((LAS i32x2*)(p + 1024));
  i32x2 e = __builtin_amdgcn_ds_read_tr8_b64_v2i32((LAS i32x2*)(p + 2048));
  i32x2 d = __builtin_amdgcn_ds_read_tr8_b64_v2i32((LAS i32x2*)(p + 3072));
  i32x8 r; r[0] = a[0]; r[1] = a[1]; r[2] = b[0]; r[3] = b[1]; r[4] = e[0]; r[5] = e[1]; r[6] = d[0]; r[7] = d[1];
  c.bf[U][CT] = r;
}
template <int NG, int U, int M> __device__ __forceinline__ void read_a(CW<NG>& c) {
  constexpr int O = ((2 * U) % NA) * 5120 + M * (U < 4 ? 2048 : 1024);
  if constexpr (U < 4) c.af[M] = cat(*(LAS u32x4*)(c.rl[0] + O), *(LAS u32x4*)(c.rl[1] + O));
  else c.af[M] = cat(*(LAS u32x4*)(c.ra[0] + O), *(LAS u32x4*)(c.ra[1] + O));
}
template <int NG, int U, int M> __device__ __forceinline__ void read_a_all(CW<NG>& c) {
  read_a<NG, U, M>(c);
  if constexpr (M + 1 < NG) read_a_all<NG, U, M + 1>(c);
}
template <int NG, int U, int E> __device__ __forceinline__ void prefill(CW<NG>& c) {
  asm volatile("s_waitcnt lgkmcnt(0)" ::: "memory");
  TICK();
  read_b<NG, U, 0>(c); read_b<NG, U, 1>(c);
  SCHED();
  if constexpr (U + 4 < E) prefill<NG, U + 4, E>(c);
}
template <int NG, int M> __device__ __forceinline__ void put(CW<NG>& c) {
  *(LAS f32x4*)(size_t)(c.wr + M * 8192) = c.acc[M][0];
  *(LAS f32x4*)(size_t)((c.wr ^ 64u) + M * 8192) = c.acc[M][1];
}
template <int NG, int U, int M> __device__ __forceinline__ void row_tile(CW<NG>& c) {
  constexpr bool RA = U + 1 < 32, RB = RA && !fixed(U + 1);
  c.acc[M][0] = __builtin_amdgcn_mfma_scale_f32_16x16x128_f8f6f4(c.bf[U][0], c.af[M], c.acc[M][0], 1, 1, 0, 0, 0, 0);
  c.acc[M][1] = __builtin_amdgcn_mfma_scale_f32_16x16x128_f8f6f4(c.bf[U][1], c.af[M], c.acc[M][1], 1, 1, 0, 0, 0, 0);
  SCHED();
  if constexpr (M == 0 && RA) TICK();
  if constexpr (RA) read_a<NG, U + 1, M>(c);
  if constexpr (RB && M < 2) read_b<NG, U + 1, M>(c);
  if constexpr (U == 31 && M > 0) put<NG, M - 1>(c);
  if constexpr (U == 31 && M == NG - 1) put<NG, M>(c);
  SCHED();
  if constexpr (M + 1 < NG) row_tile<NG, U, M + 1>(c);
}
template <int NG, int U, int E> __device__ __forceinline__ void steps(CW<NG>& c) {
  row_tile<NG, U, 0>(c);
  if constexpr (U + 1 < E) steps<NG, U + 1, E>(c);
}
template <int NG, int U> __device__ __forceinline__ void start(CW<NG>& c) {
  TICK();
  read_b<NG, U, 0>(c); read_b<NG, U, 1>(c);
  SCHED();
  read_a_all<NG, U, 0>(c);
  SCHED();
}

struct LD {
  __amdgpu_buffer_rsrc_t rsA, rsF, rsW;
  GAS unsigned* tmo;
  volatile LAS u32x2* seen;
  unsigned va, aoff, lda0, lda, vw0, vw1, woff, ldw0, ldw, scratch, vflag, need, call;
  u64 arrived, tp;
  u64 want, fix;
  unsigned dead;
  unsigned sox, sow;
  bool poll;
  int lane;
};
__device__ __forceinline__ void poll_use(LD& x, u32x2 r) {
  x.arrived |= __ballot((int)(r[0] - x.need) > 0 && (int)(r[1] - x.need) > 0) & ~0xffull;
}
__device__ __forceinline__ void poll_async(LD& x) {
  asm volatile("s_mov_b32 m0, %0\n\ts_nop 0\n\tbuffer_load_dwordx4 %1, %2, 0 offen sc0 sc1 lds"
               :: "s"(x.scratch), "v"(x.vflag), "s"(x.rsF) : "memory");
}
template <bool FIX> __device__ __forceinline__ void choose(LD& x) {
  if (FIX) {
    const unsigned q = __builtin_ctzll(x.fix | (1ull << 63));
    const bool ok = (x.arrived >> q) & 1;
    const unsigned so = __builtin_amdgcn_readlane(x.aoff, q);
    x.sox = ok ? so : OOR;
    x.fix &= ~(x.arrived & (1ull << q));
    x.poll = !ok;
  } else {
    const u64 c = x.arrived & x.want;
    const unsigned q = __builtin_ctzll(c | (1ull << 63));
    const unsigned so = __builtin_amdgcn_readlane(x.aoff, q);
    x.sow = __builtin_amdgcn_readlane(x.woff, q);
    x.sox = c ? so : OOR;
    x.poll = __builtin_popcountll(c) <= 2 && x.arrived != ~0xffull;
    x.want &= ~(c & (1ull << q));
  }
}
template <bool FIX> __device__ __forceinline__ void choose_wait(LD& x) {
  if (__builtin_expect(x.sox == OOR, 0)) {
    poll_use(x, *x.seen);
    choose<FIX>(x);
    while (x.sox == OOR) {
      u32x2 r;
      asm volatile("buffer_load_dwordx2 %0, %1, %2, 0 offen sc0 sc1\n\ts_waitcnt vmcnt(0)" : "=&v"(r) : "v"(x.vflag), "s"(x.rsF) : "memory");
      *x.seen = r;
      poll_use(x, r);
      if (__ballot(x.lane < 8 && r[0] == x.call) != 0 || rt() - x.tp > DEADLINE_TICKS) {
        __hip_atomic_store(x.tmo, x.call, __ATOMIC_RELAXED, __HIP_MEMORY_SCOPE_AGENT);
        x.dead = OOR; x.arrived = ~0xffull;
      }
      choose<FIX>(x);
    }
  }
}
__device__ __forceinline__ void next_w(LD& x) { x.ldw = x.ldw == x.ldw0 + 8 * 8192 ? x.ldw0 : x.ldw + 4 * 8192; }
__device__ __forceinline__ void load_w(LD& x, unsigned so) {
  dma_w<0, 8>(x.ldw, x.vw0, x.vw1, x.rsW, so);
  next_w(x);
}
template <int NG, bool FIX, bool NEXTFIX> __device__ __forceinline__ void item(LD& x) {
  constexpr int HEAD = 6;
  choose_wait<FIX>(x);
  dma_piece<NG>(x.lda, x.va, x.rsA, x.sox | x.dead);
  if (!FIX) dma_w<0, HEAD>(x.ldw, x.vw0, x.vw1, x.rsW, x.sow);
  const u32x2 r = *x.seen;
  VMWAIT(NG + (FIX ? 0 : HEAD));
  TICK();
  if (!FIX) dma_w<HEAD, 8>(x.ldw, x.vw0, x.vw1, x.rsW, x.sow);
  SCHED();
  x.lda = x.lda == x.lda0 + 8 * 5120 ? x.lda0 : x.lda + 4 * 5120;
  next_w(x);
  poll_use(x, r);
  choose<NEXTFIX>(x);
  SCHED();
  if (__builtin_expect(x.poll, 0)) poll_async(x);
  TICK();
}

struct Tile { const unsigned char* ax; const unsigned char* aw; float* aout; unsigned char* myws; int rank, b, tile, nt, sx, slow, help; unsigned xt; };

template <int NG>
__device__ __forceinline__ void tile_body(const Args& a, const Tile& t, LAS unsigned char* L, const int wave, const int lane) {
  const unsigned lbase = (unsigned)(size_t)L;
  LAS unsigned* ctrl = (LAS unsigned*)(L + CTRL_OFF);
  const int rank = t.rank, g0 = tile_g0(t.tile);
  GAS unsigned* cnt = (GAS unsigned*)(t.myws + OFF_CNT + (size_t)t.b * 128);
  unsigned call = 0;
  if (wave >= 4) {
    __builtin_amdgcn_s_setprio(2);
    const unsigned j = wave & 1;
    const unsigned P = (wave >> 1) & 1;
    unsigned c0;
    asm volatile("global_load_dword %0, %1, off sc1" : "=v"(c0) : "v"(cnt) : "memory");
    LD x;
    const unsigned rot = (t.tile + 2 * wave) & 7;
    x.rsW = __builtin_amdgcn_make_buffer_rsrc((void*)t.aw, (short)0, NN * NN, 0x00020000);
    const unsigned wcol = t.nt * 128 + (rot << 15);
    x.vw0 = (lane >> 3) * NN + (((lane & 7) ^ ((lane >> 3) & 6) ^ (rot >> 2)) << 4); x.vw1 = x.vw0 ^ 16u;
    x.ldw = x.ldw0 = (lbase + (2 * P + j) * 8192) | (rot << 10);
    constexpr u64 W0 = wseq(fq(0, 0), fq(2, 0), fq(4, 0), fq(6, 0), 2, 6), W1 = wseq(fq(0, 1), fq(2, 1), fq(4, 1), fq(6, 1), 3, 7);
    constexpr u64 W2 = wseq(fq(1, 0), fq(3, 0), fq(5, 0), 0, 4, 0), W3 = wseq(fq(1, 1), fq(3, 1), fq(5, 1), 1, 5, 0);
    u64 ws = P ? (j ? W3 : W2) : (j ? W1 : W0);
    auto wnext = [&]() {
      const unsigned k = ws & 63;
      ws >>= 8;
      return (((k + rank * 8) & 63) + (k < 8 ? 0 : level_add(t.tile))) * (64u * NN) + wcol;
    };
    load_w(x, wnext());
    load_w(x, wnext());
    SCHED();
    unsigned psrc, plvl;
    {
      const unsigned p = lane - 8, i = (p * 37u) >> 8, sr = p - i * 7u;
      const unsigned lo = sr < 3 ? sr : sr - 3, hi = sr < 3 ? sr + 4 : sr;
      const unsigned kl = i == lo ? 0 : i == hi ? 1 : 2 + i - (i > lo) - (i > hi);
      psrc = (rank + 1 + sr) & 7u;
      plvl = i;
      x.woff = (psrc * 8 + ((kl + level_add(t.tile)) & 7)) * (64u * NN) + wcol;
    }
    x.rsA = __builtin_amdgcn_make_buffer_rsrc((void*)(t.myws + OFF_DATA), (short)0, (int)(2 * SLOT_BYTES), 0x00020000);
    x.rsF = __builtin_amdgcn_make_buffer_rsrc((void*)t.myws, (short)0, (int)OFF_DATA, 0x00020000);
    x.tmo = (GAS unsigned*)(t.myws + OFF_TMO);
    x.va = (lane >> 2) * 64 + (((lane & 3) ^ (lane >> 5)) << 4);
    x.vflag = lane >= 8 ? (unsigned)OFF_FLAG + (t.tile * 8 + psrc) * 128 : (unsigned)OFF_TMO;
    x.lane = lane; x.arrived = 0; x.dead = 0;
    x.lda0 = lbase + LDS_A + (2 * (1 - P) + j) * 5120; x.lda = x.lda0 + 8 * 5120;
    x.scratch = lbase + CTRL_OFF + 128;
    x.seen = (volatile LAS u32x2*)(L + CTRL_OFF + 128 + lane * 16);
    if (wave == 4) *x.seen = u32x2{0, 0};
    if (P) TICK();
    asm volatile("s_waitcnt vmcnt(8) lgkmcnt(0)" : "+v"(c0) :: "memory");
    TICK();
    x.call = call = rfl(c0) + 1;
    x.need = (call - 1) * 8 + plvl;
    x.aoff = (call & 1) * (unsigned)SLOT_BYTES + (psrc * 8 + plvl) * 32768u + g0 * 1024;
    x.tp = rt();
    x.fix = j ? FM1 : FM0;
    x.want = P ? (j ? DM11 : DM10) : (j ? DM01 : DM00);
#pragma unroll 1
    for (unsigned i = P; i < 4; ++i) {
      load_w(x, wnext());
      TICK(); VMWAIT(8);
      if (wave == 4 && i == 1) poll_async(x);
      TICK();
    }
    TICK();
    poll_use(x, *x.seen);
    if (P) {
      choose<false>(x);
#pragma unroll 1
      for (unsigned i = 0; i < 14; ++i) item<NG, false, false>(x);
    } else {
      choose<false>(x);
#pragma unroll 1
      for (unsigned i = 0; i < 7; ++i) { item<NG, false, true>(x); item<NG, true, false>(x); }
    }
    unsigned tw;
    asm volatile("global_load_dword %0, %1, off sc1" : "=v"(tw) : "v"(x.tmo) : "memory");
    VMWAIT(1);
    TICK();
    if (P) TICK();
    asm volatile("s_waitcnt vmcnt(0)" : "+v"(tw) :: "memory");
    *(volatile LAS unsigned*)(ctrl + C_STATE + wave - 4) = (x.dead || tw == call) ? 2u : 1u;
    __builtin_amdgcn_s_setprio(0);
  } else {
    CW<NG> c;
    const __amdgpu_buffer_rsrc_t rsX = __builtin_amdgcn_make_buffer_rsrc((void*)t.ax, (short)0, NN * KS, 0x00020000);
    if (wave == 1 || wave == 2) {
      const unsigned u = (t.xt >> (8 * (wave - 1))) & 255;
      if ((u >> 5) != (unsigned)rank) touch(lbase + LDS_JUNK, (lane >> 2) * KS + (lane & 3) * 128, rsX, u * 16 * KS);
    }
    if (wave == 0) {
      touch(lbase + LDS_JUNK, lane * KS, rsX, (rank * KS + g0 * 16) * KS + t.sx * 128);
      if constexpr (NG == 5) touch(lbase + LDS_JUNK, lane < 16 ? lane * KS : OOR, rsX, (rank * KS + g0 * 16 + 64) * KS + t.sx * 128);
    }
    if constexpr (NG == 4) {
      const __amdgpu_buffer_rsrc_t rsW = __builtin_amdgcn_make_buffer_rsrc((void*)t.aw, (short)0, NN * NN, 0x00020000);
      auto tw = [&](unsigned u, unsigned strip) {
        const unsigned k = u >> 1, s0 = k + 3 * (u & 1), sr = umin(s0, s0 - 7u);
        const unsigned ks = u < 14 ? ((rank + 1 + sr) & 7) * 8 + (k > (sr < 3 ? sr : sr - 3)) : rank * 8 + u - 14;
        if (u < 22) touch(lbase + LDS_JUNK, lane * NN, rsW, ks * (64u * NN) + strip * 128);
      };
      const unsigned u0 = wave * 3 + t.tile - 4;
      if (t.nt != t.slow) tw(14 + u0, t.nt);
      if (t.help && wave >= 2) tw(8 + (t.help - 1) * 6 + u0 - 6, t.slow);
    }
    auto local_x = [&]() {
      const unsigned v0 = (lane >> 3) * KS + (((lane & 7) ^ (lane >> 4)) << 4), v1 = v0 ^ 64u;
      const unsigned so = (rank * KS + g0 * 16) * KS + wave * 128, ld = lbase + LDS_A + wave * 10240;
      dma_a<0 * 1024, 0 * 4096>(ld, v0, rsX, so); dma_a<1 * 1024, 1 * 4096>(ld, v1, rsX, so);
      dma_a<2 * 1024, 2 * 4096>(ld, v0, rsX, so); dma_a<3 * 1024, 3 * 4096>(ld, v1, rsX, so);
      dma_a<4 * 1024, 4 * 4096>(ld, v0, rsX, so); dma_a<5 * 1024, 5 * 4096>(ld, v1, rsX, so);
      dma_a<6 * 1024, 6 * 4096>(ld, v0, rsX, so); dma_a<7 * 1024, 7 * 4096>(ld, v1, rsX, so);
      if constexpr (NG == 5) { dma_a<8 * 1024, 8 * 4096>(ld, v0, rsX, so); dma_a<9 * 1024, 9 * 4096>(ld, v1, rsX, so); }
    };
    {
      const int g = lane >> 4, r = lane & 15;
      unsigned l0 = lbase + LDS_A + r * 128 + (((2 * g) ^ (r >> 1)) << 4), l1 = l0 ^ 16u;
      asm volatile("" : "+v"(l0), "+v"(l1));
      c.rl[0] = (LAS unsigned char*)(size_t)l0; c.rl[1] = (LAS unsigned char*)(size_t)l1;
    }
    {
      const int g = lane >> 4, q = (lane & 15) >> 1, p = lane & 1, r = lane & 15;
      const unsigned b0 = lbase + g * 4096 + q * 128 + (((wave << 1) ^ (q & 6) ^ (g & 1)) << 4) + p * 8;
      unsigned b1 = b0 ^ 16u, b2 = b0 + 65536, b3 = b1 + 65536;
      asm volatile("" : "+v"(b2), "+v"(b3));
      c.rb[0][0] = (LAS unsigned char*)(size_t)b0; c.rb[1][0] = (LAS unsigned char*)(size_t)b1;
      c.rb[0][1] = (LAS unsigned char*)(size_t)b2; c.rb[1][1] = (LAS unsigned char*)(size_t)b3;
      unsigned a0 = lbase + LDS_A + (g >> 1) * 5120 + r * 64 + (((2 * (g & 1)) ^ (r >> 3)) << 4), a1 = a0 ^ 16u;
      asm volatile("" : "+v"(a0), "+v"(a1));
      c.ra[0] = (LAS unsigned char*)(size_t)a0; c.ra[1] = (LAS unsigned char*)(size_t)a1;
    }
#pragma unroll
    for (int m = 0; m < NG; ++m)
#pragma unroll
      for (int r = 0; r < 4; ++r) { c.acc[m][0][r] = 0.f; c.acc[m][1][r] = 0.f; }
    if (wave == 0) {
      const float sc = *(const __attribute__((address_space(4))) float*)a.sx * *(const __attribute__((address_space(4))) float*)a.sw;
      *(volatile LAS unsigned*)(ctrl + C_STATE + 4) = __builtin_bit_cast(unsigned, sc);
    }
    prefill<NG, 7, 19>(c);
    local_x();
    prefill<NG, 19, 32>(c);
    asm volatile("s_waitcnt vmcnt(0) lgkmcnt(0)" ::: "memory");
    start<NG, 0>(c);
    steps<NG, 0, 31>(c);
    c.wr = lbase + (lane & 15) * 512 + wave * 128 + (((lane >> 4) ^ (lane & 7)) << 4);
    SCHED();
    steps<NG, 31, 32>(c);
  }
  const __amdgpu_buffer_rsrc_t rsO = __builtin_amdgcn_make_buffer_rsrc((void*)t.aout, (short)0, KS * NN * 4, 0x00020000);
  const unsigned row = wave * 2 + (lane >> 5);
  unsigned vo = row * (NN * 4) + (lane & 31) * 16;
  const unsigned so = (g0 * 16 * NN + t.nt * 128) * 4;
  unsigned rd = lbase + row * 512 + (((lane & 31) ^ (row & 7)) << 4);
  asm volatile("" : "+v"(rd), "+v"(vo));
  asm volatile("s_waitcnt lgkmcnt(0)" ::: "memory");
  TICK();
  const u32x4 st = *(LAS u32x4*)(ctrl + C_STATE);
  const unsigned sbits = ctrl[C_STATE + 4];
  SCHED();
  f32x4 s[NG];
#pragma unroll
  for (int m = 0; m < NG; ++m) { s[m] = *(LAS f32x4*)(size_t)(rd + m * 8192); SCHED(); }
  const float scale = (st[0] & st[1] & st[2] & st[3]) == 1u ? __builtin_bit_cast(float, sbits) : __builtin_nanf("");
  const float scale2 = -1.44269504088896341f * scale;
#pragma unroll
  for (int m = 0; m < NG; ++m) {
    f32x4 e = s[m] * scale2;
    asm volatile("" : "+v"(e));
#pragma unroll
    for (int r = 0; r < 4; ++r) e[r] = __builtin_amdgcn_exp2f(__builtin_amdgcn_fmed3f(e[r], -86.5617024533378f, 86.5617024533378f));
    e += 1.f;
    asm volatile("" : "+v"(e));
#pragma unroll
    for (int r = 0; r < 4; ++r) e[r] = __builtin_amdgcn_rcpf(e[r]);
    const f32x4 z = (s[m] * scale) * e;
    __builtin_amdgcn_raw_buffer_store_b128(__builtin_bit_cast(u32x4, z), rsO, vo, so + m * 16 * (NN * 4), 16);
    SCHED();
    asm volatile("s_nop 1");
    SCHED();
  }
  if (wave == 4 && lane == 0) __hip_atomic_store(cnt, call, __ATOMIC_RELAXED, __HIP_MEMORY_SCOPE_AGENT);
}
}

__global__ void __launch_bounds__(512, 2) a2a_gemm_kernel(Args a) {
  extern __shared__ __attribute__((aligned(16))) unsigned char lds_raw[];
  const int wave = rfl(threadIdx.x >> 6), lane = threadIdx.x & 63;
  const int b = blockIdx.x;
  Tile t;
  t.ax = a.x; t.aw = a.w; t.aout = a.out; t.myws = a.myws; t.rank = a.rank; t.b = b;
  if (__builtin_expect(b < NSEND, 1)) {
    if (wave >= 4) return;
    Stream e = a.st[b * 4 + wave];
    asm volatile("" : "+s"(t.ax), "+s"(t.myws), "+s"(t.rank), "+s"(e.peer), "+s"(e.lv), "+s"(e.pos));
    if ((e.lv >> 24) == 0) return;
    GAS unsigned* cnt = (GAS unsigned*)(t.myws + OFF_CNT + (size_t)b * 128 + wave * 16);
    const unsigned call = (e.lv >> 24) == 2 ? send_stream<2>(t.ax, e, t.rank, cnt, lane) : send_stream<5>(t.ax, e, t.rank, cnt, lane);
    if (lane == 0) __hip_atomic_store(cnt, call, __ATOMIC_RELAXED, __HIP_MEMORY_SCOPE_AGENT);
    return;
  }
  t.xt = ((const unsigned*)a.xt)[b >> 1] >> (16 * (b & 1));
  asm volatile("" : "+s"(t.ax), "+s"(t.aw), "+s"(t.aout), "+s"(t.myws), "+s"(t.rank), "+s"(t.xt));
  const int cj = (b - NSEND) >> 3;
  t.tile = cj % NTILE; t.sx = cj / NTILE;
  t.nt = t.sx * 8 + ((b + 2 * t.sx) & 7);
  {
    const unsigned d = (3u - ((unsigned)(size_t)t.aw >> 7) - b) & 7u, ss = d >> 1;
    t.slow = (d & 1) ? -1 : ss * 8 + ((b + 2 * ss) & 7);
    t.help = (d & 1) ? 0 : (t.sx - ss) & 3;
  }
  if (t.tile < 4) tile_body<5>(a, t, (LAS unsigned char*)lds_raw, wave, lane);
  else tile_body<4>(a, t, (LAS unsigned char*)lds_raw, wave, lane);
}

extern "C" void kernel_launch(int rank, int nranks, ncclComm_t comm, hipStream_t stream,
                              void* const* d_in, const int* in_sizes, int n_in,
                              void* d_out, int out_size, const int* in_dtypes, int out_dtype,
                              void* const* d_out_all, void* const* d_ws_all, size_t ws_bytes) {
  if (nranks != NRANK || (unsigned)rank >= (unsigned)NRANK || ws_bytes < WS_NEED || n_in < 4 ||
      in_sizes[0] != NN * KS || in_sizes[1] != NN * NN || out_size != KS * NN) {
    fprintf(stderr, "a2a_gemm: unexpected problem shape\n");
    return;
  }
  static bool attr_set = false;
  if (!attr_set) {
    HIP_CHECK(hipFuncSetAttribute((const void*)a2a_gemm_kernel, hipFuncAttributeMaxDynamicSharedMemorySize, LDS_TOTAL));
    attr_set = true;
  }
  Args a;
  a.x = (const unsigned char*)d_in[0]; a.w = (const unsigned char*)d_in[1];
  a.sx = (const float*)d_in[2]; a.sw = (const float*)d_in[3];
  a.out = (float*)d_out;
  a.myws = (unsigned char*)d_ws_all[rank];
  a.rank = rank; a.pad[0] = a.pad[1] = a.pad[2] = 0;
  int nx[8] = {};
  for (int b = 0; b < NWG; ++b) a.xt[b] = (unsigned short)(rank * 32 * 257);
  for (int b = 0; b < NSEND; ++b)
    for (int w = 0; w < 4; ++w) {
      Stream& e = a.st[b * 4 + w];
      e.peer = nullptr; e.lv = 0; e.pos = 0;
      const bool five = b < 28 && w == 0;
      const int v = b < 28 ? b : b + 4 * w;
      if (!five && (v >= 42 || (b < 28 && w > 1))) continue;
      const int dd = five ? b % 7 : (v + 3) % 7, tile = five ? b / 7 : 4 + v / 14, half = five ? 0 : (v / 7) & 1;
      const int dest = (rank + 1 + dd) % NRANK, g0 = tile_g0(tile) + 2 * half, ng = five ? 5 : 2;
      e.peer = (unsigned char*)d_ws_all[dest];
      for (int i = 0; i < 8; ++i)
        e.lv |= (((levels(6 - dd) >> (3 * i)) + level_add(tile)) & 7u) << (3 * i);
      e.lv |= (unsigned)ng << 24;
      e.pos = (unsigned)(dest * KS + g0 * 16) | (unsigned)g0 << 12 | (unsigned)((tile * 8 + rank) * 32 + half) << 17;
      for (int g = 0; g < ng; ++g) {
        const int n = nx[b & 7]++, cb = NSEND + (n % 28) * 8 + (b & 7), sh = 8 * (n / 28);
        a.xt[cb] = (unsigned short)((a.xt[cb] & ~(255 << sh)) | (dest * 32 + g0 + g) << sh);
      }
    }
  a2a_gemm_kernel<<<NWG, 512, LDS_TOTAL, stream>>>(a);
  HIP_CHECK(hipGetLastError());
}
```

```cpp
#include <hip/hip_runtime.h>
#include <rccl/rccl.h>
#include <cstdio>

#define HIP_CHECK(expr) do { hipError_t _e = (expr); if (_e != hipSuccess) { \
    fprintf(stderr, "HIP error %s at %s:%d\n", hipGetErrorString(_e), __FILE__, __LINE__); \
    return; } } while (0)

#define LAS __attribute__((address_space(3)))
#define GAS __attribute__((address_space(1)))
typedef unsigned u32x2 __attribute__((ext_vector_type(2)));
typedef unsigned u32x4 __attribute__((ext_vector_type(4)));
typedef int i32x2 __attribute__((ext_vector_type(2)));
typedef int i32x8 __attribute__((ext_vector_type(8)));
typedef float f32x4 __attribute__((ext_vector_type(4)));
typedef unsigned long long u64;

#define SWIN (NG == 5 ? 7 : 2)

namespace {
constexpr int NRANK = 8;
constexpr int KS = 512;
constexpr int NN = 4096;
constexpr int NWG = 256;
constexpr int NSEND = 32;
constexpr int NTILE = 7;
constexpr size_t OFF_CNT = 0;
constexpr size_t OFF_TMO = 32768;
constexpr size_t OFF_FLAG = 65536;
constexpr size_t OFF_DATA = 1u << 20;
constexpr size_t SLOT_BYTES = 2u << 20;
constexpr size_t WS_NEED = OFF_DATA + 2 * SLOT_BYTES;
constexpr u64 DEADLINE_TICKS = 100000000ull;
constexpr int NB = 12;
constexpr int NA = 12;
constexpr int LDS_A = NB * 8192;
constexpr int CTRL_OFF = LDS_A + NA * 5120;
constexpr int LDS_JUNK = CTRL_OFF + 128 + 1024;
constexpr int LDS_TOTAL = LDS_JUNK + 256;
constexpr int C_STATE = 0;
constexpr int NF = 7;
constexpr bool fixed(int u) { return u >= 4 && (u & 3) == 3; }
constexpr int witem(int u) { return fixed(u) ? u / 4 - 1 : u + NF; }
constexpr int fq(int k, int j) { return 8 + 7 * k + (k + 3 * j) % 7; }
constexpr u64 fmask(int j) { u64 m = 0; for (int k = 0; k < NF; ++k) m |= 1ull << fq(k, j); return m; }
constexpr u64 dmask(int P, int j) {
  u64 m = 0;
  int u = 4, h = 0;
  for (int q = 8; q < 64; ++q) {
    if (((fmask(0) | fmask(1)) >> q) & 1) continue;
    if (((u + 7) & 1) == P && h == j) m |= 1ull << q;
    if (++h == 2) { h = 0; ++u; if (fixed(u)) ++u; }
  }
  return m;
}
constexpr unsigned levels(int s) {
  unsigned m = 0, nf = 0, nd = 2;
  for (int i = 0; i < 8; ++i) m |= ((((fmask(0) | fmask(1)) >> (8 + 7 * i + s)) & 1) ? nf++ : nd++) << (3 * i);
  return m;
}
__host__ __device__ constexpr unsigned level_add(int tile) { return tile < 4 ? 0 : 2 * tile - 6; }
constexpr int kfix(int q) { return q < 8 ? q : ((q - 8) % 7 + 1) * 8 + ((levels((q - 8) % 7) >> (3 * ((q - 8) / 7))) & 7); }
constexpr u64 wseq(int a, int b, int c, int d, int e, int f) {
  return (u64)kfix(a) | (u64)kfix(b) << 8 | (u64)kfix(c) << 16 | (u64)kfix(d) << 24 | (u64)kfix(e) << 32 | (u64)kfix(f) << 40;
}
constexpr u64 FM0 = fmask(0), FM1 = fmask(1), DM00 = dmask(0, 0), DM01 = dmask(0, 1), DM10 = dmask(1, 0), DM11 = dmask(1, 1);
constexpr unsigned OOR = 0x80000000u;
static_assert(NB == 12 && NA == 12, "ring schedule");

struct Stream {
  unsigned char* peer;
  unsigned lv;
  unsigned pos;
};
struct Args {
  const unsigned char* x; const unsigned char* w; const float* sx; const float* sw; float* out;
  unsigned char* myws;
  int rank; int pad[3];
  Stream st[NSEND * 4];
  unsigned short xt[NWG];
};

__device__ __forceinline__ u64 rt() { return __builtin_amdgcn_s_memrealtime(); }
__device__ __forceinline__ unsigned rfl(unsigned v) { return __builtin_amdgcn_readfirstlane(v); }
__device__ __forceinline__ unsigned umin(unsigned a, unsigned b) { return a < b ? a : b; }
#define SCHED() __builtin_amdgcn_sched_barrier(0)
#define TICK() do { asm volatile("" ::: "memory"); __builtin_amdgcn_s_barrier(); asm volatile("" ::: "memory"); } while (0)
#define VMWAIT(n) asm volatile("s_waitcnt vmcnt(%0)" :: "i"(n) : "memory")
__host__ __device__ constexpr int tile_g0(int t) { return t < 4 ? 5 * t : 4 * t + 4; }

template <int G, int E>
__device__ __forceinline__ void dma_w(unsigned lds, unsigned v0, unsigned v1, __amdgpu_buffer_rsrc_t rs, unsigned soff) {
  if constexpr (G + 1 < E) {
    unsigned t;
    asm volatile("s_xor_b32 m0, %1, %6\n\ts_xor_b32 %0, %5, %7\n\tbuffer_load_dwordx4 %2, %4, %0 offen lds\n\t"
                 "s_xor_b32 m0, %1, %8\n\ts_xor_b32 %0, %5, %9\n\tbuffer_load_dwordx4 %3, %4, %0 offen lds"
                 : "=&s"(t) : "s"(lds), "v"(G < 4 ? v0 : v1), "v"(G + 1 < 4 ? v0 : v1), "s"(rs), "s"(soff),
                   "i"(G << 10), "i"(G << 15), "i"((G + 1) << 10), "i"((G + 1) << 15) : "memory", "scc");
    dma_w<G + 2, E>(lds, v0, v1, rs, soff);
  } else if constexpr (G < E) {
    unsigned t;
    asm volatile("s_xor_b32 m0, %1, %5\n\ts_xor_b32 %0, %4, %6\n\tbuffer_load_dwordx4 %2, %3, %0 offen lds"
                 : "=&s"(t) : "s"(lds), "v"(G < 4 ? v0 : v1), "s"(rs), "s"(soff), "i"(G << 10), "i"(G << 15) : "memory", "scc");
  }
}
__device__ __forceinline__ void touch(unsigned lds, unsigned voff, __amdgpu_buffer_rsrc_t rs, unsigned soff) {
  asm volatile("s_mov_b32 m0, %0\n\ts_nop 3\n\tbuffer_load_dword %1, %2, %3 offen lds" :: "s"(lds), "v"(voff), "s"(rs), "s"(rfl(soff)) : "memory");
}
template <unsigned LO, unsigned SO>
__device__ __forceinline__ void dma_a(unsigned lds, unsigned voff, __amdgpu_buffer_rsrc_t rs, unsigned soff) {
  unsigned t;
  asm volatile("s_add_u32 m0, %1, %5\n\ts_add_u32 %0, %4, %6\n\tbuffer_load_dwordx4 %2, %3, %0 offen lds"
               : "=&s"(t) : "s"(lds), "v"(voff), "s"(rs), "s"(soff), "i"(LO), "i"(SO) : "memory", "scc");
}
template <int NG>
__device__ __forceinline__ void dma_piece(unsigned lds, unsigned voff, __amdgpu_buffer_rsrc_t rs, unsigned soff) {
  unsigned t;
  asm volatile("s_mov_b32 m0, %1\n\ts_mov_b32 %0, %4\n\t"
               "buffer_load_dwordx4 %2, %3, %0 offen sc0 sc1 lds\n\t"
               "buffer_load_dwordx4 %2, %3, %0 offen offset:1024 sc0 sc1 lds\n\t"
               "buffer_load_dwordx4 %2, %3, %0 offen offset:2048 sc0 sc1 lds\n\t"
               "buffer_load_dwordx4 %2, %3, %0 offen offset:3072 sc0 sc1 lds"
               : "=&s"(t) : "s"(lds), "v"(voff), "s"(rs), "s"(soff) : "memory");
  if constexpr (NG == 5)
    asm volatile("s_add_u32 m0, %1, 0x1000\n\ts_add_u32 %0, %4, 0x1000\n\tbuffer_load_dwordx4 %2, %3, %0 offen sc0 sc1 lds"
                 : "=&s"(t) : "s"(lds), "v"(voff), "s"(rs), "s"(soff) : "memory", "scc");
}
__device__ __forceinline__ i32x8 cat(u32x4 a, u32x4 b) {
  return __builtin_bit_cast(i32x8, __builtin_shufflevector(a, b, 0, 1, 2, 3, 4, 5, 6, 7));
}

constexpr int sendN(int NG, int w) {
  int n = 0;
  for (int t = w; t < w + SWIN; ++t) {
    if (t > w && t < 8 * NG) ++n;
    if (t + SWIN + 1 < 8 * NG) ++n;
    if (t - SWIN >= 0 && (t - SWIN + 1) % NG == 0) ++n;
  }
  return n;
}
template <int NG> struct SD {
  u32x4 v[8 * NG];
  __amdgpu_buffer_rsrc_t rsX;
  unsigned vx, sx, lv, call;
  unsigned char* dst;
  unsigned* flag;
};
template <int NG, int S> __device__ __forceinline__ void send_load(SD<NG>& d) {
  asm volatile("buffer_load_dwordx4 %0, %1, %2, %3 offen" : "=v"(d.v[S])
               : "v"(d.vx), "s"(d.rsX), "s"(d.sx + ((d.lv >> (3 * (S / NG))) & 7) * 64 + (S % NG) * 16 * KS) : "memory");
}
template <int NG, int S> __device__ __forceinline__ void send_first(SD<NG>& d) {
  send_load<NG, S>(d);
  if constexpr (S < SWIN) send_first<NG, S + 1>(d);
}
template <int NG, int S> __device__ __forceinline__ void send_iter(SD<NG>& d) {
  constexpr int ST = 8 * NG, W = S - SWIN;
  if constexpr (W >= 0) {
    constexpr int N = sendN(NG, W);
    VMWAIT(N);
  }
  if constexpr (S < ST) {
    if constexpr (W < 0) asm volatile("s_waitcnt vmcnt(%1)" : "+v"(d.v[S]) : "i"(SWIN + S) : "memory");
    else asm volatile("" : "+v"(d.v[S]));
    asm volatile("global_store_dwordx4 %0, %1, off sc0 sc1\n\ts_nop 1" :: "v"(d.dst + (S / NG) * 32768 + (S % NG) * 1024), "v"(d.v[S]) : "memory");
    if constexpr (S + SWIN + 1 < ST) send_load<NG, S + SWIN + 1>(d);
  }
  if constexpr (W >= 0 && (W + 1) % NG == 0) {
    const unsigned val = (d.call - 1) * 8 + W / NG + 1;
    asm volatile("s_mov_b64 exec, %2\n\tglobal_store_dword %0, %1, off sc0 sc1\n\ts_mov_b64 exec, -1"
                 :: "v"(d.flag), "v"(val), "i"(NG == 5 ? 3 : 1) : "memory");
  }
  if constexpr (S + 1 < ST + SWIN) send_iter<NG, S + 1>(d);
}
template <int NG>
__device__ __forceinline__ unsigned send_stream(const unsigned char* x, const Stream& e, int rank, GAS unsigned* cnt, int lane) {
  static_assert(2 * SWIN + 1 <= 8 * NG, "window");
  SD<NG> d;
  d.rsX = __builtin_amdgcn_make_buffer_rsrc((void*)x, (short)0, NN * KS, 0x00020000);
  d.vx = (lane >> 2) * KS + (lane & 3) * 16; d.sx = (e.pos & 4095) * KS; d.lv = e.lv;
  unsigned c0;
  asm volatile("global_load_dword %0, %1, off sc1" : "=v"(c0) : "v"(cnt) : "memory");
  send_first<NG, 0>(d);
  SCHED();
  d.flag = (unsigned*)(e.peer + OFF_FLAG) + (e.pos >> 17) + (lane & 1);
  unsigned char* dst0 = e.peer + OFF_DATA + (size_t)rank * 8 * 32768 + ((e.pos >> 12) & 31) * 1024 + lane * 16;
  asm volatile("s_waitcnt vmcnt(%1)" : "+v"(c0) : "i"(SWIN + 1) : "memory");
  d.call = c0 + 1;
  d.dst = dst0 + (d.call & 1) * SLOT_BYTES;
  send_iter<NG, 0>(d);
  return d.call;
}

template <int NG> struct CW {
  f32x4 acc[NG][2];
  i32x8 bf[32][2];
  i32x8 af[NG];
  LAS unsigned char* rb[2][2];
  LAS unsigned char* ra[2];
  unsigned wr;
  LAS unsigned char* rl[2];
};
template <int NG, int U, int CT> __device__ __forceinline__ void read_b(CW<NG>& c) {
  constexpr int S = (2 * witem(U)) % NB;
  LAS unsigned char* p = c.rb[CT][S >> 3] + (S & 7) * 8192;
  i32x2 a = __builtin_amdgcn_ds_read_tr8_b64_v2i32((LAS i32x2*)(p));
  i32x2 b = __builtin_amdgcn_ds_read_tr8_b64_v2i32((LAS i32x2*)(p + 1024));
  i32x2 e = __builtin_amdgcn_ds_read_tr8_b64_v2i32((LAS i32x2*)(p + 2048));
  i32x2 d = __builtin_amdgcn_ds_read_tr8_b64_v2i32((LAS i32x2*)(p + 3072));
  i32x8 r; r[0] = a[0]; r[1] = a[1]; r[2] = b[0]; r[3] = b[1]; r[4] = e[0]; r[5] = e[1]; r[6] = d[0]; r[7] = d[1];
  c.bf[U][CT] = r;
}
template <int NG, int U, int M> __device__ __forceinline__ void read_a(CW<NG>& c) {
  constexpr int O = ((2 * U) % NA) * 5120 + M * (U < 4 ? 2048 : 1024);
  if constexpr (U < 4) c.af[M] = cat(*(LAS u32x4*)(c.rl[0] + O), *(LAS u32x4*)(c.rl[1] + O));
  else c.af[M] = cat(*(LAS u32x4*)(c.ra[0] + O), *(LAS u32x4*)(c.ra[1] + O));
}
template <int NG, int U, int M> __device__ __forceinline__ void read_a_all(CW<NG>& c) {
  read_a<NG, U, M>(c);
  if constexpr (M + 1 < NG) read_a_all<NG, U, M + 1>(c);
}
template <int NG, int U, int E> __device__ __forceinline__ void prefill(CW<NG>& c) {
  asm volatile("s_waitcnt lgkmcnt(0)" ::: "memory");
  TICK();
  read_b<NG, U, 0>(c); read_b<NG, U, 1>(c);
  SCHED();
  if constexpr (U + 4 < E) prefill<NG, U + 4, E>(c);
}
template <int NG, int M> __device__ __forceinline__ void put(CW<NG>& c) {
  *(LAS f32x4*)(size_t)(c.wr + M * 8192) = c.acc[M][0];
  *(LAS f32x4*)(size_t)((c.wr ^ 64u) + M * 8192) = c.acc[M][1];
}
template <int NG, int U, int M> __device__ __forceinline__ void row_tile(CW<NG>& c) {
  constexpr bool RA = U + 1 < 32, RB = RA && !fixed(U + 1);
  c.acc[M][0] = __builtin_amdgcn_mfma_scale_f32_16x16x128_f8f6f4(c.bf[U][0], c.af[M], c.acc[M][0], 1, 1, 0, 0, 0, 0);
  SCHED();
  if constexpr (RA && M > 0) read_a<NG, U + 1, M - 1>(c);
  SCHED();
  c.acc[M][1] = __builtin_amdgcn_mfma_scale_f32_16x16x128_f8f6f4(c.bf[U][1], c.af[M], c.acc[M][1], 1, 1, 0, 0, 0, 0);
  SCHED();
  if constexpr (M == 0 && RA) TICK();
  if constexpr (RB && (M == 1 || M == 2)) read_b<NG, U + 1, M - 1>(c);
  if constexpr (RA && M == NG - 1) read_a<NG, U + 1, M>(c);
  if constexpr (U == 31 && M > 0) put<NG, M - 1>(c);
  if constexpr (U == 31 && M == NG - 1) put<NG, M>(c);
  SCHED();
  if constexpr (M + 1 < NG) row_tile<NG, U, M + 1>(c);
}
template <int NG, int U, int E> __device__ __forceinline__ void steps(CW<NG>& c) {
  row_tile<NG, U, 0>(c);
  if constexpr (U + 1 < E) steps<NG, U + 1, E>(c);
}
template <int NG, int U> __device__ __forceinline__ void start(CW<NG>& c) {
  TICK();
  read_b<NG, U, 0>(c); read_b<NG, U, 1>(c);
  SCHED();
  read_a_all<NG, U, 0>(c);
  SCHED();
}

struct LD {
  __amdgpu_buffer_rsrc_t rsA, rsF, rsW;
  GAS unsigned* tmo;
  volatile LAS u32x2* seen;
  unsigned va, aoff, lda0, lda, vw0, vw1, woff, ldw0, ldw, scratch, vflag, need, call;
  u64 arrived, tp;
  u64 want, fix;
  unsigned dead;
  unsigned sox, sow;
  bool poll;
  int lane;
};
__device__ __forceinline__ void poll_use(LD& x, u32x2 r) {
  x.arrived |= __ballot((int)(r[0] - x.need) > 0 && (int)(r[1] - x.need) > 0) & ~0xffull;
}
__device__ __forceinline__ void poll_async(LD& x) {
  asm volatile("s_mov_b32 m0, %0\n\ts_nop 0\n\tbuffer_load_dwordx4 %1, %2, 0 offen sc0 sc1 lds"
               :: "s"(x.scratch), "v"(x.vflag), "s"(x.rsF) : "memory");
}
template <bool FIX, bool LOOKF = false> __device__ __forceinline__ void choose(LD& x) {
  if (FIX) {
    const unsigned q = __builtin_ctzll(x.fix | (1ull << 63));
    const bool ok = (x.arrived >> q) & 1;
    const unsigned so = __builtin_amdgcn_readlane(x.aoff, q);
    x.sox = ok ? so : OOR;
    x.fix &= ~(x.arrived & (1ull << q));
    x.poll = !ok;
  } else {
    const u64 c = x.arrived & x.want;
    const unsigned q = __builtin_ctzll(c | (1ull << 63));
    const unsigned so = __builtin_amdgcn_readlane(x.aoff, q);
    x.sow = __builtin_amdgcn_readlane(x.woff, q);
    x.sox = c ? so : OOR;
    x.poll = (__builtin_popcountll(c) <= 2 && x.arrived != ~0xffull) || (LOOKF && (~x.arrived & x.fix & -x.fix) != 0);
    x.want &= ~(c & (1ull << q));
  }
}
template <bool FIX> __device__ __forceinline__ void choose_wait(LD& x) {
  if (__builtin_expect(x.sox == OOR, 0)) {
    poll_use(x, *x.seen);
    choose<FIX>(x);
    while (x.sox == OOR) {
      u32x2 r;
      asm volatile("buffer_load_dwordx2 %0, %1, %2, 0 offen sc0 sc1\n\ts_waitcnt vmcnt(0)" : "=&v"(r) : "v"(x.vflag), "s"(x.rsF) : "memory");
      *x.seen = r;
      poll_use(x, r);
      if (__ballot(x.lane < 8 && r[0] == x.call) != 0 || rt() - x.tp > DEADLINE_TICKS) {
        __hip_atomic_store(x.tmo, x.call, __ATOMIC_RELAXED, __HIP_MEMORY_SCOPE_AGENT);
        x.dead = OOR; x.arrived = ~0xffull;
      }
      choose<FIX>(x);
    }
  }
}
__device__ __forceinline__ void next_w(LD& x) { x.ldw = x.ldw == x.ldw0 + 8 * 8192 ? x.ldw0 : x.ldw + 4 * 8192; }
__device__ __forceinline__ void load_w(LD& x, unsigned so) {
  dma_w<0, 8>(x.ldw, x.vw0, x.vw1, x.rsW, so);
  next_w(x);
}
template <int NG, bool FIX, bool NEXTFIX> __device__ __forceinline__ void item(LD& x) {
  constexpr int HEAD = 6;
  choose_wait<FIX>(x);
  dma_piece<NG>(x.lda, x.va, x.rsA, x.sox | x.dead);
  if (!FIX) dma_w<0, HEAD>(x.ldw, x.vw0, x.vw1, x.rsW, x.sow);
  const u32x2 r = *x.seen;
  VMWAIT(NG + (FIX ? 0 : HEAD));
  TICK();
  if (!FIX) dma_w<HEAD, 8>(x.ldw, x.vw0, x.vw1, x.rsW, x.sow);
  SCHED();
  x.lda = x.lda == x.lda0 + 8 * 5120 ? x.lda0 : x.lda + 4 * 5120;
  next_w(x);
  poll_use(x, r);
  choose<NEXTFIX, FIX>(x);
  SCHED();
  if (__builtin_expect(x.poll, 0)) poll_async(x);
  TICK();
}

struct Tile { const unsigned char* ax; const unsigned char* aw; float* aout; unsigned char* myws; int rank, b, tile, nt, sx, slow, help; unsigned xt; };

template <int NG>
__device__ __forceinline__ void tile_body(const Args& a, const Tile& t, LAS unsigned char* L, const int wave, const int lane) {
  const unsigned lbase = (unsigned)(size_t)L;
  LAS unsigned* ctrl = (LAS unsigned*)(L + CTRL_OFF);
  const int rank = t.rank, g0 = tile_g0(t.tile);
  GAS unsigned* cnt = (GAS unsigned*)(t.myws + OFF_CNT + (size_t)t.b * 128);
  unsigned call = 0;
  if (wave >= 4) {
    __builtin_amdgcn_s_setprio(2);
    const unsigned j = wave & 1;
    const unsigned P = (wave >> 1) & 1;
    unsigned c0;
    asm volatile("global_load_dword %0, %1, off sc1" : "=v"(c0) : "v"(cnt) : "memory");
    LD x;
    const unsigned rot = (t.tile + 2 * wave) & 7;
    x.rsW = __builtin_amdgcn_make_buffer_rsrc((void*)t.aw, (short)0, NN * NN, 0x00020000);
    const unsigned wcol = t.nt * 128 + (rot << 15);
    x.vw0 = (lane >> 3) * NN + (((lane & 7) ^ ((lane >> 3) & 6) ^ (rot >> 2)) << 4); x.vw1 = x.vw0 ^ 16u;
    x.ldw = x.ldw0 = (lbase + (2 * P + j) * 8192) | (rot << 10);
    constexpr u64 W0 = wseq(fq(0, 0), fq(2, 0), fq(4, 0), fq(6, 0), 2, 6), W1 = wseq(fq(0, 1), fq(2, 1), fq(4, 1), fq(6, 1), 3, 7);
    constexpr u64 W2 = wseq(fq(1, 0), fq(3, 0), fq(5, 0), 0, 4, 0), W3 = wseq(fq(1, 1), fq(3, 1), fq(5, 1), 1, 5, 0);
    u64 ws = P ? (j ? W3 : W2) : (j ? W1 : W0);
    auto wnext = [&]() {
      const unsigned k = ws & 63;
      ws >>= 8;
      return (((k + rank * 8) & 63) + (k < 8 ? 0 : level_add(t.tile))) * (64u * NN) + wcol;
    };
    load_w(x, wnext());
    load_w(x, wnext());
    SCHED();
    unsigned psrc, plvl;
    {
      const unsigned p = lane - 8, i = (p * 37u) >> 8, sr = p - i * 7u;
      const unsigned lo = sr < 3 ? sr : sr - 3, hi = sr < 3 ? sr + 4 : sr;
      const unsigned kl = i == lo ? 0 : i == hi ? 1 : 2 + i - (i > lo) - (i > hi);
      psrc = (rank + 1 + sr) & 7u;
      plvl = i;
      x.woff = (psrc * 8 + ((kl + level_add(t.tile)) & 7)) * (64u * NN) + wcol;
    }
    x.rsA = __builtin_amdgcn_make_buffer_rsrc((void*)(t.myws + OFF_DATA), (short)0, (int)(2 * SLOT_BYTES), 0x00020000);
    x.rsF = __builtin_amdgcn_make_buffer_rsrc((void*)t.myws, (short)0, (int)OFF_DATA, 0x00020000);
    x.tmo = (GAS unsigned*)(t.myws + OFF_TMO);
    x.va = (lane >> 2) * 64 + (((lane & 3) ^ (lane >> 5)) << 4);
    x.vflag = lane >= 8 ? (unsigned)OFF_FLAG + (t.tile * 8 + psrc) * 128 : (unsigned)OFF_TMO;
    x.lane = lane; x.arrived = 0; x.dead = 0;
    x.lda0 = lbase + LDS_A + (2 * (1 - P) + j) * 5120; x.lda = x.lda0 + 8 * 5120;
    x.scratch = lbase + CTRL_OFF + 128;
    x.seen = (volatile LAS u32x2*)(L + CTRL_OFF + 128 + lane * 16);
    if (wave == 4) *x.seen = u32x2{0, 0};
    if (P) TICK();
    asm volatile("s_waitcnt vmcnt(8) lgkmcnt(0)" : "+v"(c0) :: "memory");
    TICK();
    x.call = call = rfl(c0) + 1;
    x.need = (call - 1) * 8 + plvl;
    x.aoff = (call & 1) * (unsigned)SLOT_BYTES + (psrc * 8 + plvl) * 32768u + g0 * 1024;
    x.tp = rt();
    x.fix = j ? FM1 : FM0;
    x.want = P ? (j ? DM11 : DM10) : (j ? DM01 : DM00);
#pragma unroll 1
    for (unsigned i = P; i < 4; ++i) {
      load_w(x, wnext());
      TICK(); VMWAIT(8);
      if (wave == 4 && i == 1) poll_async(x);
      if (wave == 6 && i == 3) {
        poll_use(x, *x.seen);
        if ((x.arrived & 0x3fff00ull) != 0x3fff00ull) poll_async(x);
      }
      TICK();
    }
    TICK();
    poll_use(x, *x.seen);
    if (P) {
      choose<false>(x);
      if (x.poll && j == 0) poll_async(x);
#pragma unroll 1
      for (unsigned i = 0; i < 14; ++i) item<NG, false, false>(x);
    } else {
      choose<false, true>(x);
      if (x.poll) poll_async(x);
#pragma unroll 1
      for (unsigned i = 0; i < 7; ++i) { item<NG, false, true>(x); item<NG, true, false>(x); }
    }
    unsigned tw;
    asm volatile("global_load_dword %0, %1, off sc1" : "=v"(tw) : "v"(x.tmo) : "memory");
    VMWAIT(1);
    TICK();
    if (P) TICK();
    asm volatile("s_waitcnt vmcnt(0)" : "+v"(tw) :: "memory");
    *(volatile LAS unsigned*)(ctrl + C_STATE + wave - 4) = (x.dead || tw == call) ? 2u : 1u;
    __builtin_amdgcn_s_setprio(0);
  } else {
    CW<NG> c;
    const __amdgpu_buffer_rsrc_t rsX = __builtin_amdgcn_make_buffer_rsrc((void*)t.ax, (short)0, NN * KS, 0x00020000);
    if (wave == 1 || wave == 2) {
      const unsigned u = (t.xt >> (8 * (wave - 1))) & 255;
      if ((u >> 5) != (unsigned)rank) touch(lbase + LDS_JUNK, (lane >> 2) * KS + (lane & 3) * 128, rsX, u * 16 * KS);
    }
    if (wave == 0) {
      touch(lbase + LDS_JUNK, lane * KS, rsX, (rank * KS + g0 * 16) * KS + t.sx * 128);
      if constexpr (NG == 5) touch(lbase + LDS_JUNK, lane < 16 ? lane * KS : OOR, rsX, (rank * KS + g0 * 16 + 64) * KS + t.sx * 128);
    }
    if constexpr (NG == 4) {
      const __amdgpu_buffer_rsrc_t rsW = __builtin_amdgcn_make_buffer_rsrc((void*)t.aw, (short)0, NN * NN, 0x00020000);
      auto tw = [&](unsigned u, unsigned strip) {
        const unsigned k = u >> 1, s0 = k + 3 * (u & 1), sr = umin(s0, s0 - 7u);
        const unsigned ks = u < 14 ? ((rank + 1 + sr) & 7) * 8 + (k > (sr < 3 ? sr : sr - 3)) : rank * 8 + u - 14;
        if (u < 22) touch(lbase + LDS_JUNK, lane * NN, rsW, ks * (64u * NN) + strip * 128);
      };
      const unsigned u0 = wave * 3 + t.tile - 4;
      if (t.nt != t.slow) tw(14 + u0, t.nt);
      if (t.help && wave >= 2) tw(8 + (t.help - 1) * 6 + u0 - 6, t.slow);
    }
    auto local_x = [&]() {
      const unsigned v0 = (lane >> 3) * KS + (((lane & 7) ^ ((lane >> 4) & 1)) << 4), v1 = v0 ^ 64u;
      const unsigned so = (rank * KS + g0 * 16) * KS + wave * 128, ld = lbase + LDS_A + wave * 10240;
      dma_a<0 * 1024, 0 * 4096>(ld, v0, rsX, so); dma_a<1 * 1024, 1 * 4096>(ld, v1, rsX, so);
      dma_a<2 * 1024, 2 * 4096>(ld, v0, rsX, so); dma_a<3 * 1024, 3 * 4096>(ld, v1, rsX, so);
      dma_a<4 * 1024, 4 * 4096>(ld, v0, rsX, so); dma_a<5 * 1024, 5 * 4096>(ld, v1, rsX, so);
      dma_a<6 * 1024, 6 * 4096>(ld, v0, rsX, so); dma_a<7 * 1024, 7 * 4096>(ld, v1, rsX, so);
      if constexpr (NG == 5) { dma_a<8 * 1024, 8 * 4096>(ld, v0, rsX, so); dma_a<9 * 1024, 9 * 4096>(ld, v1, rsX, so); }
    };
    {
      const int g = lane >> 4, r = lane & 15;
      unsigned l0 = lbase + LDS_A + r * 128 + (((2 * g) ^ ((r >> 1) & 5)) << 4), l1 = l0 ^ 16u;
      asm volatile("" : "+v"(l0), "+v"(l1));
      c.rl[0] = (LAS unsigned char*)(size_t)l0; c.rl[1] = (LAS unsigned char*)(size_t)l1;
    }
    {
      const int g = lane >> 4, q = (lane & 15) >> 1, p = lane & 1, r = lane & 15;
      const unsigned b0 = lbase + g * 4096 + q * 128 + (((wave << 1) ^ (q & 6) ^ (g & 1)) << 4) + p * 8;
      unsigned b1 = b0 ^ 16u, b2 = b0 + 65536, b3 = b1 + 65536;
      asm volatile("" : "+v"(b2), "+v"(b3));
      c.rb[0][0] = (LAS unsigned char*)(size_t)b0; c.rb[1][0] = (LAS unsigned char*)(size_t)b1;
      c.rb[0][1] = (LAS unsigned char*)(size_t)b2; c.rb[1][1] = (LAS unsigned char*)(size_t)b3;
      unsigned a0 = lbase + LDS_A + (g >> 1) * 5120 + r * 64 + (((2 * (g & 1)) ^ (r >> 3)) << 4), a1 = a0 ^ 16u;
      asm volatile("" : "+v"(a0), "+v"(a1));
      c.ra[0] = (LAS unsigned char*)(size_t)a0; c.ra[1] = (LAS unsigned char*)(size_t)a1;
    }
#pragma unroll
    for (int m = 0; m < NG; ++m)
#pragma unroll
      for (int r = 0; r < 4; ++r) { c.acc[m][0][r] = 0.f; c.acc[m][1][r] = 0.f; }
    if (wave == 0) {
      const float sc = *(const __attribute__((address_space(4))) float*)a.sx * *(const __attribute__((address_space(4))) float*)a.sw;
      *(volatile LAS unsigned*)(ctrl + C_STATE + 4) = __builtin_bit_cast(unsigned, sc);
    }
    prefill<NG, 7, 19>(c);
    local_x();
    prefill<NG, 19, 32>(c);
    asm volatile("s_waitcnt vmcnt(0) lgkmcnt(0)" ::: "memory");
    start<NG, 0>(c);
    steps<NG, 0, 31>(c);
    c.wr = lbase + (lane & 15) * 512 + wave * 128 + (((lane >> 4) ^ (lane & 7)) << 4);
    SCHED();
    steps<NG, 31, 32>(c);
  }
  const __amdgpu_buffer_rsrc_t rsO = __builtin_amdgcn_make_buffer_rsrc((void*)t.aout, (short)0, KS * NN * 4, 0x00020000);
  const unsigned row = wave * 2 + (lane >> 5);
  unsigned vo = row * (NN * 4) + (lane & 31) * 16;
  const unsigned so = (g0 * 16 * NN + t.nt * 128) * 4;
  unsigned rd = lbase + row * 512 + (((lane & 31) ^ (row & 7)) << 4);
  asm volatile("" : "+v"(rd), "+v"(vo));
  asm volatile("s_waitcnt lgkmcnt(0)" ::: "memory");
  TICK();
  const u32x4 st = *(LAS u32x4*)(ctrl + C_STATE);
  const unsigned sbits = ctrl[C_STATE + 4];
  SCHED();
  f32x4 s[NG];
#pragma unroll
  for (int m = 0; m < NG; ++m) { s[m] = *(LAS f32x4*)(size_t)(rd + m * 8192); SCHED(); }
  const float scale = (st[0] & st[1] & st[2] & st[3]) == 1u ? __builtin_bit_cast(float, sbits) : __builtin_nanf("");
  const float scale2 = -1.44269504088896341f * scale;
#pragma unroll
  for (int m = 0; m < NG; ++m) {
    f32x4 e = s[m] * scale2;
    asm volatile("" : "+v"(e));
#pragma unroll
    for (int r = 0; r < 4; ++r) e[r] = __builtin_amdgcn_exp2f(__builtin_amdgcn_fmed3f(e[r], -86.5617024533378f, 86.5617024533378f));
    e += 1.f;
    asm volatile("" : "+v"(e));
#pragma unroll
    for (int r = 0; r < 4; ++r) e[r] = __builtin_amdgcn_rcpf(e[r]);
    const f32x4 z = (s[m] * scale) * e;
    __builtin_amdgcn_raw_buffer_store_b128(__builtin_bit_cast(u32x4, z), rsO, vo, so + m * 16 * (NN * 4), 16);
    SCHED();
    asm volatile("s_nop 1");
    SCHED();
  }
  if (wave == 4 && lane == 0) __hip_atomic_store(cnt, call, __ATOMIC_RELAXED, __HIP_MEMORY_SCOPE_AGENT);
}
}

__global__ void __launch_bounds__(512, 2) a2a_gemm_kernel(Args a) {
  extern __shared__ __attribute__((aligned(16))) unsigned char lds_raw[];
  const int wave = rfl(threadIdx.x >> 6), lane = threadIdx.x & 63;
  const int b = blockIdx.x;
  Tile t;
  t.ax = a.x; t.aw = a.w; t.aout = a.out; t.myws = a.myws; t.rank = a.rank; t.b = b;
  if (__builtin_expect(b < NSEND, 1)) {
    if (wave >= 4) return;
    Stream e = a.st[b * 4 + wave];
    asm volatile("" : "+s"(t.ax), "+s"(t.myws), "+s"(t.rank), "+s"(e.peer), "+s"(e.lv), "+s"(e.pos));
    if ((e.lv >> 24) == 0) return;
    GAS unsigned* cnt = (GAS unsigned*)(t.myws + OFF_CNT + (size_t)b * 128 + wave * 16);
    const unsigned call = (e.lv >> 24) == 2 ? send_stream<2>(t.ax, e, t.rank, cnt, lane) : send_stream<5>(t.ax, e, t.rank, cnt, lane);
    if (lane == 0) __hip_atomic_store(cnt, call, __ATOMIC_RELAXED, __HIP_MEMORY_SCOPE_AGENT);
    return;
  }
  t.xt = ((const unsigned*)a.xt)[b >> 1] >> (16 * (b & 1));
  asm volatile("" : "+s"(t.ax), "+s"(t.aw), "+s"(t.aout), "+s"(t.myws), "+s"(t.rank), "+s"(t.xt));
  const int cj = (b - NSEND) >> 3;
  t.tile = cj % NTILE; t.sx = cj / NTILE;
  t.nt = t.sx * 8 + ((b + 2 * t.sx) & 7);
  {
    const unsigned d = (3u - ((unsigned)(size_t)t.aw >> 7) - b) & 7u, ss = d >> 1;
    t.slow = (d & 1) ? -1 : ss * 8 + ((b + 2 * ss) & 7);
    t.help = (d & 1) ? 0 : (t.sx - ss) & 3;
  }
  if (t.tile < 4) tile_body<5>(a, t, (LAS unsigned char*)lds_raw, wave, lane);
  else tile_body<4>(a, t, (LAS unsigned char*)lds_raw, wave, lane);
}

extern "C" void kernel_launch(int rank, int nranks, ncclComm_t comm, hipStream_t stream,
                              void* const* d_in, const int* in_sizes, int n_in,
                              void* d_out, int out_size, const int* in_dtypes, int out_dtype,
                              void* const* d_out_all, void* const* d_ws_all, size_t ws_bytes) {
  if (nranks != NRANK || (unsigned)rank >= (unsigned)NRANK || ws_bytes < WS_NEED || n_in < 4 ||
      in_sizes[0] != NN * KS || in_sizes[1] != NN * NN || out_size != KS * NN) {
    fprintf(stderr, "a2a_gemm: unexpected problem shape\n");
    return;
  }
  static bool attr_set = false;
  if (!attr_set) {
    HIP_CHECK(hipFuncSetAttribute((const void*)a2a_gemm_kernel, hipFuncAttributeMaxDynamicSharedMemorySize, LDS_TOTAL));
    attr_set = true;
  }
  Args a;
  a.x = (const unsigned char*)d_in[0]; a.w = (const unsigned char*)d_in[1];
  a.sx = (const float*)d_in[2]; a.sw = (const float*)d_in[3];
  a.out = (float*)d_out;
  a.myws = (unsigned char*)d_ws_all[rank];
  a.rank = rank; a.pad[0] = a.pad[1] = a.pad[2] = 0;
  int nx[8] = {};
  for (int b = 0; b < NWG; ++b) a.xt[b] = (unsigned short)(rank * 32 * 257);
  for (int b = 0; b < NSEND; ++b)
    for (int w = 0; w < 4; ++w) {
      Stream& e = a.st[b * 4 + w];
      e.peer = nullptr; e.lv = 0; e.pos = 0;
      const bool five = b < 28 && w == 0;
      const int v = b < 28 ? b : b + 4 * w;
      if (!five && (v >= 42 || (b < 28 && w > 1))) continue;
      const int dd = five ? b % 7 : (v + 3) % 7, tile = five ? b / 7 : 4 + v / 14, half = five ? 0 : (v / 7) & 1;
      const int dest = (rank + 1 + dd) % NRANK, g0 = tile_g0(tile) + 2 * half, ng = five ? 5 : 2;
      e.peer = (unsigned char*)d_ws_all[dest];
      for (int i = 0; i < 8; ++i)
        e.lv |= (((levels(6 - dd) >> (3 * i)) + level_add(tile)) & 7u) << (3 * i);
      e.lv |= (unsigned)ng << 24;
      e.pos = (unsigned)(dest * KS + g0 * 16) | (unsigned)g0 << 12 | (unsigned)((tile * 8 + rank) * 32 + half) << 17;
      for (int g = 0; g < ng; ++g) {
        const int n = nx[b & 7]++, cb = NSEND + (n % 28) * 8 + (b & 7), sh = 8 * (n / 28);
        a.xt[cb] = (unsigned short)((a.xt[cb] & ~(255 << sh)) | (dest * 32 + g0 + g) << sh);
      }
    }
  a2a_gemm_kernel<<<NWG, 512, LDS_TOTAL, stream>>>(a);
  HIP_CHECK(hipGetLastError());
}
```

```cpp
#include <hip/hip_runtime.h>
#include <rccl/rccl.h>
#include <cstdio>

#define HIP_CHECK(expr) do { hipError_t _e = (expr); if (_e != hipSuccess) { \
    fprintf(stderr, "HIP error %s at %s:%d\n", hipGetErrorString(_e), __FILE__, __LINE__); \
    return; } } while (0)

#define LAS __attribute__((address_space(3)))
#define GAS __attribute__((address_space(1)))
typedef unsigned u32x2 __attribute__((ext_vector_type(2)));
typedef unsigned u32x4 __attribute__((ext_vector_type(4)));
typedef int i32x2 __attribute__((ext_vector_type(2)));
typedef int i32x8 __attribute__((ext_vector_type(8)));
typedef float f32x4 __attribute__((ext_vector_type(4)));
typedef unsigned long long u64;

#define SWIN (NG == 5 ? 7 : 2)

namespace {
constexpr int NRANK = 8;
constexpr int KS = 512;
constexpr int NN = 4096;
constexpr int NWG = 256;
constexpr int NSEND = 32;
constexpr int NTILE = 7;
constexpr size_t OFF_CNT = 0;
constexpr size_t OFF_TMO = 32768;
constexpr size_t OFF_FLAG = 65536;
constexpr size_t OFF_DATA = 1u << 20;
constexpr size_t SLOT_BYTES = 2u << 20;
constexpr size_t WS_NEED = OFF_DATA + 2 * SLOT_BYTES;
constexpr u64 DEADLINE_TICKS = 100000000ull;
constexpr int NB = 12;
constexpr int NA = 12;
constexpr int LDS_A = NB * 8192;
constexpr int CTRL_OFF = LDS_A + NA * 5120;
constexpr int LDS_JUNK = CTRL_OFF + 128 + 1024;
constexpr int LDS_TOTAL = LDS_JUNK + 256;
constexpr int C_STATE = 0;
constexpr int NF = 7;
constexpr bool fixed(int u) { return u >= 4 && (u & 3) == 3; }
constexpr int witem(int u) { return fixed(u) ? u / 4 - 1 : u + NF; }
constexpr int fq(int k, int j) { return 8 + 7 * k + (k + 3 * j) % 7; }
constexpr u64 fmask(int j) { u64 m = 0; for (int k = 0; k < NF; ++k) m |= 1ull << fq(k, j); return m; }
constexpr u64 dmask(int P, int j) {
  u64 m = 0;
  int u = 4, h = 0;
  for (int q = 8; q < 64; ++q) {
    if (((fmask(0) | fmask(1)) >> q) & 1) continue;
    if (((u + 7) & 1) == P && h == j) m |= 1ull << q;
    if (++h == 2) { h = 0; ++u; if (fixed(u)) ++u; }
  }
  return m;
}
constexpr unsigned levels(int s) {
  unsigned m = 0, nf = 0, nd = 2;
  for (int i = 0; i < 8; ++i) m |= ((((fmask(0) | fmask(1)) >> (8 + 7 * i + s)) & 1) ? nf++ : nd++) << (3 * i);
  return m;
}
__host__ __device__ constexpr unsigned level_add(int tile) { return tile < 4 ? 0 : 2 * tile - 6; }
constexpr int kfix(int q) { return q < 8 ? q : ((q - 8) % 7 + 1) * 8 + ((levels((q - 8) % 7) >> (3 * ((q - 8) / 7))) & 7); }
constexpr u64 wseq(int a, int b, int c, int d, int e, int f) {
  return (u64)kfix(a) | (u64)kfix(b) << 8 | (u64)kfix(c) << 16 | (u64)kfix(d) << 24 | (u64)kfix(e) << 32 | (u64)kfix(f) << 40;
}
constexpr u64 FM0 = fmask(0), FM1 = fmask(1), DM00 = dmask(0, 0), DM01 = dmask(0, 1), DM10 = dmask(1, 0), DM11 = dmask(1, 1);
constexpr unsigned OOR = 0x80000000u;
static_assert(NB == 12 && NA == 12, "ring schedule");

struct Stream {
  unsigned char* peer;
  unsigned lv;
  unsigned pos;
};
struct Args {
  const unsigned char* x; const unsigned char* w; const float* sx; const float* sw; float* out;
  unsigned char* myws;
  int rank; int pad[3];
  Stream st[NSEND * 4];
  unsigned short xt[NWG];
};

__device__ __forceinline__ u64 rt() { return __builtin_amdgcn_s_memrealtime(); }
__device__ __forceinline__ unsigned rfl(unsigned v) { return __builtin_amdgcn_readfirstlane(v); }
__device__ __forceinline__ unsigned umin(unsigned a, unsigned b) { return a < b ? a : b; }
#define SCHED() __builtin_amdgcn_sched_barrier(0)
#define TICK() do { asm volatile("" ::: "memory"); __builtin_amdgcn_s_barrier(); asm volatile("" ::: "memory"); } while (0)
#define VMWAIT(n) asm volatile("s_waitcnt vmcnt(%0)" :: "i"(n) : "memory")
__host__ __device__ constexpr int tile_g0(int t) { return t < 4 ? 5 * t : 4 * t + 4; }

__device__ __forceinline__ void dma_w_head(unsigned lds, const unsigned* v, __amdgpu_buffer_rsrc_t rs, unsigned soff) {
  unsigned t;
  asm volatile("s_mov_b32 m0, %1\n\ts_mov_b32 %0, %9\n\t"
               "buffer_load_dwordx4 %2, %8, %0 offen lds\n\tbuffer_load_dwordx4 %3, %8, %0 offen offset:1024 lds\n\t"
               "buffer_load_dwordx4 %4, %8, %0 offen offset:2048 lds\n\tbuffer_load_dwordx4 %5, %8, %0 offen offset:3072 lds\n\t"
               "s_xor_b32 m0, %1, 0x1000\n\ts_nop 0\n\t"
               "buffer_load_dwordx4 %6, %8, %0 offen lds\n\tbuffer_load_dwordx4 %7, %8, %0 offen offset:1024 lds"
               : "=&s"(t) : "s"(lds), "v"(v[0]), "v"(v[1]), "v"(v[2]), "v"(v[3]), "v"(v[4]), "v"(v[5]), "s"(rs), "s"(soff) : "memory", "scc");
}
__device__ __forceinline__ void dma_w_tail(unsigned lds, const unsigned* v, __amdgpu_buffer_rsrc_t rs, unsigned soff) {
  unsigned t;
  asm volatile("s_xor_b32 m0, %1, 0x1000\n\ts_mov_b32 %0, %5\n\t"
               "buffer_load_dwordx4 %2, %4, %0 offen offset:2048 lds\n\tbuffer_load_dwordx4 %3, %4, %0 offen offset:3072 lds"
               : "=&s"(t) : "s"(lds), "v"(v[6]), "v"(v[7]), "s"(rs), "s"(soff) : "memory", "scc");
}
__device__ __forceinline__ void touch(unsigned lds, unsigned voff, __amdgpu_buffer_rsrc_t rs, unsigned soff) {
  asm volatile("s_mov_b32 m0, %0\n\ts_nop 3\n\tbuffer_load_dword %1, %2, %3 offen lds" :: "s"(lds), "v"(voff), "s"(rs), "s"(rfl(soff)) : "memory");
}
template <unsigned LO, unsigned SO>
__device__ __forceinline__ void dma_a(unsigned lds, unsigned voff, __amdgpu_buffer_rsrc_t rs, unsigned soff) {
  unsigned t;
  asm volatile("s_add_u32 m0, %1, %5\n\ts_add_u32 %0, %4, %6\n\tbuffer_load_dwordx4 %2, %3, %0 offen lds"
               : "=&s"(t) : "s"(lds), "v"(voff), "s"(rs), "s"(soff), "i"(LO), "i"(SO) : "memory", "scc");
}
template <int NG>
__device__ __forceinline__ void dma_piece(unsigned lds, unsigned voff, __amdgpu_buffer_rsrc_t rs, unsigned soff0, unsigned soff1) {
  unsigned t;
  if constexpr (NG == 5)
    asm volatile("s_mov_b32 m0, %1\n\ts_or_b32 %0, %4, %5\n\t"
                 "buffer_load_dwordx4 %2, %3, %0 offen sc0 sc1 lds\n\t"
                 "buffer_load_dwordx4 %2, %3, %0 offen offset:1024 sc0 sc1 lds\n\t"
                 "buffer_load_dwordx4 %2, %3, %0 offen offset:2048 sc0 sc1 lds\n\t"
                 "buffer_load_dwordx4 %2, %3, %0 offen offset:3072 sc0 sc1 lds\n\t"
                 "s_add_u32 m0, %1, 0x1000\n\ts_add_u32 %0, %0, 0x1000\n\tbuffer_load_dwordx4 %2, %3, %0 offen sc0 sc1 lds"
                 : "=&s"(t) : "s"(lds), "v"(voff), "s"(rs), "s"(soff0), "s"(soff1) : "memory", "scc");
  else
    asm volatile("s_mov_b32 m0, %1\n\ts_or_b32 %0, %4, %5\n\t"
                 "buffer_load_dwordx4 %2, %3, %0 offen sc0 sc1 lds\n\t"
                 "buffer_load_dwordx4 %2, %3, %0 offen offset:1024 sc0 sc1 lds\n\t"
                 "buffer_load_dwordx4 %2, %3, %0 offen offset:2048 sc0 sc1 lds\n\t"
                 "buffer_load_dwordx4 %2, %3, %0 offen offset:3072 sc0 sc1 lds"
                 : "=&s"(t) : "s"(lds), "v"(voff), "s"(rs), "s"(soff0), "s"(soff1) : "memory", "scc");
}
__device__ __forceinline__ i32x8 cat(u32x4 a, u32x4 b) {
  return __builtin_bit_cast(i32x8, __builtin_shufflevector(a, b, 0, 1, 2, 3, 4, 5, 6, 7));
}

constexpr int sendN(int NG, int w) {
  int n = 0;
  for (int t = w; t < w + SWIN; ++t) {
    if (t > w && t < 8 * NG) ++n;
    if (t + SWIN + 1 < 8 * NG) ++n;
    if (t - SWIN >= 0 && (t - SWIN + 1) % NG == 0) ++n;
  }
  return n;
}
template <int NG> struct SD {
  u32x4 v[8 * NG];
  __amdgpu_buffer_rsrc_t rsX;
  unsigned vx, sx, lv, call;
  unsigned char* dst;
  unsigned* flag;
};
template <int NG, int S> __device__ __forceinline__ void send_load(SD<NG>& d) {
  asm volatile("buffer_load_dwordx4 %0, %1, %2, %3 offen" : "=v"(d.v[S])
               : "v"(d.vx), "s"(d.rsX), "s"(d.sx + ((d.lv >> (3 * (S / NG))) & 7) * 64 + (S % NG) * 16 * KS) : "memory");
}
template <int NG, int S> __device__ __forceinline__ void send_first(SD<NG>& d) {
  send_load<NG, S>(d);
  if constexpr (S < SWIN) send_first<NG, S + 1>(d);
}
template <int NG, int S> __device__ __forceinline__ void send_iter(SD<NG>& d) {
  constexpr int ST = 8 * NG, W = S - SWIN;
  if constexpr (W >= 0) {
    constexpr int N = sendN(NG, W);
    VMWAIT(N);
  }
  if constexpr (S < ST) {
    if constexpr (W < 0) asm volatile("s_waitcnt vmcnt(%1)" : "+v"(d.v[S]) : "i"(SWIN + S) : "memory");
    else asm volatile("" : "+v"(d.v[S]));
    asm volatile("global_store_dwordx4 %0, %1, off sc0 sc1\n\ts_nop 1" :: "v"(d.dst + (S / NG) * 32768 + (S % NG) * 1024), "v"(d.v[S]) : "memory");
    if constexpr (S + SWIN + 1 < ST) send_load<NG, S + SWIN + 1>(d);
  }
  if constexpr (W >= 0 && (W + 1) % NG == 0) {
    const unsigned val = (d.call - 1) * 8 + W / NG + 1;
    asm volatile("s_mov_b64 exec, %2\n\tglobal_store_dword %0, %1, off sc0 sc1\n\ts_mov_b64 exec, -1"
                 :: "v"(d.flag), "v"(val), "i"(NG == 5 ? 3 : 1) : "memory");
  }
  if constexpr (S + 1 < ST + SWIN) send_iter<NG, S + 1>(d);
}
template <int NG>
__device__ __forceinline__ unsigned send_stream(const unsigned char* x, const Stream& e, int rank, GAS unsigned* cnt, int lane) {
  static_assert(2 * SWIN + 1 <= 8 * NG, "window");
  SD<NG> d;
  d.rsX = __builtin_amdgcn_make_buffer_rsrc((void*)x, (short)0, NN * KS, 0x00020000);
  d.vx = (lane >> 2) * KS + (lane & 3) * 16; d.sx = (e.pos & 4095) * KS; d.lv = e.lv;
  unsigned c0;
  asm volatile("global_load_dword %0, %1, off sc1" : "=v"(c0) : "v"(cnt) : "memory");
  send_first<NG, 0>(d);
  SCHED();
  d.flag = (unsigned*)(e.peer + OFF_FLAG) + (e.pos >> 17) + (lane & 1);
  unsigned char* dst0 = e.peer + OFF_DATA + (size_t)rank * 8 * 32768 + ((e.pos >> 12) & 31) * 1024 + lane * 16;
  asm volatile("s_waitcnt vmcnt(%1)" : "+v"(c0) : "i"(SWIN + 1) : "memory");
  d.call = c0 + 1;
  d.dst = dst0 + (d.call & 1) * SLOT_BYTES;
  send_iter<NG, 0>(d);
  return d.call;
}

template <int NG> struct CW {
  f32x4 acc[NG][2];
  i32x8 bf[32][2];
  i32x8 af[NG];
  LAS unsigned char* rb[2][2];
  LAS unsigned char* ra[2];
  unsigned wr;
  LAS unsigned char* rl[2];
};
template <int NG, int U, int CT> __device__ __forceinline__ void read_b(CW<NG>& c) {
  constexpr int S = (2 * witem(U)) % NB;
  LAS unsigned char* p = c.rb[CT][S >> 3] + (S & 7) * 8192;
  i32x2 a = __builtin_amdgcn_ds_read_tr8_b64_v2i32((LAS i32x2*)(p));
  i32x2 b = __builtin_amdgcn_ds_read_tr8_b64_v2i32((LAS i32x2*)(p + 1024));
  i32x2 e = __builtin_amdgcn_ds_read_tr8_b64_v2i32((LAS i32x2*)(p + 2048));
  i32x2 d = __builtin_amdgcn_ds_read_tr8_b64_v2i32((LAS i32x2*)(p + 3072));
  i32x8 r; r[0] = a[0]; r[1] = a[1]; r[2] = b[0]; r[3] = b[1]; r[4] = e[0]; r[5] = e[1]; r[6] = d[0]; r[7] = d[1];
  c.bf[U][CT] = r;
}
template <int NG, int U, int M> __device__ __forceinline__ void read_a(CW<NG>& c) {
  constexpr int O = ((2 * U) % NA) * 5120 + M * (U < 4 ? 2048 : 1024);
  if constexpr (U < 4) c.af[M] = cat(*(LAS u32x4*)(c.rl[0] + O), *(LAS u32x4*)(c.rl[1] + O));
  else c.af[M] = cat(*(LAS u32x4*)(c.ra[0] + O), *(LAS u32x4*)(c.ra[1] + O));
}
template <int NG, int U, int M> __device__ __forceinline__ void read_a_all(CW<NG>& c) {
  read_a<NG, U, M>(c);
  if constexpr (M + 1 < NG) read_a_all<NG, U, M + 1>(c);
}
template <int NG, int U, int E> __device__ __forceinline__ void prefill(CW<NG>& c) {
  asm volatile("s_waitcnt lgkmcnt(0)" ::: "memory");
  TICK();
  read_b<NG, U, 0>(c); read_b<NG, U, 1>(c);
  SCHED();
  if constexpr (U + 4 < E) prefill<NG, U + 4, E>(c);
}
template <int NG, int M> __device__ __forceinline__ void put(CW<NG>& c) {
  *(LAS f32x4*)(size_t)(c.wr + M * 8192) = c.acc[M][0];
  *(LAS f32x4*)(size_t)((c.wr ^ 64u) + M * 8192) = c.acc[M][1];
}
template <int NG, int U, int M> __device__ __forceinline__ void row_tile(CW<NG>& c) {
  constexpr bool RA = U + 1 < 32, RB = RA && !fixed(U + 1);
  c.acc[M][0] = __builtin_amdgcn_mfma_scale_f32_16x16x128_f8f6f4(c.bf[U][0], c.af[M], c.acc[M][0], 1, 1, 0, 0, 0, 0);
  SCHED();
  if constexpr (RA && M > 0) read_a<NG, U + 1, M - 1>(c);
  SCHED();
  c.acc[M][1] = __builtin_amdgcn_mfma_scale_f32_16x16x128_f8f6f4(c.bf[U][1], c.af[M], c.acc[M][1], 1, 1, 0, 0, 0, 0);
  SCHED();
  if constexpr (M == 0 && RA) TICK();
  if constexpr (RB && (M == 1 || M == 2)) read_b<NG, U + 1, M - 1>(c);
  if constexpr (RA && M == NG - 1) read_a<NG, U + 1, M>(c);
  if constexpr (U == 31 && M > 0) put<NG, M - 1>(c);
  if constexpr (U == 31 && M == NG - 1) put<NG, M>(c);
  SCHED();
  if constexpr (M + 1 < NG) row_tile<NG, U, M + 1>(c);
}
template <int NG, int U, int E> __device__ __forceinline__ void steps(CW<NG>& c) {
  row_tile<NG, U, 0>(c);
  if constexpr (U + 1 < E) steps<NG, U + 1, E>(c);
}
template <int NG, int U> __device__ __forceinline__ void start(CW<NG>& c) {
  TICK();
  read_b<NG, U, 0>(c); read_b<NG, U, 1>(c);
  SCHED();
  read_a_all<NG, U, 0>(c);
  SCHED();
}

struct LD {
  __amdgpu_buffer_rsrc_t rsA, rsF, rsW;
  GAS unsigned* tmo;
  volatile LAS u32x2* seen;
  unsigned va, aoff, lda0, lda, vw[8], woff, ldw0, ldw, scratch, vflag, need, call;
  u64 arrived, tp;
  u64 want, fix;
  unsigned dead;
  unsigned sox, sow;
  bool poll;
  int lane;
};
__device__ __forceinline__ void poll_use(LD& x, u32x2 r) {
  x.arrived |= __ballot((int)(r[0] - x.need) > 0 && (int)(r[1] - x.need) > 0) & ~0xffull;
}
__device__ __forceinline__ void poll_async(LD& x) {
  asm volatile("s_mov_b32 m0, %0\n\ts_nop 0\n\tbuffer_load_dwordx4 %1, %2, 0 offen sc0 sc1 lds"
               :: "s"(x.scratch), "v"(x.vflag), "s"(x.rsF) : "memory");
}
template <bool FIX, bool LOOKF = false> __device__ __forceinline__ void choose(LD& x) {
  if (FIX) {
    const unsigned q = __builtin_ctzll(x.fix | (1ull << 63));
    const bool ok = (x.arrived >> q) & 1;
    const unsigned so = __builtin_amdgcn_readlane(x.aoff, q);
    x.sox = ok ? so : OOR;
    x.fix &= ~(x.arrived & (1ull << q));
    x.poll = !ok;
  } else {
    const u64 c = x.arrived & x.want;
    const unsigned q = __builtin_ctzll(c | (1ull << 63));
    const unsigned so = __builtin_amdgcn_readlane(x.aoff, q);
    x.sow = __builtin_amdgcn_readlane(x.woff, q);
    x.sox = c ? so : OOR;
    x.poll = (__builtin_popcountll(c) <= 2 && x.arrived != ~0xffull) || (LOOKF && (~x.arrived & x.fix & -x.fix) != 0);
    x.want &= ~(c & (1ull << q));
  }
}
template <bool FIX> __device__ __forceinline__ void choose_wait(LD& x) {
  if (__builtin_expect(x.sox == OOR, 0)) {
    poll_use(x, *x.seen);
    choose<FIX>(x);
    while (x.sox == OOR) {
      u32x2 r;
      asm volatile("buffer_load_dwordx2 %0, %1, %2, 0 offen sc0 sc1\n\ts_waitcnt vmcnt(0)" : "=&v"(r) : "v"(x.vflag), "s"(x.rsF) : "memory");
      *x.seen = r;
      poll_use(x, r);
      if (__ballot(x.lane < 8 && r[0] == x.call) != 0 || rt() - x.tp > DEADLINE_TICKS) {
        __hip_atomic_store(x.tmo, x.call, __ATOMIC_RELAXED, __HIP_MEMORY_SCOPE_AGENT);
        x.dead = OOR; x.arrived = ~0xffull;
      }
      choose<FIX>(x);
    }
  }
}
__device__ __forceinline__ void next_w(LD& x) { x.ldw = x.ldw == x.ldw0 + 8 * 8192 ? x.ldw0 : x.ldw + 4 * 8192; }
__device__ __forceinline__ void load_w(LD& x, unsigned so) {
  dma_w_head(x.ldw, x.vw, x.rsW, so);
  dma_w_tail(x.ldw, x.vw, x.rsW, so);
  next_w(x);
}
template <int NG, bool FIX, bool NEXTFIX> __device__ __forceinline__ void item(LD& x) {
  choose_wait<FIX>(x);
  dma_piece<NG>(x.lda, x.va, x.rsA, x.sox, x.dead);
  if (!FIX) dma_w_head(x.ldw, x.vw, x.rsW, x.sow);
  const u32x2 r = *x.seen;
  VMWAIT(NG + (FIX ? 0 : 6));
  TICK();
  if (!FIX) dma_w_tail(x.ldw, x.vw, x.rsW, x.sow);
  SCHED();
  x.lda = x.lda == x.lda0 + 8 * 5120 ? x.lda0 : x.lda + 4 * 5120;
  next_w(x);
  poll_use(x, r);
  choose<NEXTFIX, FIX>(x);
  SCHED();
  if (__builtin_expect(x.poll, 0)) poll_async(x);
  asm volatile("" : "+s"(x.lda), "+s"(x.ldw), "+s"(x.sox), "+s"(x.sow), "+s"(x.want), "+s"(x.fix));
  TICK();
}

struct Tile { const unsigned char* ax; const unsigned char* aw; float* aout; unsigned char* myws; int rank, b, tile, nt, sx, slow, help; unsigned xt; };

template <int NG>
__device__ __forceinline__ void tile_body(const Args& a, const Tile& t, LAS unsigned char* L, const int wave, const int lane) {
  const unsigned lbase = (unsigned)(size_t)L;
  LAS unsigned* ctrl = (LAS unsigned*)(L + CTRL_OFF);
  const int rank = t.rank, g0 = tile_g0(t.tile);
  GAS unsigned* cnt = (GAS unsigned*)(t.myws + OFF_CNT + (size_t)t.b * 128);
  unsigned call = 0;
  if (wave >= 4) {
    __builtin_amdgcn_s_setprio(2);
    const unsigned j = wave & 1;
    const unsigned P = (wave >> 1) & 1;
    unsigned c0;
    asm volatile("global_load_dword %0, %1, off sc1" : "=v"(c0) : "v"(cnt) : "memory");
    LD x;
    const unsigned rot = (t.tile + wave) & 1;
    x.rsW = __builtin_amdgcn_make_buffer_rsrc((void*)t.aw, (short)0, NN * NN, 0x00020000);
    const unsigned wcol = t.nt * 128;
#pragma unroll
    for (int g = 0; g < 8; ++g) {
      const unsigned h = (g >> 2) ^ rot;
      x.vw[g] = (h * 32 + (g & 3) * 8 + (lane >> 3)) * NN + (((lane & 7) ^ ((lane >> 3) & 6) ^ h) << 4) - (g & 3) * 1024;
    }
    x.ldw = x.ldw0 = (lbase + (2 * P + j) * 8192) | (rot << 12);
    constexpr u64 W0 = wseq(fq(0, 0), fq(2, 0), fq(4, 0), fq(6, 0), 2, 6), W1 = wseq(fq(0, 1), fq(2, 1), fq(4, 1), fq(6, 1), 3, 7);
    constexpr u64 W2 = wseq(fq(1, 0), fq(3, 0), fq(5, 0), 0, 4, 0), W3 = wseq(fq(1, 1), fq(3, 1), fq(5, 1), 1, 5, 0);
    u64 ws = P ? (j ? W3 : W2) : (j ? W1 : W0);
    auto wnext = [&]() {
      const unsigned k = ws & 63;
      ws >>= 8;
      return (((k + rank * 8) & 63) + (k < 8 ? 0 : level_add(t.tile))) * (64u * NN) + wcol;
    };
    load_w(x, wnext());
    load_w(x, wnext());
    SCHED();
    unsigned psrc, plvl;
    {
      const unsigned p = lane - 8, i = (p * 37u) >> 8, sr = p - i * 7u;
      const unsigned lo = sr < 3 ? sr : sr - 3, hi = sr < 3 ? sr + 4 : sr;
      const unsigned kl = i == lo ? 0 : i == hi ? 1 : 2 + i - (i > lo) - (i > hi);
      psrc = (rank + 1 + sr) & 7u;
      plvl = i;
      x.woff = (psrc * 8 + ((kl + level_add(t.tile)) & 7)) * (64u * NN) + wcol;
    }
    x.rsA = __builtin_amdgcn_make_buffer_rsrc((void*)(t.myws + OFF_DATA), (short)0, (int)(2 * SLOT_BYTES), 0x00020000);
    x.rsF = __builtin_amdgcn_make_buffer_rsrc((void*)t.myws, (short)0, (int)OFF_DATA, 0x00020000);
    x.tmo = (GAS unsigned*)(t.myws + OFF_TMO);
    x.va = (lane >> 2) * 64 + (((lane & 3) ^ (lane >> 5)) << 4);
    x.vflag = lane >= 8 ? (unsigned)OFF_FLAG + (t.tile * 8 + psrc) * 128 : (unsigned)OFF_TMO;
    x.lane = lane; x.arrived = 0; x.dead = 0;
    x.lda0 = lbase + LDS_A + (2 * (1 - P) + j) * 5120; x.lda = x.lda0 + 8 * 5120;
    x.scratch = lbase + CTRL_OFF + 128;
    x.seen = (volatile LAS u32x2*)(L + CTRL_OFF + 128 + lane * 16);
    if (wave == 4) *x.seen = u32x2{0, 0};
    if (P) TICK();
    asm volatile("s_waitcnt vmcnt(8) lgkmcnt(0)" : "+v"(c0) :: "memory");
    TICK();
    x.call = call = rfl(c0) + 1;
    x.need = (call - 1) * 8 + plvl;
    x.aoff = (call & 1) * (unsigned)SLOT_BYTES + (psrc * 8 + plvl) * 32768u + g0 * 1024;
    x.tp = rt();
    x.fix = j ? FM1 : FM0;
    x.want = P ? (j ? DM11 : DM10) : (j ? DM01 : DM00);
    asm volatile("" : "+s"(x.want), "+s"(x.fix), "+s"(x.rsA), "+s"(x.rsF), "+s"(x.lda0), "+s"(x.lda), "+s"(x.tmo), "+s"(x.call), "+s"(x.scratch));
    asm volatile("" : "+v"(x.va), "+v"(x.aoff), "+v"(x.woff), "+v"(x.need), "+v"(x.seen), "+v"(x.vflag));
#pragma unroll 1
    for (unsigned i = P; i < 4; ++i) {
      load_w(x, wnext());
      TICK(); VMWAIT(8);
      if (wave == 4 && i == 1) poll_async(x);
      if (wave == 6 && i == 3) {
        poll_use(x, *x.seen);
        if ((x.arrived & 0x3fff00ull) != 0x3fff00ull) poll_async(x);
      }
      TICK();
    }
    poll_use(x, *x.seen);
    if (P) {
      choose<false>(x);
      if (x.poll && j == 0) poll_async(x);
      asm volatile("" : "+s"(x.lda), "+s"(x.ldw), "+s"(x.sox), "+s"(x.sow), "+s"(x.want), "+s"(x.arrived), "+s"(x.dead));
      TICK();
#pragma unroll 1
      for (unsigned i = 0; i < 7; ++i) { item<NG, false, false>(x); item<NG, false, false>(x); }
    } else {
      choose<false, true>(x);
      if (x.poll) poll_async(x);
      asm volatile("" : "+s"(x.lda), "+s"(x.ldw), "+s"(x.sox), "+s"(x.sow), "+s"(x.want), "+s"(x.arrived), "+s"(x.dead));
      TICK();
#pragma unroll 1
      for (unsigned i = 0; i < 7; ++i) { item<NG, false, true>(x); item<NG, true, false>(x); }
    }
    unsigned tw;
    asm volatile("global_load_dword %0, %1, off sc1" : "=v"(tw) : "v"(x.tmo) : "memory");
    VMWAIT(1);
    TICK();
    if (P) TICK();
    asm volatile("s_waitcnt vmcnt(0)" : "+v"(tw) :: "memory");
    *(volatile LAS unsigned*)(ctrl + C_STATE + wave - 4) = (x.dead || tw == call) ? 2u : 1u;
    __builtin_amdgcn_s_setprio(0);
  } else {
    CW<NG> c;
    const __amdgpu_buffer_rsrc_t rsX = __builtin_amdgcn_make_buffer_rsrc((void*)t.ax, (short)0, NN * KS, 0x00020000);
    if (wave == 1 || wave == 2) {
      const unsigned u = (t.xt >> (8 * (wave - 1))) & 255;
      if ((u >> 5) != (unsigned)rank) touch(lbase + LDS_JUNK, (lane >> 2) * KS + (lane & 3) * 128, rsX, u * 16 * KS);
    }
    if (wave == 0) {
      touch(lbase + LDS_JUNK, lane * KS, rsX, (rank * KS + g0 * 16) * KS + t.sx * 128);
      if constexpr (NG == 5) touch(lbase + LDS_JUNK, lane < 16 ? lane * KS : OOR, rsX, (rank * KS + g0 * 16 + 64) * KS + t.sx * 128);
    }
    if constexpr (NG == 4) {
      const __amdgpu_buffer_rsrc_t rsW = __builtin_amdgcn_make_buffer_rsrc((void*)t.aw, (short)0, NN * NN, 0x00020000);
      auto tw = [&](unsigned u, unsigned strip) {
        const unsigned k = u >> 1, s0 = k + 3 * (u & 1), sr = umin(s0, s0 - 7u);
        const unsigned ks = u < 14 ? ((rank + 1 + sr) & 7) * 8 + (k > (sr < 3 ? sr : sr - 3)) : rank * 8 + u - 14;
        if (u < 22) touch(lbase + LDS_JUNK, lane * NN, rsW, ks * (64u * NN) + strip * 128);
      };
      const unsigned u0 = wave * 3 + t.tile - 4;
      if (t.nt != t.slow) tw(14 + u0, t.nt);
      if (t.help && wave >= 2) tw(8 + (t.help - 1) * 6 + u0 - 6, t.slow);
    }
    auto local_x = [&]() {
      const unsigned v0 = (lane >> 3) * KS + (((lane & 7) ^ ((lane >> 4) & 1)) << 4), v1 = v0 ^ 64u;
      const unsigned so = (rank * KS + g0 * 16) * KS + wave * 128, ld = lbase + LDS_A + wave * 10240;
      dma_a<0 * 1024, 0 * 4096>(ld, v0, rsX, so); dma_a<1 * 1024, 1 * 4096>(ld, v1, rsX, so);
      dma_a<2 * 1024, 2 * 4096>(ld, v0, rsX, so); dma_a<3 * 1024, 3 * 4096>(ld, v1, rsX, so);
      dma_a<4 * 1024, 4 * 4096>(ld, v0, rsX, so); dma_a<5 * 1024, 5 * 4096>(ld, v1, rsX, so);
      dma_a<6 * 1024, 6 * 4096>(ld, v0, rsX, so); dma_a<7 * 1024, 7 * 4096>(ld, v1, rsX, so);
      if constexpr (NG == 5) { dma_a<8 * 1024, 8 * 4096>(ld, v0, rsX, so); dma_a<9 * 1024, 9 * 4096>(ld, v1, rsX, so); }
    };
    {
      const int g = lane >> 4, r = lane & 15;
      unsigned l0 = lbase + LDS_A + r * 128 + (((2 * g) ^ ((r >> 1) & 5)) << 4), l1 = l0 ^ 16u;
      asm volatile("" : "+v"(l0), "+v"(l1));
      c.rl[0] = (LAS unsigned char*)(size_t)l0; c.rl[1] = (LAS unsigned char*)(size_t)l1;
    }
    {
      const int g = lane >> 4, q = (lane & 15) >> 1, p = lane & 1, r = lane & 15;
      const unsigned b0 = lbase + g * 4096 + q * 128 + (((wave << 1) ^ (q & 6) ^ (g & 1)) << 4) + p * 8;
      unsigned b1 = b0 ^ 16u, b2 = b0 + 65536, b3 = b1 + 65536;
      asm volatile("" : "+v"(b2), "+v"(b3));
      c.rb[0][0] = (LAS unsigned char*)(size_t)b0; c.rb[1][0] = (LAS unsigned char*)(size_t)b1;
      c.rb[0][1] = (LAS unsigned char*)(size_t)b2; c.rb[1][1] = (LAS unsigned char*)(size_t)b3;
      unsigned a0 = lbase + LDS_A + (g >> 1) * 5120 + r * 64 + (((2 * (g & 1)) ^ (r >> 3)) << 4), a1 = a0 ^ 16u;
      asm volatile("" : "+v"(a0), "+v"(a1));
      c.ra[0] = (LAS unsigned char*)(size_t)a0; c.ra[1] = (LAS unsigned char*)(size_t)a1;
    }
#pragma unroll
    for (int m = 0; m < NG; ++m)
#pragma unroll
      for (int r = 0; r < 4; ++r) { c.acc[m][0][r] = 0.f; c.acc[m][1][r] = 0.f; }
    if (wave == 0) {
      const float sc = *(const __attribute__((address_space(4))) float*)a.sx * *(const __attribute__((address_space(4))) float*)a.sw;
      *(volatile LAS unsigned*)(ctrl + C_STATE + 4) = __builtin_bit_cast(unsigned, sc);
    }
    prefill<NG, 7, 19>(c);
    local_x();
    prefill<NG, 19, 32>(c);
    asm volatile("s_waitcnt vmcnt(0) lgkmcnt(0)" ::: "memory");
    start<NG, 0>(c);
    steps<NG, 0, 31>(c);
    c.wr = lbase + (lane & 15) * 512 + wave * 128 + (((lane >> 4) ^ (lane & 7)) << 4);
    SCHED();
    steps<NG, 31, 32>(c);
  }
  const __amdgpu_buffer_rsrc_t rsO = __builtin_amdgcn_make_buffer_rsrc((void*)t.aout, (short)0, KS * NN * 4, 0x00020000);
  const unsigned row = wave * 2 + (lane >> 5);
  unsigned vo = row * (NN * 4) + (lane & 31) * 16;
  const unsigned so = (g0 * 16 * NN + t.nt * 128) * 4;
  unsigned rd = lbase + row * 512 + (((lane & 31) ^ (row & 7)) << 4);
  asm volatile("" : "+v"(rd), "+v"(vo));
  asm volatile("s_waitcnt lgkmcnt(0)" ::: "memory");
  TICK();
  const u32x4 st = *(LAS u32x4*)(ctrl + C_STATE);
  const unsigned sbits = ctrl[C_STATE + 4];
  SCHED();
  f32x4 s[NG];
#pragma unroll
  for (int m = 0; m < NG; ++m) { s[m] = *(LAS f32x4*)(size_t)(rd + m * 8192); SCHED(); }
  const float scale = (st[0] & st[1] & st[2] & st[3]) == 1u ? __builtin_bit_cast(float, sbits) : __builtin_nanf("");
  const float scale2 = -1.44269504088896341f * scale;
#pragma unroll
  for (int m = 0; m < NG; ++m) {
    f32x4 e = s[m] * scale2;
    asm volatile("" : "+v"(e));
#pragma unroll
    for (int r = 0; r < 4; ++r) e[r] = __builtin_amdgcn_exp2f(__builtin_amdgcn_fmed3f(e[r], -86.5617024533378f, 86.5617024533378f));
    e += 1.f;
    asm volatile("" : "+v"(e));
#pragma unroll
    for (int r = 0; r < 4; ++r) e[r] = __builtin_amdgcn_rcpf(e[r]);
    const f32x4 z = (s[m] * scale) * e;
    __builtin_amdgcn_raw_buffer_store_b128(__builtin_bit_cast(u32x4, z), rsO, vo, so + m * 16 * (NN * 4), 16);
    SCHED();
    asm volatile("s_nop 1");
    SCHED();
  }
  if (wave == 4 && lane == 0) __hip_atomic_store(cnt, call, __ATOMIC_RELAXED, __HIP_MEMORY_SCOPE_AGENT);
}
}

__global__ void __launch_bounds__(512, 2) a2a_gemm_kernel(Args a) {
  extern __shared__ __attribute__((aligned(16))) unsigned char lds_raw[];
  const int wave = rfl(threadIdx.x >> 6), lane = threadIdx.x & 63;
  const int b = blockIdx.x;
  Tile t;
  t.ax = a.x; t.aw = a.w; t.aout = a.out; t.myws = a.myws; t.rank = a.rank; t.b = b;
  if (__builtin_expect(b < NSEND, 1)) {
    if (wave >= 4) return;
    Stream e = a.st[b * 4 + wave];
    asm volatile("" : "+s"(t.ax), "+s"(t.myws), "+s"(t.rank), "+s"(e.peer), "+s"(e.lv), "+s"(e.pos));
    if ((e.lv >> 24) == 0) return;
    GAS unsigned* cnt = (GAS unsigned*)(t.myws + OFF_CNT + (size_t)b * 128 + wave * 16);
    const unsigned call = (e.lv >> 24) == 2 ? send_stream<2>(t.ax, e, t.rank, cnt, lane) : send_stream<5>(t.ax, e, t.rank, cnt, lane);
    if (lane == 0) __hip_atomic_store(cnt, call, __ATOMIC_RELAXED, __HIP_MEMORY_SCOPE_AGENT);
    return;
  }
  t.xt = ((const unsigned*)a.xt)[b >> 1] >> (16 * (b & 1));
  asm volatile("" : "+s"(t.ax), "+s"(t.aw), "+s"(t.aout), "+s"(t.myws), "+s"(t.rank), "+s"(t.xt));
  const int cj = (b - NSEND) >> 3;
  t.tile = cj % NTILE; t.sx = cj / NTILE;
  t.nt = t.sx * 8 + ((b + 2 * t.sx) & 7);
  {
    const unsigned d = (3u - ((unsigned)(size_t)t.aw >> 7) - b) & 7u, ss = d >> 1;
    t.slow = (d & 1) ? -1 : ss * 8 + ((b + 2 * ss) & 7);
    t.help = (d & 1) ? 0 : (t.sx - ss) & 3;
  }
  if (t.tile < 4) tile_body<5>(a, t, (LAS unsigned char*)lds_raw, wave, lane);
  else tile_body<4>(a, t, (LAS unsigned char*)lds_raw, wave, lane);
}

extern "C" void kernel_launch(int rank, int nranks, ncclComm_t comm, hipStream_t stream,
                              void* const* d_in, const int* in_sizes, int n_in,
                              void* d_out, int out_size, const int* in_dtypes, int out_dtype,
                              void* const* d_out_all, void* const* d_ws_all, size_t ws_bytes) {
  if (nranks != NRANK || (unsigned)rank >= (unsigned)NRANK || ws_bytes < WS_NEED || n_in < 4 ||
      in_sizes[0] != NN * KS || in_sizes[1] != NN * NN || out_size != KS * NN) {
    fprintf(stderr, "a2a_gemm: unexpected problem shape\n");
    return;
  }
  static bool attr_set = false;
  if (!attr_set) {
    HIP_CHECK(hipFuncSetAttribute((const void*)a2a_gemm_kernel, hipFuncAttributeMaxDynamicSharedMemorySize, LDS_TOTAL));
    attr_set = true;
  }
  Args a;
  a.x = (const unsigned char*)d_in[0]; a.w = (const unsigned char*)d_in[1];
  a.sx = (const float*)d_in[2]; a.sw = (const float*)d_in[3];
  a.out = (float*)d_out;
  a.myws = (unsigned char*)d_ws_all[rank];
  a.rank = rank; a.pad[0] = a.pad[1] = a.pad[2] = 0;
  int nx[8] = {};
  for (int b = 0; b < NWG; ++b) a.xt[b] = (unsigned short)(rank * 32 * 257);
  for (int b = 0; b < NSEND; ++b)
    for (int w = 0; w < 4; ++w) {
      Stream& e = a.st[b * 4 + w];
      e.peer = nullptr; e.lv = 0; e.pos = 0;
      const bool five = b < 28 && w == 0;
      const int v = b < 28 ? b : b + 4 * w;
      if (!five && (v >= 42 || (b < 28 && w > 1))) continue;
      const int dd = five ? b % 7 : (v + 3) % 7, tile = five ? b / 7 : 4 + v / 14, half = five ? 0 : (v / 7) & 1;
      const int dest = (rank + 1 + dd) % NRANK, g0 = tile_g0(tile) + 2 * half, ng = five ? 5 : 2;
      e.peer = (unsigned char*)d_ws_all[dest];
      for (int i = 0; i < 8; ++i)
        e.lv |= (((levels(6 - dd) >> (3 * i)) + level_add(tile)) & 7u) << (3 * i);
      e.lv |= (unsigned)ng << 24;
      e.pos = (unsigned)(dest * KS + g0 * 16) | (unsigned)g0 << 12 | (unsigned)((tile * 8 + rank) * 32 + half) << 17;
      for (int g = 0; g < ng; ++g) {
        const int n = nx[b & 7]++, cb = NSEND + (n % 28) * 8 + (b & 7), sh = 8 * (n / 28);
        a.xt[cb] = (unsigned short)((a.xt[cb] & ~(255 << sh)) | (dest * 32 + g0 + g) << sh);
      }
    }
  a2a_gemm_kernel<<<NWG, 512, LDS_TOTAL, stream>>>(a);
  HIP_CHECK(hipGetLastError());
}
```
